```python
import numpy as np

kernel_name = "gpu_a2a_gemm_m4096_k4096_n4096_e4m3e4m3_dqrelu"


ABSMAX_RTOL = 0.03

COMPUTE_DTYPE = "float32"
EPILOGUE = "dequant_relu"
IN_QUANT = None
OUT_QUANT = None
IN_FP8 = "e4m3e4m3"
IN_DTYPES = {"x": "e4m3", "w_mat": "e4m3",
             "scale_x": "float32", "scale_w": "float32"}
WORLD_SIZE = 8

def _snap_fp8(x, mant_bits, max_abs, min_normal):
    a = np.abs(x.astype(np.float64))
    _m, _e = np.frexp(a)
    q = np.where(a >= min_normal, np.ldexp(1.0, _e - 1 - mant_bits),
                 min_normal * 2.0 ** -mant_bits)
    snapped = np.minimum(np.round(a / q) * q, max_abs)
    return (np.sign(x) * snapped).astype(np.float32)

def setup_inputs():

    x = _snap_fp8(np.random.randn(4096, 4096).astype(np.float32), 3, 448.0, 2.0 ** -6)
    w_mat = _snap_fp8(np.random.randn(4096, 4096).astype(np.float32), 3, 448.0, 2.0 ** -6)
    scale_x = (0.02 * (1.0 + np.random.rand(1))).astype(np.float32)
    scale_w = (0.02 * (1.0 + np.random.rand(1))).astype(np.float32)
    return {"x": x, "w_mat": w_mat, "scale_x": scale_x, "scale_w": scale_w}

def reference(x, w_mat, scale_x, scale_w):
    acc = (x @ w_mat).astype(np.float32)
    y = (acc * (scale_x * scale_w)).astype(np.float32)
    return np.maximum(y, 0.0).astype(np.float32)
```

```cpp
#include <hip/hip_runtime.h>
#include <rccl/rccl.h>
#include <cstdio>
#include <cstdint>

constexpr size_t WS_END = 8454400;

struct Args {
  const uint8_t* w; const uint8_t* xown; uint8_t* myws; float* out;
  const float* sx; const float* sw; const uint8_t* x; unsigned rank, colrot;
  uint8_t tab[28][4][8];
  struct { uint16_t item[4][4]; uint8_t* peer; unsigned misc, src; } svc[32];
};

__global__ __launch_bounds__(256, 1) void a2a_gemm(Args a) {
  __shared__ __attribute__((aligned(1024))) uint8_t lds[163840];
  asm volatile(
      "s_load_dwordx16 s[8:23], %0, 0x0\n\t"
      "s_memrealtime s[68:69]\n\t"
      "v_readfirstlane_b32 s48, %2\n\t"
      "v_and_b32 v2, 63, %2\n\t"
      "s_lshr_b32 s48, s48, 6\n\t"
      "s_cmp_lt_u32 %1, 32\n\t"
      "s_cbranch_scc0 CMP_%=\n\t"
      "SVC_%=:\n\t"
      "s_cmp_lg_u32 s48, 3\n\t"
      "s_cbranch_scc1 CPE_%=\n\t"
      "s_getpc_b64 s[40:41]\n\t"
      "CP0_%=:\n\t"
      "s_and_b32 s41, s41, 0xffff\n\t"
      "s_mov_b32 s42, KEND_%=-CP0_%=\n\t"
      "s_mov_b32 s43, 0x20000\n\t"
      "s_lshr_b32 s54, %1, 3\n\t"
      "s_lshl_b32 s54, s54, 13\n\t"
      "v_lshlrev_b32 v28, 7, v2\n\t"
      "buffer_load_dword v29, v28, s[40:43], s54 offen\n\t"
      "s_add_u32 s54, s54, 0x8000\n\t"
      "buffer_load_dword v29, v28, s[40:43], s54 offen\n\t"
      "CPE_%=:\n\t"
      "s_mul_i32 s54, %1, 48\n\t"
      "s_load_dwordx4 s[92:95], %0, s54 offset:0x3e0\n\t"
      "s_lshl_b32 s55, s48, 3\n\t"
      "s_add_u32 s54, s54, s55\n\t"
      "s_load_dwordx2 s[88:89], %0, s54 offset:0x3c0\n\t"
      "s_mul_i32 s51, s48, 40960\n\t"
      "s_mov_b32 s35, 0x20000\n\t"
      "s_mov_b32 s39, 0x20000\n\t"
      "s_mov_b32 s47, 0x20000\n\t"
      "s_mov_b32 s31, 0x20000\n\t"
      "v_lshrrev_b32 v11, 3, v2\n\t"
      "v_and_b32 v26, 6, v11\n\t"
      "v_xor_b32 v26, v26, v2\n\t"
      "v_and_b32 v26, 7, v26\n\t"
      "v_lshlrev_b32 v26, 4, v26\n\t"
      "v_lshl_or_b32 v3, v11, 9, v26\n\t"
      "v_cmp_gt_u32 vcc, 2, v11\n\t"
      "v_bfrev_b32 v4, 1\n\t"
      "s_nop 0\n\t"
      "v_cndmask_b32 v4, v4, v3, vcc\n\t"
      "s_lshl_b32 s53, %1, 8\n\t"
      "s_lshl_b32 s54, s48, 2\n\t"
      "s_add_u32 s53, s53, s54\n\t"
      "s_add_u32 s53, s53, 256\n\t"
      "v_mov_b32 v24, 0\n\t"
      "v_lshrrev_b32 v11, 4, v2\n\t"
      "v_and_b32 v26, 15, v2\n\t"
      "v_and_b32 v27, 6, v26\n\t"
      "v_xor_b32 v27, v27, v11\n\t"
      "v_lshlrev_b32 v27, 4, v27\n\t"
      "v_lshl_or_b32 v5, v26, 7, v27\n\t"
      "v_add_u32 v5, s51, v5\n\t"
      "v_xor_b32 v6, 64, v5\n\t"
      "v_and_b32 v26, 3, v2\n\t"
      "v_lshl_or_b32 v26, v11, 2, v26\n\t"
      "v_and_or_b32 v27, v2, 1, 8\n\t"
      "v_cmp_gt_u32 vcc, 32, v2\n\t"
      "v_bfe_u32 v11, v2, 2, 2\n\t"
      "v_bfe_u32 v22, v2, 1, 2\n\t"
      "v_cndmask_b32 v26, v27, v26, vcc\n\t"
      "v_cndmask_b32 v11, v22, v11, vcc\n\t"
      "v_and_b32 v27, 6, v26\n\t"
      "v_xor_b32 v27, v27, v11\n\t"
      "v_lshlrev_b32 v27, 4, v27\n\t"
      "v_lshl_or_b32 v22, v26, 7, v27\n\t"
      "v_add_u32 v22, s51, v22\n\t"
      "v_xor_b32 v23, 64, v22\n\t"
      "v_lshlrev_b32 v7, 4, v2\n\t"
      "v_cmp_gt_u32 vcc, 40, v2\n\t"
      "v_bfrev_b32 v27, 1\n\t"
      "s_nop 0\n\t"
      "v_cndmask_b32 v8, v27, v7, vcc\n\t"
      "s_waitcnt lgkmcnt(0)\n\t"
      "s_bitcmp1_b32 s94, 6\n\t"
      "s_cbranch_scc1 PREF_%=\n\t"
      "s_cmp_lg_u32 s48, 3\n\t"
      "s_cbranch_scc1 SWE_%=\n\t"
      "s_mov_b32 s54, 10112\n\t"
      "s_load_dword s16, s[92:93], s54\n\t"
      "s_mov_b32 s54, 655360\n\t"
      "s_load_dword s16, s[92:93], s54\n\t"
      "SWE_%=:\n\t"
      "s_cmp_eq_u32 s48, 0\n\t"
      "s_cbranch_scc1 SFIRST_%=\n\t"
      "s_mov_b32 s54, s48\n\t"
      "SSTAG_%=:\n\t"
      "s_sleep 3\n\t"
      "s_sub_u32 s54, s54, 1\n\t"
      "s_cmp_lg_u32 s54, 0\n\t"
      "s_cbranch_scc1 SSTAG_%=\n\t"
      "SFIRST_%=:\n\t"
      "s_bfe_u32 s61, s88, 131072\n\t"
      "s_bfe_u32 s49, s88, 196610\n\t"
      "s_bfe_u32 s52, s88, 65541\n\t"
      "s_mul_i32 s54, s49, 37376\n\t"
      "s_add_u32 s54, s54, s95\n\t"
      "s_lshl_b32 s55, s61, 7\n\t"
      "s_add_u32 s54, s54, s55\n\t"
      "s_lshl_b32 s55, s52, 14\n\t"
      "s_add_u32 s54, s54, s55\n\t"
      "s_add_u32 s36, s20, s54\n\t"
      "s_addc_u32 s37, s21, 0\n\t"
      "s_and_b32 s37, s37, 0xffff\n\t"
      "s_mul_i32 s38, s52, 5120\n\t"
      "s_add_u32 s38, s38, 16000\n\t"
      "s_mov_b32 s55, 0\n\t"
      "s_add_u32 m0, s51, 0\n\t"
      "s_nop 0\n\t"
      "buffer_load_dwordx4 v3, s[36:39], s55 offen lds\n\t"
      "s_add_u32 m0, s51, 1024\n\t"
      "s_add_u32 s55, s55, 0x1000\n\t"
      "buffer_load_dwordx4 v3, s[36:39], s55 offen lds\n\t"
      "s_add_u32 m0, s51, 2048\n\t"
      "s_add_u32 s55, s55, 0x1000\n\t"
      "buffer_load_dwordx4 v3, s[36:39], s55 offen lds\n\t"
      "s_add_u32 m0, s51, 3072\n\t"
      "s_add_u32 s55, s55, 0x1000\n\t"
      "buffer_load_dwordx4 v3, s[36:39], s55 offen lds\n\t"
      "s_add_u32 m0, s51, 4096\n\t"
      "s_add_u32 s55, s55, 0x1000\n\t"
      "buffer_load_dwordx4 v3, s[36:39], s55 offen lds\n\t"
      "s_add_u32 m0, s51, 5120\n\t"
      "s_add_u32 s55, s55, 0x1000\n\t"
      "buffer_load_dwordx4 v4, s[36:39], s55 offen lds\n\t"
      "s_add_u32 s44, s12, 8388608\n\t"
      "s_addc_u32 s45, s13, 0\n\t"
      "s_and_b32 s45, s45, 0xffff\n\t"
      "s_mov_b32 s46, 65792\n\t"
      "buffer_load_dword v25, v24, s[44:47], s53 offen sc1\n\t"
      "s_bfe_u32 s54, s94, 0x80008\n\t"
      "s_add_u32 s65, s68, s54\n\t"
      "s_sleep 6\n\t"
      "s_bfe_u32 s61, s88, 131088\n\t"
      "s_bfe_u32 s49, s88, 196626\n\t"
      "s_bfe_u32 s52, s88, 65557\n\t"
      "s_mul_i32 s54, s49, 37376\n\t"
      "s_add_u32 s54, s54, s95\n\t"
      "s_lshl_b32 s55, s61, 7\n\t"
      "s_add_u32 s54, s54, s55\n\t"
      "s_lshl_b32 s55, s52, 14\n\t"
      "s_add_u32 s54, s54, s55\n\t"
      "s_add_u32 s36, s20, s54\n\t"
      "s_addc_u32 s37, s21, 0\n\t"
      "s_and_b32 s37, s37, 0xffff\n\t"
      "s_mul_i32 s38, s52, 5120\n\t"
      "s_add_u32 s38, s38, 16000\n\t"
      "s_mov_b32 s55, 0\n\t"
      "s_add_u32 m0, s51, 8192\n\t"
      "s_nop 0\n\t"
      "buffer_load_dwordx4 v3, s[36:39], s55 offen lds\n\t"
      "s_add_u32 m0, s51, 9216\n\t"
      "s_add_u32 s55, s55, 0x1000\n\t"
      "buffer_load_dwordx4 v3, s[36:39], s55 offen lds\n\t"
      "s_add_u32 m0, s51, 10240\n\t"
      "s_add_u32 s55, s55, 0x1000\n\t"
      "buffer_load_dwordx4 v3, s[36:39], s55 offen lds\n\t"
      "s_add_u32 m0, s51, 11264\n\t"
      "s_add_u32 s55, s55, 0x1000\n\t"
      "buffer_load_dwordx4 v3, s[36:39], s55 offen lds\n\t"
      "s_add_u32 m0, s51, 12288\n\t"
      "s_add_u32 s55, s55, 0x1000\n\t"
      "buffer_load_dwordx4 v3, s[36:39], s55 offen lds\n\t"
      "s_add_u32 m0, s51, 13312\n\t"
      "s_add_u32 s55, s55, 0x1000\n\t"
      "buffer_load_dwordx4 v4, s[36:39], s55 offen lds\n\t"
      "s_bfe_u32 s61, s88, 131072\n\t"
      "s_bfe_u32 s49, s88, 196610\n\t"
      "s_bfe_u32 s52, s88, 65541\n\t"
      "s_bfe_u32 s58, s88, 393222\n\t"
      "s_lshl_b32 s56, s49, 2\n\t"
      "s_add_u32 s56, s56, s61\n\t"
      "s_mul_i32 s56, s56, 10240\n\t"
      "s_lshl_b32 s54, s52, 7\n\t"
      "s_add_u32 s54, s54, s56\n\t"
      "s_add_u32 s54, s54, 8960\n\t"
      "v_mov_b32 v16, s54\n\t"
      "s_lshl_b32 s55, s52, 12\n\t"
      "s_add_u32 s56, s56, s55\n\t"
      "s_add_u32 s32, s92, s56\n\t"
      "s_addc_u32 s33, s93, 0\n\t"
      "s_and_b32 s33, s33, 0xffff\n\t"
      "s_mov_b32 s34, 333824\n\t"
      "s_and_b32 s55, s58, 3\n\t"
      "s_mul_i32 s54, s52, 10\n\t"
      "s_add_u32 s54, s54, 32\n\t"
      "s_mul_i32 s54, s54, s55\n\t"
      "s_lshl_b32 s55, s52, 7\n\t"
      "s_add_u32 s54, s54, s55\n\t"
      "s_lshr_b32 s55, s58, 3\n\t"
      "s_mul_i32 s55, s55, 296\n\t"
      "s_add_u32 s54, s54, s55\n\t"
      "s_mul_i32 s62, s54, 232\n\t"
      "s_lshr_b32 s62, s62, 10\n\t"
      "s_add_u32 s62, s62, s65\n\t"
      "SW0_%=:\n\t"
      "s_memrealtime s[70:71]\n\t"
      "s_waitcnt lgkmcnt(0)\n\t"
      "s_sub_u32 s70, s70, s62\n\t"
      "s_cmp_lt_i32 s70, 0\n\t"
      "s_cbranch_scc1 SW0_%=\n\t"
      "s_waitcnt vmcnt(6)\n\t"
      "ds_read_b128 v[72:75], v5 offset:0\n\t"
      "ds_read_b128 v[76:79], v6 offset:0\n\t"
      "ds_read_b128 v[80:83], v5 offset:2048\n\t"
      "ds_read_b128 v[84:87], v6 offset:2048\n\t"
      "ds_read_b128 v[88:91], v22 offset:4096\n\t"
      "ds_read_b128 v[92:95], v23 offset:4096\n\t"
      "v_readfirstlane_b32 s59, v25\n\t"
      "s_nop 0\n\t"
      "s_add_u32 s59, s59, 1\n\t"
      "v_mov_b32 v17, s59\n\t"
      "s_bitcmp1_b32 s59, 0\n\t"
      "s_cselect_b32 s50, 327680, 0\n\t"
      "s_add_u32 s64, s50, 0x1000\n\t"
      "s_waitcnt lgkmcnt(5)\n\t"
      "buffer_store_dwordx4 v[72:75], v7, s[32:35], s50 offen sc0 sc1\n\t"
      "s_waitcnt lgkmcnt(4)\n\t"
      "buffer_store_dwordx4 v[76:79], v7, s[32:35], s50 offen offset:1024 sc0 sc1\n\t"
      "s_waitcnt lgkmcnt(3)\n\t"
      "buffer_store_dwordx4 v[80:83], v7, s[32:35], s50 offen offset:2048 sc0 sc1\n\t"
      "s_waitcnt lgkmcnt(2)\n\t"
      "buffer_store_dwordx4 v[84:87], v7, s[32:35], s50 offen offset:3072 sc0 sc1\n\t"
      "s_waitcnt lgkmcnt(0)\n\t"
      "s_cmp_eq_u32 s52, 0\n\t"
      "s_cbranch_scc1 SH0_%=\n\t"
      "buffer_store_dwordx4 v[88:91], v8, s[32:35], s64 offen sc0 sc1\n\t"
      "buffer_store_dwordx4 v[92:95], v8, s[32:35], s64 offen offset:1024 sc0 sc1\n\t"
      "SH0_%=:\n\t"
      "v_add_u32 v18, s50, v16\n\t"
      "s_bfe_u32 s61, s88, 131088\n\t"
      "s_bfe_u32 s49, s88, 196626\n\t"
      "s_bfe_u32 s52, s88, 65557\n\t"
      "s_bfe_u32 s58, s88, 393238\n\t"
      "s_lshl_b32 s56, s49, 2\n\t"
      "s_add_u32 s56, s56, s61\n\t"
      "s_mul_i32 s56, s56, 10240\n\t"
      "s_lshl_b32 s54, s52, 7\n\t"
      "s_add_u32 s54, s54, s56\n\t"
      "s_add_u32 s54, s54, 8960\n\t"
      "v_mov_b32 v16, s54\n\t"
      "s_lshl_b32 s55, s52, 12\n\t"
      "s_add_u32 s56, s56, s55\n\t"
      "s_add_u32 s32, s92, s56\n\t"
      "s_addc_u32 s33, s93, 0\n\t"
      "s_and_b32 s33, s33, 0xffff\n\t"
      "s_mov_b32 s34, 333824\n\t"
      "s_and_b32 s55, s58, 3\n\t"
      "s_mul_i32 s54, s52, 10\n\t"
      "s_add_u32 s54, s54, 32\n\t"
      "s_mul_i32 s54, s54, s55\n\t"
      "s_lshl_b32 s55, s52, 7\n\t"
      "s_add_u32 s54, s54, s55\n\t"
      "s_lshr_b32 s55, s58, 3\n\t"
      "s_mul_i32 s55, s55, 296\n\t"
      "s_add_u32 s54, s54, s55\n\t"
      "s_mul_i32 s62, s54, 232\n\t"
      "s_lshr_b32 s62, s62, 10\n\t"
      "s_add_u32 s62, s62, s65\n\t"
      "s_mov_b64 exec, 1\n\t"
      "s_waitcnt vmcnt(0)\n\t"
      "global_store_dword v18, v17, s[92:93] sc0 sc1\n\t"
      "s_mov_b64 exec, -1\n\t"
      "s_bfe_u32 s61, s89, 131072\n\t"
      "s_bfe_u32 s49, s89, 196610\n\t"
      "s_bfe_u32 s52, s89, 65541\n\t"
      "s_mul_i32 s54, s49, 37376\n\t"
      "s_add_u32 s54, s54, s95\n\t"
      "s_lshl_b32 s55, s61, 7\n\t"
      "s_add_u32 s54, s54, s55\n\t"
      "s_lshl_b32 s55, s52, 14\n\t"
      "s_add_u32 s54, s54, s55\n\t"
      "s_add_u32 s36, s20, s54\n\t"
      "s_addc_u32 s37, s21, 0\n\t"
      "s_and_b32 s37, s37, 0xffff\n\t"
      "s_mul_i32 s38, s52, 5120\n\t"
      "s_add_u32 s38, s38, 16000\n\t"
      "s_mov_b32 s55, 0\n\t"
      "s_add_u32 m0, s51, 16384\n\t"
      "s_nop 0\n\t"
      "buffer_load_dwordx4 v3, s[36:39], s55 offen lds\n\t"
      "s_add_u32 m0, s51, 17408\n\t"
      "s_add_u32 s55, s55, 0x1000\n\t"
      "buffer_load_dwordx4 v3, s[36:39], s55 offen lds\n\t"
      "s_add_u32 m0, s51, 18432\n\t"
      "s_add_u32 s55, s55, 0x1000\n\t"
      "buffer_load_dwordx4 v3, s[36:39], s55 offen lds\n\t"
      "s_add_u32 m0, s51, 19456\n\t"
      "s_add_u32 s55, s55, 0x1000\n\t"
      "buffer_load_dwordx4 v3, s[36:39], s55 offen lds\n\t"
      "s_add_u32 m0, s51, 20480\n\t"
      "s_add_u32 s55, s55, 0x1000\n\t"
      "buffer_load_dwordx4 v3, s[36:39], s55 offen lds\n\t"
      "s_add_u32 m0, s51, 21504\n\t"
      "s_add_u32 s55, s55, 0x1000\n\t"
      "buffer_load_dwordx4 v4, s[36:39], s55 offen lds\n\t"
      "s_bfe_u32 s61, s88, 131088\n\t"
      "s_bfe_u32 s49, s88, 196626\n\t"
      "s_bfe_u32 s52, s88, 65557\n\t"
      "ds_read_b128 v[96:99], v5 offset:8192\n\t"
      "ds_read_b128 v[100:103], v6 offset:8192\n\t"
      "ds_read_b128 v[104:107], v5 offset:10240\n\t"
      "ds_read_b128 v[108:111], v6 offset:10240\n\t"
      "ds_read_b128 v[112:115], v22 offset:12288\n\t"
      "ds_read_b128 v[116:119], v23 offset:12288\n\t"
      "SW1_%=:\n\t"
      "s_memrealtime s[70:71]\n\t"
      "s_waitcnt lgkmcnt(0)\n\t"
      "s_sub_u32 s70, s70, s62\n\t"
      "s_cmp_lt_i32 s70, 0\n\t"
      "s_cbranch_scc1 SW1_%=\n\t"
      "buffer_store_dwordx4 v[96:99], v7, s[32:35], s50 offen sc0 sc1\n\t"
      "buffer_store_dwordx4 v[100:103], v7, s[32:35], s50 offen offset:1024 sc0 sc1\n\t"
      "buffer_store_dwordx4 v[104:107], v7, s[32:35], s50 offen offset:2048 sc0 sc1\n\t"
      "buffer_store_dwordx4 v[108:111], v7, s[32:35], s50 offen offset:3072 sc0 sc1\n\t"
      "s_cmp_eq_u32 s52, 0\n\t"
      "s_cbranch_scc1 SH1_%=\n\t"
      "buffer_store_dwordx4 v[112:115], v8, s[32:35], s64 offen sc0 sc1\n\t"
      "buffer_store_dwordx4 v[116:119], v8, s[32:35], s64 offen offset:1024 sc0 sc1\n\t"
      "SH1_%=:\n\t"
      "v_add_u32 v18, s50, v16\n\t"
      "s_bfe_u32 s61, s89, 131072\n\t"
      "s_bfe_u32 s49, s89, 196610\n\t"
      "s_bfe_u32 s52, s89, 65541\n\t"
      "s_bfe_u32 s58, s89, 393222\n\t"
      "s_lshl_b32 s56, s49, 2\n\t"
      "s_add_u32 s56, s56, s61\n\t"
      "s_mul_i32 s56, s56, 10240\n\t"
      "s_lshl_b32 s54, s52, 7\n\t"
      "s_add_u32 s54, s54, s56\n\t"
      "s_add_u32 s54, s54, 8960\n\t"
      "v_mov_b32 v16, s54\n\t"
      "s_lshl_b32 s55, s52, 12\n\t"
      "s_add_u32 s56, s56, s55\n\t"
      "s_add_u32 s32, s92, s56\n\t"
      "s_addc_u32 s33, s93, 0\n\t"
      "s_and_b32 s33, s33, 0xffff\n\t"
      "s_mov_b32 s34, 333824\n\t"
      "s_and_b32 s55, s58, 3\n\t"
      "s_mul_i32 s54, s52, 10\n\t"
      "s_add_u32 s54, s54, 32\n\t"
      "s_mul_i32 s54, s54, s55\n\t"
      "s_lshl_b32 s55, s52, 7\n\t"
      "s_add_u32 s54, s54, s55\n\t"
      "s_lshr_b32 s55, s58, 3\n\t"
      "s_mul_i32 s55, s55, 296\n\t"
      "s_add_u32 s54, s54, s55\n\t"
      "s_mul_i32 s62, s54, 232\n\t"
      "s_lshr_b32 s62, s62, 10\n\t"
      "s_add_u32 s62, s62, s65\n\t"
      "s_mov_b64 exec, 1\n\t"
      "s_waitcnt vmcnt(0)\n\t"
      "global_store_dword v18, v17, s[92:93] sc0 sc1\n\t"
      "s_mov_b64 exec, -1\n\t"
      "s_bfe_u32 s61, s89, 131088\n\t"
      "s_bfe_u32 s49, s89, 196626\n\t"
      "s_bfe_u32 s52, s89, 65557\n\t"
      "s_mul_i32 s54, s49, 37376\n\t"
      "s_add_u32 s54, s54, s95\n\t"
      "s_lshl_b32 s55, s61, 7\n\t"
      "s_add_u32 s54, s54, s55\n\t"
      "s_lshl_b32 s55, s52, 14\n\t"
      "s_add_u32 s54, s54, s55\n\t"
      "s_add_u32 s36, s20, s54\n\t"
      "s_addc_u32 s37, s21, 0\n\t"
      "s_and_b32 s37, s37, 0xffff\n\t"
      "s_mul_i32 s38, s52, 5120\n\t"
      "s_add_u32 s38, s38, 16000\n\t"
      "s_mov_b32 s55, 0\n\t"
      "s_add_u32 m0, s51, 24576\n\t"
      "s_nop 0\n\t"
      "buffer_load_dwordx4 v3, s[36:39], s55 offen lds\n\t"
      "s_add_u32 m0, s51, 25600\n\t"
      "s_add_u32 s55, s55, 0x1000\n\t"
      "buffer_load_dwordx4 v3, s[36:39], s55 offen lds\n\t"
      "s_add_u32 m0, s51, 26624\n\t"
      "s_add_u32 s55, s55, 0x1000\n\t"
      "buffer_load_dwordx4 v3, s[36:39], s55 offen lds\n\t"
      "s_add_u32 m0, s51, 27648\n\t"
      "s_add_u32 s55, s55, 0x1000\n\t"
      "buffer_load_dwordx4 v3, s[36:39], s55 offen lds\n\t"
      "s_add_u32 m0, s51, 28672\n\t"
      "s_add_u32 s55, s55, 0x1000\n\t"
      "buffer_load_dwordx4 v3, s[36:39], s55 offen lds\n\t"
      "s_add_u32 m0, s51, 29696\n\t"
      "s_add_u32 s55, s55, 0x1000\n\t"
      "buffer_load_dwordx4 v4, s[36:39], s55 offen lds\n\t"
      "s_bfe_u32 s61, s89, 131072\n\t"
      "s_bfe_u32 s49, s89, 196610\n\t"
      "s_bfe_u32 s52, s89, 65541\n\t"
      "ds_read_b128 v[72:75], v5 offset:16384\n\t"
      "ds_read_b128 v[76:79], v6 offset:16384\n\t"
      "ds_read_b128 v[80:83], v5 offset:18432\n\t"
      "ds_read_b128 v[84:87], v6 offset:18432\n\t"
      "ds_read_b128 v[88:91], v22 offset:20480\n\t"
      "ds_read_b128 v[92:95], v23 offset:20480\n\t"
      "SW2_%=:\n\t"
      "s_memrealtime s[70:71]\n\t"
      "s_waitcnt lgkmcnt(0)\n\t"
      "s_sub_u32 s70, s70, s62\n\t"
      "s_cmp_lt_i32 s70, 0\n\t"
      "s_cbranch_scc1 SW2_%=\n\t"
      "buffer_store_dwordx4 v[72:75], v7, s[32:35], s50 offen sc0 sc1\n\t"
      "buffer_store_dwordx4 v[76:79], v7, s[32:35], s50 offen offset:1024 sc0 sc1\n\t"
      "buffer_store_dwordx4 v[80:83], v7, s[32:35], s50 offen offset:2048 sc0 sc1\n\t"
      "buffer_store_dwordx4 v[84:87], v7, s[32:35], s50 offen offset:3072 sc0 sc1\n\t"
      "s_cmp_eq_u32 s52, 0\n\t"
      "s_cbranch_scc1 SH2_%=\n\t"
      "buffer_store_dwordx4 v[88:91], v8, s[32:35], s64 offen sc0 sc1\n\t"
      "buffer_store_dwordx4 v[92:95], v8, s[32:35], s64 offen offset:1024 sc0 sc1\n\t"
      "SH2_%=:\n\t"
      "v_add_u32 v18, s50, v16\n\t"
      "s_bfe_u32 s61, s89, 131088\n\t"
      "s_bfe_u32 s49, s89, 196626\n\t"
      "s_bfe_u32 s52, s89, 65557\n\t"
      "s_bfe_u32 s58, s89, 393238\n\t"
      "s_lshl_b32 s56, s49, 2\n\t"
      "s_add_u32 s56, s56, s61\n\t"
      "s_mul_i32 s56, s56, 10240\n\t"
      "s_lshl_b32 s54, s52, 7\n\t"
      "s_add_u32 s54, s54, s56\n\t"
      "s_add_u32 s54, s54, 8960\n\t"
      "v_mov_b32 v16, s54\n\t"
      "s_lshl_b32 s55, s52, 12\n\t"
      "s_add_u32 s56, s56, s55\n\t"
      "s_add_u32 s32, s92, s56\n\t"
      "s_addc_u32 s33, s93, 0\n\t"
      "s_and_b32 s33, s33, 0xffff\n\t"
      "s_mov_b32 s34, 333824\n\t"
      "s_and_b32 s55, s58, 3\n\t"
      "s_mul_i32 s54, s52, 10\n\t"
      "s_add_u32 s54, s54, 32\n\t"
      "s_mul_i32 s54, s54, s55\n\t"
      "s_lshl_b32 s55, s52, 7\n\t"
      "s_add_u32 s54, s54, s55\n\t"
      "s_lshr_b32 s55, s58, 3\n\t"
      "s_mul_i32 s55, s55, 296\n\t"
      "s_add_u32 s54, s54, s55\n\t"
      "s_mul_i32 s62, s54, 232\n\t"
      "s_lshr_b32 s62, s62, 10\n\t"
      "s_add_u32 s62, s62, s65\n\t"
      "s_mov_b64 exec, 1\n\t"
      "s_waitcnt vmcnt(0)\n\t"
      "global_store_dword v18, v17, s[92:93] sc0 sc1\n\t"
      "s_mov_b64 exec, -1\n\t"
      "s_cmp_ge_u32 s48, 2\n\t"
      "s_cbranch_scc1 SEND_%=\n\t"
      "ds_read_b128 v[96:99], v5 offset:24576\n\t"
      "ds_read_b128 v[100:103], v6 offset:24576\n\t"
      "ds_read_b128 v[104:107], v5 offset:26624\n\t"
      "ds_read_b128 v[108:111], v6 offset:26624\n\t"
      "ds_read_b128 v[112:115], v22 offset:28672\n\t"
      "ds_read_b128 v[116:119], v23 offset:28672\n\t"
      "SW3_%=:\n\t"
      "s_memrealtime s[70:71]\n\t"
      "s_waitcnt lgkmcnt(0)\n\t"
      "s_sub_u32 s70, s70, s62\n\t"
      "s_cmp_lt_i32 s70, 0\n\t"
      "s_cbranch_scc1 SW3_%=\n\t"
      "buffer_store_dwordx4 v[96:99], v7, s[32:35], s50 offen sc0 sc1\n\t"
      "buffer_store_dwordx4 v[100:103], v7, s[32:35], s50 offen offset:1024 sc0 sc1\n\t"
      "buffer_store_dwordx4 v[104:107], v7, s[32:35], s50 offen offset:2048 sc0 sc1\n\t"
      "buffer_store_dwordx4 v[108:111], v7, s[32:35], s50 offen offset:3072 sc0 sc1\n\t"
      "s_cmp_eq_u32 s52, 0\n\t"
      "s_cbranch_scc1 SH3_%=\n\t"
      "buffer_store_dwordx4 v[112:115], v8, s[32:35], s64 offen sc0 sc1\n\t"
      "buffer_store_dwordx4 v[116:119], v8, s[32:35], s64 offen offset:1024 sc0 sc1\n\t"
      "SH3_%=:\n\t"
      "v_add_u32 v18, s50, v16\n\t"
      "s_mov_b64 exec, 1\n\t"
      "s_waitcnt vmcnt(0)\n\t"
      "global_store_dword v18, v17, s[92:93] sc0 sc1\n\t"
      "s_mov_b64 exec, -1\n\t"
      "SEND_%=:\n\t"
      "s_and_b32 s54, %1, 3\n\t"
      "s_lshl_b32 s54, s54, 3\n\t"
      "s_and_b32 s55, %1, 4\n\t"
      "s_or_b32 s54, s54, s55\n\t"
      "s_sub_u32 s54, s54, s23\n\t"
      "v_and_b32 v20, 3, v2\n\t"
      "v_add_u32 v20, s54, v20\n\t"
      "v_and_b32 v20, 31, v20\n\t"
      "v_lshlrev_b32 v20, 7, v20\n\t"
      "v_lshrrev_b32 v21, 2, v2\n\t"
      "v_lshl_or_b32 v20, v21, 12, v20\n\t"
      "s_add_u32 s40, s8, 0\n\t"
      "s_addc_u32 s41, s9, 0\n\t"
      "s_and_b32 s41, s41, 0xffff\n\t"
      "s_mov_b32 s42, 16777216\n\t"
      "s_mov_b32 s43, 0x20000\n\t"
      "s_barrier\n\t"
      "s_lshr_b32 s54, %1, 3\n\t"
      "s_lshl_b32 s55, s48, 2\n\t"
      "s_add_u32 s54, s54, s55\n\t"
      "s_lshr_b32 s63, s54, 2\n\t"
      "s_add_u32 s63, s63, s22\n\t"
      "s_add_u32 s63, s63, 1\n\t"
      "s_and_b32 s63, s63, 7\n\t"
      "s_lshl_b32 s63, s63, 4\n\t"
      "s_and_b32 s54, s54, 3\n\t"
      "s_add_u32 s63, s63, s54\n\t"
      "s_add_u32 s63, s63, 12\n\t"
      "s_lshl_b32 s63, s63, 17\n\t"
      "buffer_load_dword v21, v20, s[40:43], s63 offen\n\t"
      "s_add_u32 s63, s63, 0x10000\n\t"
      "buffer_load_dword v21, v20, s[40:43], s63 offen\n\t"
      "s_add_u32 s63, s63, 0x10000\n\t"
      "s_cmp_eq_u32 s48, 3\n\t"
      "s_cbranch_scc1 SPE_%=\n\t"
      "s_lshr_b32 s54, %1, 3\n\t"
      "s_lshl_b32 s55, s48, 2\n\t"
      "s_add_u32 s54, s54, s55\n\t"
      "s_add_u32 s54, s54, 16\n\t"
      "s_lshr_b32 s63, s54, 2\n\t"
      "s_add_u32 s63, s63, s22\n\t"
      "s_add_u32 s63, s63, 1\n\t"
      "s_and_b32 s63, s63, 7\n\t"
      "s_lshl_b32 s63, s63, 4\n\t"
      "s_and_b32 s54, s54, 3\n\t"
      "s_add_u32 s63, s63, s54\n\t"
      "s_add_u32 s63, s63, 12\n\t"
      "s_lshl_b32 s63, s63, 17\n\t"
      "buffer_load_dword v21, v20, s[40:43], s63 offen\n\t"
      "s_add_u32 s63, s63, 0x10000\n\t"
      "buffer_load_dword v21, v20, s[40:43], s63 offen\n\t"
      "s_add_u32 s63, s63, 0x10000\n\t"
      "SPE_%=:\n\t"
      "s_mov_b64 exec, 1\n\t"
      "buffer_store_dword v17, v24, s[44:47], s53 offen sc1\n\t"
      "s_endpgm\n\t"
      "PREF_%=:\n\t"
      "s_and_b32 s50, %1, 3\n\t"
      "s_lshl_b32 s50, s50, 3\n\t"
      "s_or_b32 s50, s50, 3\n\t"
      "s_and_b32 s54, %1, 4\n\t"
      "s_or_b32 s50, s50, s54\n\t"
      "v_lshlrev_b32 v12, 12, v2\n\t"
      "s_lshr_b32 s56, s48, 1\n\t"
      "s_and_b32 s57, s48, 1\n\t"
      "s_lshl_b32 s57, s57, 18\n\t"
      "s_waitcnt lgkmcnt(0)\n\t"
      "s_sub_u32 s50, s50, s23\n\t"
      "s_and_b32 s50, s50, 31\n\t"
      "s_lshl_b32 s54, s50, 7\n\t"
      "s_add_u32 s28, s8, s54\n\t"
      "s_addc_u32 s29, s9, 0\n\t"
      "s_and_b32 s29, s29, 0xffff\n\t"
      "s_sub_u32 s30, 0x1000000, s54\n\t"
      "s_add_u32 s32, s10, 0\n\t"
      "s_addc_u32 s33, s11, 0\n\t"
      "s_and_b32 s33, s33, 0xffff\n\t"
      "s_mov_b32 s34, 262144\n\t"
      "s_bfe_u32 s54, s94, 0x20004\n\t"
      "s_lshl_b32 s54, s54, 2\n\t"
      "s_add_u32 s54, s54, s48\n\t"
      "s_lshl_b32 s54, s54, 14\n\t"
      "v_lshlrev_b32 v14, 7, v2\n\t"
      "buffer_load_dword v13, v14, s[32:35], s54 offen\n\t"
      "s_add_u32 s54, s54, 0x2000\n\t"
      "buffer_load_dword v13, v14, s[32:35], s54 offen\n\t"
      "s_cmp_eq_u32 s56, 0\n\t"
      "s_cselect_b32 s54, 1, 2\n\t"
      "s_cselect_b32 s55, 1, 1\n\t"
      "s_add_u32 s54, s54, s22\n\t"
      "s_and_b32 s54, s54, 7\n\t"
      "s_lshl_b32 s54, s54, 2\n\t"
      "s_add_u32 s54, s54, s55\n\t"
      "s_lshl_b32 s54, s54, 19\n\t"
      "s_add_u32 s54, s54, s57\n\t"
      "buffer_load_dword v13, v12, s[28:31], s54 offen\n\t"
      "s_cmp_eq_u32 s56, 0\n\t"
      "s_cselect_b32 s54, 3, 4\n\t"
      "s_cselect_b32 s55, 1, 1\n\t"
      "s_add_u32 s54, s54, s22\n\t"
      "s_and_b32 s54, s54, 7\n\t"
      "s_lshl_b32 s54, s54, 2\n\t"
      "s_add_u32 s54, s54, s55\n\t"
      "s_lshl_b32 s54, s54, 19\n\t"
      "s_add_u32 s54, s54, s57\n\t"
      "buffer_load_dword v13, v12, s[28:31], s54 offen\n\t"
      "s_cmp_eq_u32 s56, 0\n\t"
      "s_cselect_b32 s54, 5, 6\n\t"
      "s_cselect_b32 s55, 1, 1\n\t"
      "s_add_u32 s54, s54, s22\n\t"
      "s_and_b32 s54, s54, 7\n\t"
      "s_lshl_b32 s54, s54, 2\n\t"
      "s_add_u32 s54, s54, s55\n\t"
      "s_lshl_b32 s54, s54, 19\n\t"
      "s_add_u32 s54, s54, s57\n\t"
      "buffer_load_dword v13, v12, s[28:31], s54 offen\n\t"
      "s_cmp_eq_u32 s56, 0\n\t"
      "s_cselect_b32 s54, 7, 7\n\t"
      "s_cselect_b32 s55, 1, 1\n\t"
      "s_add_u32 s54, s54, s22\n\t"
      "s_and_b32 s54, s54, 7\n\t"
      "s_lshl_b32 s54, s54, 2\n\t"
      "s_add_u32 s54, s54, s55\n\t"
      "s_lshl_b32 s54, s54, 19\n\t"
      "s_add_u32 s54, s54, s57\n\t"
      "buffer_load_dword v13, v12, s[28:31], s54 offen\n\t"
      "s_cmp_eq_u32 s56, 0\n\t"
      "s_cselect_b32 s54, 1, 2\n\t"
      "s_cselect_b32 s55, 2, 2\n\t"
      "s_add_u32 s54, s54, s22\n\t"
      "s_and_b32 s54, s54, 7\n\t"
      "s_lshl_b32 s54, s54, 2\n\t"
      "s_add_u32 s54, s54, s55\n\t"
      "s_lshl_b32 s54, s54, 19\n\t"
      "s_add_u32 s54, s54, s57\n\t"
      "buffer_load_dword v13, v12, s[28:31], s54 offen\n\t"
      "s_cmp_eq_u32 s56, 0\n\t"
      "s_cselect_b32 s54, 3, 4\n\t"
      "s_cselect_b32 s55, 2, 2\n\t"
      "s_add_u32 s54, s54, s22\n\t"
      "s_and_b32 s54, s54, 7\n\t"
      "s_lshl_b32 s54, s54, 2\n\t"
      "s_add_u32 s54, s54, s55\n\t"
      "s_lshl_b32 s54, s54, 19\n\t"
      "s_add_u32 s54, s54, s57\n\t"
      "buffer_load_dword v13, v12, s[28:31], s54 offen\n\t"
      "s_cmp_eq_u32 s56, 0\n\t"
      "s_cselect_b32 s54, 5, 6\n\t"
      "s_cselect_b32 s55, 2, 2\n\t"
      "s_add_u32 s54, s54, s22\n\t"
      "s_and_b32 s54, s54, 7\n\t"
      "s_lshl_b32 s54, s54, 2\n\t"
      "s_add_u32 s54, s54, s55\n\t"
      "s_lshl_b32 s54, s54, 19\n\t"
      "s_add_u32 s54, s54, s57\n\t"
      "buffer_load_dword v13, v12, s[28:31], s54 offen\n\t"
      "s_cmp_eq_u32 s56, 0\n\t"
      "s_cselect_b32 s54, 7, 7\n\t"
      "s_cselect_b32 s55, 2, 2\n\t"
      "s_add_u32 s54, s54, s22\n\t"
      "s_and_b32 s54, s54, 7\n\t"
      "s_lshl_b32 s54, s54, 2\n\t"
      "s_add_u32 s54, s54, s55\n\t"
      "s_lshl_b32 s54, s54, 19\n\t"
      "s_add_u32 s54, s54, s57\n\t"
      "buffer_load_dword v13, v12, s[28:31], s54 offen\n\t"
      "s_and_b32 s54, %1, 3\n\t"
      "s_lshl_b32 s54, s54, 3\n\t"
      "s_and_b32 s55, %1, 4\n\t"
      "s_or_b32 s54, s54, s55\n\t"
      "s_sub_u32 s54, s54, s23\n\t"
      "v_and_b32 v20, 3, v2\n\t"
      "v_add_u32 v20, s54, v20\n\t"
      "v_and_b32 v20, 31, v20\n\t"
      "v_lshlrev_b32 v20, 7, v20\n\t"
      "v_lshrrev_b32 v21, 2, v2\n\t"
      "v_lshl_or_b32 v20, v21, 12, v20\n\t"
      "s_add_u32 s40, s8, 0\n\t"
      "s_addc_u32 s41, s9, 0\n\t"
      "s_and_b32 s41, s41, 0xffff\n\t"
      "s_mov_b32 s42, 16777216\n\t"
      "s_mov_b32 s43, 0x20000\n\t"
      "s_lshl_b32 s54, s48, 2\n\t"
      "s_add_u32 s54, s54, 3\n\t"
      "s_lshr_b32 s63, s54, 2\n\t"
      "s_add_u32 s63, s63, s22\n\t"
      "s_add_u32 s63, s63, 1\n\t"
      "s_and_b32 s63, s63, 7\n\t"
      "s_lshl_b32 s63, s63, 4\n\t"
      "s_and_b32 s54, s54, 3\n\t"
      "s_add_u32 s63, s63, s54\n\t"
      "s_add_u32 s63, s63, 12\n\t"
      "s_lshl_b32 s63, s63, 17\n\t"
      "buffer_load_dword v21, v20, s[40:43], s63 offen\n\t"
      "s_add_u32 s63, s63, 0x10000\n\t"
      "buffer_load_dword v21, v20, s[40:43], s63 offen\n\t"
      "s_add_u32 s63, s63, 0x10000\n\t"
      "s_cmp_eq_u32 s48, 3\n\t"
      "s_cbranch_scc1 PFE_%=\n\t"
      "s_lshl_b32 s54, s48, 2\n\t"
      "s_add_u32 s54, s54, 19\n\t"
      "s_lshr_b32 s63, s54, 2\n\t"
      "s_add_u32 s63, s63, s22\n\t"
      "s_add_u32 s63, s63, 1\n\t"
      "s_and_b32 s63, s63, 7\n\t"
      "s_lshl_b32 s63, s63, 4\n\t"
      "s_and_b32 s54, s54, 3\n\t"
      "s_add_u32 s63, s63, s54\n\t"
      "s_add_u32 s63, s63, 12\n\t"
      "s_lshl_b32 s63, s63, 17\n\t"
      "buffer_load_dword v21, v20, s[40:43], s63 offen\n\t"
      "s_add_u32 s63, s63, 0x10000\n\t"
      "buffer_load_dword v21, v20, s[40:43], s63 offen\n\t"
      "s_add_u32 s63, s63, 0x10000\n\t"
      "PFE_%=:\n\t"
      "s_waitcnt vmcnt(0)\n\t"
      "s_endpgm\n\t"
      "CMP_%=:\n\t"
      "s_lshr_b32 s49, %1, 5\n\t"
      "s_sub_u32 s49, s49, 1\n\t"
      "s_lshl_b32 s54, s49, 2\n\t"
      "s_add_u32 s54, s54, s48\n\t"
      "s_lshl_b32 s55, s54, 5\n\t"
      "s_and_b32 s56, %1, 24\n\t"
      "s_add_u32 s55, s55, s56\n\t"
      "s_load_dwordx2 s[24:25], %0, s55 offset:0x40\n\t"
      "s_mul_i32 s51, s48, 40960\n\t"
      "s_and_b32 s50, %1, 3\n\t"
      "s_lshl_b32 s50, s50, 3\n\t"
      "s_and_b32 s54, 4, %1\n\t"
      "s_or_b32 s50, s50, s54\n\t"
      "s_bfe_u32 s54, %1, 0x20003\n\t"
      "s_or_b32 s50, s50, s54\n\t"
      "s_mul_i32 s56, s49, 37376\n\t"
      "s_lshl_b32 s54, s48, 7\n\t"
      "s_add_u32 s56, s56, s54\n\t"
      "s_mov_b32 s38, 37504\n\t"
      "s_mov_b32 s31, 0x20000\n\t"
      "s_mov_b32 s35, 0x20000\n\t"
      "s_mov_b32 s39, 0x20000\n\t"
      "s_mov_b32 s43, 0x20000\n\t"
      "s_mov_b32 s47, 0x20000\n\t"
      "v_lshrrev_b32 v11, 3, v2\n\t"
      "v_and_b32 v26, 6, v11\n\t"
      "v_xor_b32 v26, v26, v2\n\t"
      "v_and_b32 v26, 7, v26\n\t"
      "v_lshlrev_b32 v26, 4, v26\n\t"
      "v_lshl_or_b32 v3, v11, 9, v26\n\t"
      "v_cmp_gt_u32 vcc, 2, v11\n\t"
      "v_bfrev_b32 v4, 1\n\t"
      "s_nop 0\n\t"
      "v_cndmask_b32 v4, v4, v3, vcc\n\t"
      "v_lshl_or_b32 v12, v11, 12, v26\n\t"
      "v_add_u32 v13, 31744, v12\n\t"
      "v_xor_b32 v14, 16, v12\n\t"
      "v_add_u32 v15, 95232, v14\n\t"
      "v_add_u32 v14, 63488, v14\n\t"
      "s_and_b32 s52, s49, 1\n\t"
      "s_lshl_b32 s52, s52, 18\n\t"
      "s_and_b32 s26, s48, 1\n\t"
      "s_lshl_b32 s26, s26, 18\n\t"
      "s_lshl_b32 s53, %1, 8\n\t"
      "s_add_u32 s53, s53, 256\n\t"
      "v_mov_b32 v25, 0\n\t"
      "s_waitcnt lgkmcnt(0)\n\t"
      "s_add_u32 s36, s10, s56\n\t"
      "s_addc_u32 s37, s11, 0\n\t"
      "s_and_b32 s37, s37, 0xffff\n\t"
      "s_sub_u32 s50, s50, s23\n\t"
      "s_and_b32 s50, s50, 31\n\t"
      "s_lshl_b32 s54, s50, 7\n\t"
      "s_add_u32 s28, s8, s54\n\t"
      "s_addc_u32 s29, s9, 0\n\t"
      "s_and_b32 s29, s29, 0xffff\n\t"
      "s_sub_u32 s30, 0x1000000, s54\n\t"
      "s_bfe_u32 s54, s24, 524288\n\t"
      "s_add_u32 m0, s51, 0\n\t"
      "s_lshl_b32 s54, s54, 19\n\t"
      "s_xor_b32 s54, s54, s52\n\t"
      "buffer_load_dwordx4 v12, s[28:31], s54 offen lds\n\t"
      "buffer_load_dwordx4 v13, s[28:31], s54 offen offset:1024 lds\n\t"
      "buffer_load_dwordx4 v14, s[28:31], s54 offen offset:2048 lds\n\t"
      "buffer_load_dwordx4 v15, s[28:31], s54 offen offset:3072 lds\n\t"
      "s_add_u32 m0, s51, 4096\n\t"
      "s_add_u32 s54, s54, 0x20000\n\t"
      "buffer_load_dwordx4 v12, s[28:31], s54 offen lds\n\t"
      "buffer_load_dwordx4 v13, s[28:31], s54 offen offset:1024 lds\n\t"
      "buffer_load_dwordx4 v14, s[28:31], s54 offen offset:2048 lds\n\t"
      "buffer_load_dwordx4 v15, s[28:31], s54 offen offset:3072 lds\n\t"
      "s_add_u32 s44, s12, 8388608\n\t"
      "s_addc_u32 s45, s13, 0\n\t"
      "s_and_b32 s45, s45, 0xffff\n\t"
      "s_mov_b32 s46, 65792\n\t"
      "s_load_dword s64, s[16:17], 0x0\n\t"
      "s_load_dword s65, s[18:19], 0x0\n\t"
      "s_bfe_u32 s54, %1, 0x20003\n\t"
      "s_lshl_b32 s54, s54, 2\n\t"
      "s_add_u32 s54, s54, s48\n\t"
      "s_lshl_b32 s54, s54, 15\n\t"
      "s_add_u32 s56, s49, s22\n\t"
      "s_add_u32 s56, s56, 1\n\t"
      "s_and_b32 s56, s56, 7\n\t"
      "s_lshl_b32 s56, s56, 21\n\t"
      "s_add_u32 s54, s54, s56\n\t"
      "s_add_u32 s56, s50, s23\n\t"
      "s_or_b32 s56, s56, 3\n\t"
      "s_sub_u32 s56, s56, s23\n\t"
      "s_and_b32 s56, s56, 31\n\t"
      "s_lshl_b32 s56, s56, 7\n\t"
      "s_add_u32 s54, s54, s56\n\t"
      "s_add_u32 s10, s8, s54\n\t"
      "s_addc_u32 s11, s9, 0\n\t"
      "buffer_load_dword v25, v25, s[44:47], s53 offen sc1\n\t"
      "s_bitcmp1_b32 %1, 2\n\t"
      "s_cbranch_scc1 NOPF1_%=\n\t"
      "s_mov_b32 s27, 0\n\t"
      "s_load_dword s20, s[10:11], s27 offset:0x0\n\t"
      "s_load_dword s20, s[10:11], s27 offset:0x40\n\t"
      "s_load_dword s20, s[10:11], s27 offset:0x1000\n\t"
      "s_load_dword s20, s[10:11], s27 offset:0x1040\n\t"
      "s_load_dword s20, s[10:11], s27 offset:0x2000\n\t"
      "s_load_dword s20, s[10:11], s27 offset:0x2040\n\t"
      "s_load_dword s20, s[10:11], s27 offset:0x3000\n\t"
      "s_load_dword s20, s[10:11], s27 offset:0x3040\n\t"
      "s_load_dword s20, s[10:11], s27 offset:0x4000\n\t"
      "s_load_dword s20, s[10:11], s27 offset:0x4040\n\t"
      "s_load_dword s20, s[10:11], s27 offset:0x5000\n\t"
      "s_load_dword s20, s[10:11], s27 offset:0x5040\n\t"
      "s_load_dword s20, s[10:11], s27 offset:0x6000\n\t"
      "s_load_dword s20, s[10:11], s27 offset:0x6040\n\t"
      "s_load_dword s20, s[10:11], s27 offset:0x7000\n\t"
      "s_load_dword s20, s[10:11], s27 offset:0x7040\n\t"
      "NOPF1_%=:\n\t"
      "s_mov_b32 s55, 0\n\t"
      "s_add_u32 m0, s51, 24576\n\t"
      "s_nop 0\n\t"
      "buffer_load_dwordx4 v3, s[36:39], s55 offen lds\n\t"
      "s_add_u32 m0, s51, 25600\n\t"
      "s_add_u32 s55, s55, 0x1000\n\t"
      "buffer_load_dwordx4 v3, s[36:39], s55 offen lds\n\t"
      "s_add_u32 m0, s51, 26624\n\t"
      "s_add_u32 s55, s55, 0x1000\n\t"
      "buffer_load_dwordx4 v3, s[36:39], s55 offen lds\n\t"
      "s_add_u32 m0, s51, 27648\n\t"
      "s_add_u32 s55, s55, 0x1000\n\t"
      "buffer_load_dwordx4 v3, s[36:39], s55 offen lds\n\t"
      "s_add_u32 m0, s51, 28672\n\t"
      "s_add_u32 s55, s55, 0x1000\n\t"
      "buffer_load_dwordx4 v3, s[36:39], s55 offen lds\n\t"
      "s_add_u32 m0, s51, 29696\n\t"
      "s_add_u32 s55, s55, 0x1000\n\t"
      "buffer_load_dwordx4 v3, s[36:39], s55 offen lds\n\t"
      "s_add_u32 m0, s51, 30720\n\t"
      "s_add_u32 s55, s55, 0x1000\n\t"
      "buffer_load_dwordx4 v3, s[36:39], s55 offen lds\n\t"
      "s_add_u32 m0, s51, 31744\n\t"
      "s_add_u32 s55, s55, 0x1000\n\t"
      "buffer_load_dwordx4 v3, s[36:39], s55 offen lds\n\t"
      "s_add_u32 m0, s51, 32768\n\t"
      "s_add_u32 s55, s55, 0x1000\n\t"
      "buffer_load_dwordx4 v3, s[36:39], s55 offen lds\n\t"
      "s_add_u32 m0, s51, 33792\n\t"
      "s_add_u32 s55, s55, 0x1000\n\t"
      "buffer_load_dwordx4 v4, s[36:39], s55 offen lds\n\t"
      "v_lshrrev_b32 v11, 4, v2\n\t"
      "v_and_b32 v26, 15, v2\n\t"
      "v_lshrrev_b32 v27, 1, v26\n\t"
      "v_lshlrev_b32 v16, 11, v11\n\t"
      "v_lshl_or_b32 v16, v27, 7, v16\n\t"
      "v_and_b32 v27, 6, v27\n\t"
      "v_and_or_b32 v27, v11, 1, v27\n\t"
      "v_lshl_or_b32 v16, v27, 4, v16\n\t"
      "v_and_b32 v27, 1, v2\n\t"
      "v_lshl_or_b32 v16, v27, 3, v16\n\t"
      "v_add_u32 v16, s51, v16\n\t"
      "v_xor_b32 v17, 16, v16\n\t"
      "v_xor_b32 v18, 32, v16\n\t"
      "v_xor_b32 v19, 48, v16\n\t"
      "v_xor_b32 v20, 64, v16\n\t"
      "v_xor_b32 v21, 80, v16\n\t"
      "v_xor_b32 v22, 96, v16\n\t"
      "v_xor_b32 v23, 112, v16\n\t"
      "v_and_b32 v27, 6, v26\n\t"
      "v_xor_b32 v27, v27, v11\n\t"
      "v_lshlrev_b32 v27, 4, v27\n\t"
      "v_lshl_or_b32 v5, v26, 7, v27\n\t"
      "s_lshr_b32 s54, s52, 12\n\t"
      "v_xor_b32 v5, s54, v5\n\t"
      "v_add_u32 v5, s51, v5\n\t"
      "v_xor_b32 v6, 64, v5\n\t"
      "s_mov_b32 s57, 0x76543210\n\t"
      "v_lshrrev_b32 v27, 1, v2\n\t"
      "v_and_b32 v27, 28, v27\n\t"
      "v_lshrrev_b32 v27, v27, s57\n\t"
      "v_and_b32 v27, 15, v27\n\t"
      "v_and_b32 v7, 7, v2\n\t"
      "v_lshlrev_b32 v7, 4, v7\n\t"
      "v_lshl_or_b32 v7, v27, 7, v7\n\t"
      "v_and_b32 v8, 3, v26\n\t"
      "v_lshl_or_b32 v8, v11, 2, v8\n\t"
      "v_lshrrev_b32 v27, 2, v26\n\t"
      "v_lshl_or_b32 v8, v27, 4, v8\n\t"
      "v_and_b32 v27, 1, v26\n\t"
      "v_lshl_or_b32 v27, v11, 1, v27\n\t"
      "v_cmp_gt_u32 vcc, 8, v26\n\t"
      "v_or_b32 v27, 32, v27\n\t"
      "s_nop 0\n\t"
      "v_cndmask_b32 v28, v27, v8, vcc\n\t"
      "v_lshrrev_b32 v27, 1, v28\n\t"
      "v_and_b32 v27, 28, v27\n\t"
      "v_lshrrev_b32 v27, v27, s57\n\t"
      "v_and_b32 v27, 15, v27\n\t"
      "v_and_b32 v8, 7, v28\n\t"
      "v_lshlrev_b32 v8, 4, v8\n\t"
      "v_lshl_or_b32 v8, v27, 7, v8\n\t"
      "v_cmp_gt_u32 vcc, 10, v26\n\t"
      "v_bfrev_b32 v27, 1\n\t"
      "s_nop 0\n\t"
      "v_cndmask_b32 v8, v27, v8, vcc\n\t"
      "s_lshr_b32 s54, s26, 8\n\t"
      "v_or_b32 v7, s54, v7\n\t"
      "v_or_b32 v8, s54, v8\n\t"
      "v_xor_b32 v3, 0x400, v7\n\t"
      "v_xor_b32 v4, 0x400, v8\n\t"
      "v_cmp_gt_u32 vcc, 2, v2\n\t"
      "s_movk_i32 s54, 8960\n\t"
      "v_lshl_add_u32 v9, v2, 7, s54\n\t"
      "s_nop 0\n\t"
      "v_cndmask_b32 v9, v27, v9, vcc\n\t"
      "s_mul_i32 s54, s49, 40960\n\t"
      "s_add_u32 s32, s12, s54\n\t"
      "s_addc_u32 s33, s13, 0\n\t"
      "s_and_b32 s33, s33, 0xffff\n\t"
      "s_sub_u32 s34, 8388608, s54\n\t"
      "s_mul_i32 s54, s49, 1196032\n\t"
      "s_lshl_b32 s55, s50, 9\n\t"
      "s_add_u32 s54, s54, s55\n\t"
      "s_add_u32 s40, s14, s54\n\t"
      "s_addc_u32 s41, s15, 0\n\t"
      "s_and_b32 s41, s41, 0xffff\n\t"
      "s_mov_b32 s42, 1196544\n\t"
      "s_mov_b32 s58, 0\n\t"
      "s_mov_b32 s61, 0\n\t"
      "s_barrier\n\t"
      "s_bfe_u32 s54, s24, 524296\n\t"
      "s_add_u32 m0, s51, 16384\n\t"
      "s_lshl_b32 s54, s54, 19\n\t"
      "s_xor_b32 s54, s54, s26\n\t"
      "buffer_load_dwordx4 v12, s[28:31], s54 offen lds\n\t"
      "buffer_load_dwordx4 v13, s[28:31], s54 offen offset:1024 lds\n\t"
      "buffer_load_dwordx4 v14, s[28:31], s54 offen offset:2048 lds\n\t"
      "buffer_load_dwordx4 v15, s[28:31], s54 offen offset:3072 lds\n\t"
      "s_add_u32 m0, s51, 20480\n\t"
      "s_add_u32 s54, s54, 0x20000\n\t"
      "buffer_load_dwordx4 v12, s[28:31], s54 offen lds\n\t"
      "buffer_load_dwordx4 v13, s[28:31], s54 offen offset:1024 lds\n\t"
      "buffer_load_dwordx4 v14, s[28:31], s54 offen offset:2048 lds\n\t"
      "buffer_load_dwordx4 v15, s[28:31], s54 offen offset:3072 lds\n\t"
      "s_waitcnt vmcnt(8)\n\t"
      "ds_read_b128 a[0:3], v5 offset:24576\n\t"
      "ds_read_b128 a[4:7], v6 offset:24576\n\t"
      "ds_read_b128 a[8:11], v5 offset:26624\n\t"
      "ds_read_b128 a[12:15], v6 offset:26624\n\t"
      "ds_read_b128 a[16:19], v5 offset:28672\n\t"
      "ds_read_b128 a[20:23], v6 offset:28672\n\t"
      "ds_read_b128 a[24:27], v5 offset:30720\n\t"
      "ds_read_b128 a[28:31], v6 offset:30720\n\t"
      "ds_read_b128 a[32:35], v5 offset:32768\n\t"
      "ds_read_b128 a[36:39], v6 offset:32768\n\t"
      "s_waitcnt lgkmcnt(0)\n\t"
      "s_bfe_u32 s54, s24, 524288\n\t"
      "s_add_u32 m0, s51, 8192\n\t"
      "s_lshl_b32 s54, s54, 19\n\t"
      "s_xor_b32 s54, s54, s52\n\t"
      "s_xor_b32 s54, s54, 0x40000\n\t"
      "buffer_load_dwordx4 v12, s[28:31], s54 offen lds\n\t"
      "buffer_load_dwordx4 v13, s[28:31], s54 offen offset:1024 lds\n\t"
      "buffer_load_dwordx4 v14, s[28:31], s54 offen offset:2048 lds\n\t"
      "buffer_load_dwordx4 v15, s[28:31], s54 offen offset:3072 lds\n\t"
      "s_add_u32 m0, s51, 12288\n\t"
      "s_add_u32 s54, s54, 0x20000\n\t"
      "buffer_load_dwordx4 v12, s[28:31], s54 offen lds\n\t"
      "buffer_load_dwordx4 v13, s[28:31], s54 offen offset:1024 lds\n\t"
      "buffer_load_dwordx4 v14, s[28:31], s54 offen offset:2048 lds\n\t"
      "buffer_load_dwordx4 v15, s[28:31], s54 offen offset:3072 lds\n\t"
      "s_bfe_u32 s54, s24, 524296\n\t"
      "s_add_u32 m0, s51, 24576\n\t"
      "s_lshl_b32 s54, s54, 19\n\t"
      "s_xor_b32 s54, s54, s26\n\t"
      "s_xor_b32 s54, s54, 0x40000\n\t"
      "buffer_load_dwordx4 v12, s[28:31], s54 offen lds\n\t"
      "buffer_load_dwordx4 v13, s[28:31], s54 offen offset:1024 lds\n\t"
      "buffer_load_dwordx4 v14, s[28:31], s54 offen offset:2048 lds\n\t"
      "buffer_load_dwordx4 v15, s[28:31], s54 offen offset:3072 lds\n\t"
      "s_add_u32 m0, s51, 28672\n\t"
      "s_add_u32 s54, s54, 0x20000\n\t"
      "buffer_load_dwordx4 v12, s[28:31], s54 offen lds\n\t"
      "buffer_load_dwordx4 v13, s[28:31], s54 offen offset:1024 lds\n\t"
      "buffer_load_dwordx4 v14, s[28:31], s54 offen offset:2048 lds\n\t"
      "buffer_load_dwordx4 v15, s[28:31], s54 offen offset:3072 lds\n\t"
      "ds_read_b64_tr_b8 v[192:193], v16 offset:0\n\t"
      "ds_read_b64_tr_b8 v[194:195], v16 offset:1024\n\t"
      "ds_read_b64_tr_b8 v[200:201], v17 offset:0\n\t"
      "ds_read_b64_tr_b8 v[202:203], v17 offset:1024\n\t"
      "ds_read_b64_tr_b8 v[208:209], v18 offset:0\n\t"
      "ds_read_b64_tr_b8 v[210:211], v18 offset:1024\n\t"
      "ds_read_b64_tr_b8 v[216:217], v19 offset:0\n\t"
      "ds_read_b64_tr_b8 v[218:219], v19 offset:1024\n\t"
      "ds_read_b64_tr_b8 v[224:225], v20 offset:0\n\t"
      "ds_read_b64_tr_b8 v[226:227], v20 offset:1024\n\t"
      "ds_read_b64_tr_b8 v[232:233], v21 offset:0\n\t"
      "ds_read_b64_tr_b8 v[234:235], v21 offset:1024\n\t"
      "ds_read_b64_tr_b8 v[240:241], v22 offset:0\n\t"
      "ds_read_b64_tr_b8 v[242:243], v22 offset:1024\n\t"
      "ds_read_b64_tr_b8 v[248:249], v23 offset:0\n\t"
      "ds_read_b64_tr_b8 v[250:251], v23 offset:1024\n\t"
      "s_bfe_u32 s54, s24, 524304\n\t"
      "s_add_u32 m0, s51, 32768\n\t"
      "s_lshl_b32 s54, s54, 19\n\t"
      "s_xor_b32 s54, s54, s26\n\t"
      "buffer_load_dwordx4 v12, s[28:31], s54 offen lds\n\t"
      "buffer_load_dwordx4 v13, s[28:31], s54 offen offset:1024 lds\n\t"
      "buffer_load_dwordx4 v14, s[28:31], s54 offen offset:2048 lds\n\t"
      "buffer_load_dwordx4 v15, s[28:31], s54 offen offset:3072 lds\n\t"
      "s_add_u32 m0, s51, 36864\n\t"
      "s_add_u32 s54, s54, 0x20000\n\t"
      "buffer_load_dwordx4 v12, s[28:31], s54 offen lds\n\t"
      "buffer_load_dwordx4 v13, s[28:31], s54 offen offset:1024 lds\n\t"
      "buffer_load_dwordx4 v14, s[28:31], s54 offen offset:2048 lds\n\t"
      "buffer_load_dwordx4 v15, s[28:31], s54 offen offset:3072 lds\n\t"
      "s_waitcnt vmcnt(16)\n\t"
      "ds_read_b64_tr_b8 v[196:197], v16 offset:8192\n\t"
      "ds_read_b64_tr_b8 v[198:199], v16 offset:9216\n\t"
      "ds_read_b64_tr_b8 v[204:205], v17 offset:8192\n\t"
      "ds_read_b64_tr_b8 v[206:207], v17 offset:9216\n\t"
      "ds_read_b64_tr_b8 v[212:213], v18 offset:8192\n\t"
      "ds_read_b64_tr_b8 v[214:215], v18 offset:9216\n\t"
      "ds_read_b64_tr_b8 v[220:221], v19 offset:8192\n\t"
      "ds_read_b64_tr_b8 v[222:223], v19 offset:9216\n\t"
      "ds_read_b64_tr_b8 v[228:229], v20 offset:8192\n\t"
      "ds_read_b64_tr_b8 v[230:231], v20 offset:9216\n\t"
      "ds_read_b64_tr_b8 v[236:237], v21 offset:8192\n\t"
      "ds_read_b64_tr_b8 v[238:239], v21 offset:9216\n\t"
      "ds_read_b64_tr_b8 v[244:245], v22 offset:8192\n\t"
      "ds_read_b64_tr_b8 v[246:247], v22 offset:9216\n\t"
      "ds_read_b64_tr_b8 v[252:253], v23 offset:8192\n\t"
      "ds_read_b64_tr_b8 v[254:255], v23 offset:9216\n\t"
      "s_barrier\n\t"
      "s_waitcnt lgkmcnt(15)\n\t"
      "s_waitcnt lgkmcnt(0)\n\t"
      "v_mfma_f32_16x16x128_f8f6f4 v[32:35], a[0:7], v[192:199], 0\n\t"
      "s_bfe_u32 s54, s24, 524304\n\t"
      "s_add_u32 m0, s51, 0\n\t"
      "s_lshl_b32 s54, s54, 19\n\t"
      "s_xor_b32 s54, s54, s26\n\t"
      "s_xor_b32 s54, s54, 0x40000\n\t"
      "buffer_load_dwordx4 v12, s[28:31], s54 offen lds\n\t"
      "v_mfma_f32_16x16x128_f8f6f4 v[36:39], a[0:7], v[200:207], 0\n\t"
      "buffer_load_dwordx4 v13, s[28:31], s54 offen offset:1024 lds\n\t"
      "v_mfma_f32_16x16x128_f8f6f4 v[40:43], a[0:7], v[208:215], 0\n\t"
      "v_mfma_f32_16x16x128_f8f6f4 v[44:47], a[0:7], v[216:223], 0\n\t"
      "buffer_load_dwordx4 v14, s[28:31], s54 offen offset:2048 lds\n\t"
      "v_mfma_f32_16x16x128_f8f6f4 v[48:51], a[0:7], v[224:231], 0\n\t"
      "buffer_load_dwordx4 v15, s[28:31], s54 offen offset:3072 lds\n\t"
      "v_mfma_f32_16x16x128_f8f6f4 v[52:55], a[0:7], v[232:239], 0\n\t"
      "v_mfma_f32_16x16x128_f8f6f4 v[56:59], a[0:7], v[240:247], 0\n\t"
      "s_add_u32 m0, s51, 4096\n\t"
      "s_add_u32 s54, s54, 0x20000\n\t"
      "buffer_load_dwordx4 v12, s[28:31], s54 offen lds\n\t"
      "v_mfma_f32_16x16x128_f8f6f4 v[60:63], a[0:7], v[248:255], 0\n\t"
      "v_mfma_f32_16x16x128_f8f6f4 v[64:67], a[8:15], v[192:199], 0\n\t"
      "buffer_load_dwordx4 v13, s[28:31], s54 offen offset:1024 lds\n\t"
      "v_mfma_f32_16x16x128_f8f6f4 v[68:71], a[8:15], v[200:207], 0\n\t"
      "buffer_load_dwordx4 v14, s[28:31], s54 offen offset:2048 lds\n\t"
      "v_mfma_f32_16x16x128_f8f6f4 v[72:75], a[8:15], v[208:215], 0\n\t"
      "v_mfma_f32_16x16x128_f8f6f4 v[76:79], a[8:15], v[216:223], 0\n\t"
      "buffer_load_dwordx4 v15, s[28:31], s54 offen offset:3072 lds\n\t"
      "v_mfma_f32_16x16x128_f8f6f4 v[80:83], a[8:15], v[224:231], 0\n\t"
      "v_mfma_f32_16x16x128_f8f6f4 v[84:87], a[8:15], v[232:239], 0\n\t"
      "s_bfe_u32 s54, s24, 524312\n\t"
      "s_add_u32 m0, s51, 8192\n\t"
      "s_lshl_b32 s54, s54, 19\n\t"
      "s_xor_b32 s54, s54, s26\n\t"
      "buffer_load_dwordx4 v12, s[28:31], s54 offen lds\n\t"
      "v_mfma_f32_16x16x128_f8f6f4 v[88:91], a[8:15], v[240:247], 0\n\t"
      "buffer_load_dwordx4 v13, s[28:31], s54 offen offset:1024 lds\n\t"
      "ds_read_b64_tr_b8 a[120:121], v16 offset:16384\n\t"
      "ds_read_b64_tr_b8 a[122:123], v16 offset:17408\n\t"
      "v_mfma_f32_16x16x128_f8f6f4 v[92:95], a[8:15], v[248:255], 0\n\t"
      "ds_read_b64_tr_b8 a[128:129], v17 offset:16384\n\t"
      "ds_read_b64_tr_b8 a[130:131], v17 offset:17408\n\t"
      "v_mfma_f32_16x16x128_f8f6f4 v[96:99], a[16:23], v[192:199], 0\n\t"
      "buffer_load_dwordx4 v14, s[28:31], s54 offen offset:2048 lds\n\t"
      "ds_read_b64_tr_b8 a[136:137], v18 offset:16384\n\t"
      "ds_read_b64_tr_b8 a[138:139], v18 offset:17408\n\t"
      "v_mfma_f32_16x16x128_f8f6f4 v[100:103], a[16:23], v[200:207], 0\n\t"
      "buffer_load_dwordx4 v15, s[28:31], s54 offen offset:3072 lds\n\t"
      "ds_read_b64_tr_b8 a[144:145], v19 offset:16384\n\t"
      "ds_read_b64_tr_b8 a[146:147], v19 offset:17408\n\t"
      "v_mfma_f32_16x16x128_f8f6f4 v[104:107], a[16:23], v[208:215], 0\n\t"
      "ds_read_b64_tr_b8 a[152:153], v20 offset:16384\n\t"
      "ds_read_b64_tr_b8 a[154:155], v20 offset:17408\n\t"
      "v_mfma_f32_16x16x128_f8f6f4 v[108:111], a[16:23], v[216:223], 0\n\t"
      "s_add_u32 m0, s51, 12288\n\t"
      "s_add_u32 s54, s54, 0x20000\n\t"
      "buffer_load_dwordx4 v12, s[28:31], s54 offen lds\n\t"
      "ds_read_b64_tr_b8 a[160:161], v21 offset:16384\n\t"
      "ds_read_b64_tr_b8 a[162:163], v21 offset:17408\n\t"
      "v_mfma_f32_16x16x128_f8f6f4 v[112:115], a[16:23], v[224:231], 0\n\t"
      "ds_read_b64_tr_b8 a[168:169], v22 offset:16384\n\t"
      "ds_read_b64_tr_b8 a[170:171], v22 offset:17408\n\t"
      "v_mfma_f32_16x16x128_f8f6f4 v[116:119], a[16:23], v[232:239], 0\n\t"
      "buffer_load_dwordx4 v13, s[28:31], s54 offen offset:1024 lds\n\t"
      "ds_read_b64_tr_b8 a[176:177], v23 offset:16384\n\t"
      "ds_read_b64_tr_b8 a[178:179], v23 offset:17408\n\t"
      "v_mfma_f32_16x16x128_f8f6f4 v[120:123], a[16:23], v[240:247], 0\n\t"
      "buffer_load_dwordx4 v14, s[28:31], s54 offen offset:2048 lds\n\t"
      "v_mfma_f32_16x16x128_f8f6f4 v[124:127], a[16:23], v[248:255], 0\n\t"
      "v_mfma_f32_16x16x128_f8f6f4 v[128:131], a[24:31], v[192:199], 0\n\t"
      "buffer_load_dwordx4 v15, s[28:31], s54 offen offset:3072 lds\n\t"
      "v_mfma_f32_16x16x128_f8f6f4 v[132:135], a[24:31], v[200:207], 0\n\t"
      "v_mfma_f32_16x16x128_f8f6f4 v[136:139], a[24:31], v[208:215], 0\n\t"
      "v_readfirstlane_b32 s59, v25\n\t"
      "s_nop 0\n\t"
      "s_add_u32 s59, s59, 1\n\t"
      "s_bitcmp1_b32 s59, 0\n\t"
      "s_cselect_b32 s60, 327680, 0\n\t"
      "s_bfe_u32 s62, s24, 131080\n\t"
      "s_mul_i32 s62, s62, 10240\n\t"
      "s_bfe_u32 s55, s24, 196618\n\t"
      "s_lshl_b32 s55, s55, 20\n\t"
      "s_add_u32 s62, s62, s60\n\t"
      "s_add_u32 s62, s62, s55\n\t"
      "buffer_load_dword v24, v9, s[32:35], s62 offen sc0 sc1\n\t"
      "v_mfma_f32_16x16x128_f8f6f4 v[140:143], a[24:31], v[216:223], 0\n\t"
      "s_waitcnt lgkmcnt(0)\n\t"
      "s_bfe_u32 s54, s24, 524312\n\t"
      "s_add_u32 m0, s51, 16384\n\t"
      "s_lshl_b32 s54, s54, 19\n\t"
      "s_xor_b32 s54, s54, s26\n\t"
      "s_xor_b32 s54, s54, 0x40000\n\t"
      "buffer_load_dwordx4 v12, s[28:31], s54 offen lds\n\t"
      "v_mfma_f32_16x16x128_f8f6f4 v[144:147], a[24:31], v[224:231], 0\n\t"
      "buffer_load_dwordx4 v13, s[28:31], s54 offen offset:1024 lds\n\t"
      "v_mfma_f32_16x16x128_f8f6f4 v[148:151], a[24:31], v[232:239], 0\n\t"
      "v_mfma_f32_16x16x128_f8f6f4 v[152:155], a[24:31], v[240:247], 0\n\t"
      "buffer_load_dwordx4 v14, s[28:31], s54 offen offset:2048 lds\n\t"
      "v_mfma_f32_16x16x128_f8f6f4 v[156:159], a[24:31], v[248:255], 0\n\t"
      "buffer_load_dwordx4 v15, s[28:31], s54 offen offset:3072 lds\n\t"
      "v_mfma_f32_16x16x128_f8f6f4 v[160:163], a[32:39], v[192:199], 0\n\t"
      "v_mfma_f32_16x16x128_f8f6f4 v[164:167], a[32:39], v[200:207], 0\n\t"
      "s_add_u32 m0, s51, 20480\n\t"
      "s_add_u32 s54, s54, 0x20000\n\t"
      "buffer_load_dwordx4 v12, s[28:31], s54 offen lds\n\t"
      "v_mfma_f32_16x16x128_f8f6f4 v[168:171], a[32:39], v[208:215], 0\n\t"
      "v_mfma_f32_16x16x128_f8f6f4 v[172:175], a[32:39], v[216:223], 0\n\t"
      "buffer_load_dwordx4 v13, s[28:31], s54 offen offset:1024 lds\n\t"
      "v_mfma_f32_16x16x128_f8f6f4 v[176:179], a[32:39], v[224:231], 0\n\t"
      "buffer_load_dwordx4 v14, s[28:31], s54 offen offset:2048 lds\n\t"
      "v_mfma_f32_16x16x128_f8f6f4 v[180:183], a[32:39], v[232:239], 0\n\t"
      "v_mfma_f32_16x16x128_f8f6f4 v[184:187], a[32:39], v[240:247], 0\n\t"
      "buffer_load_dwordx4 v15, s[28:31], s54 offen offset:3072 lds\n\t"
      "v_mfma_f32_16x16x128_f8f6f4 v[188:191], a[32:39], v[248:255], 0\n\t"
      "s_barrier\n\t"
      "s_waitcnt vmcnt(33)\n\t"
      "ds_read_b64_tr_b8 a[124:125], v16 offset:24576\n\t"
      "ds_read_b64_tr_b8 a[126:127], v16 offset:25600\n\t"
      "ds_read_b64_tr_b8 a[132:133], v17 offset:24576\n\t"
      "ds_read_b64_tr_b8 a[134:135], v17 offset:25600\n\t"
      "ds_read_b64_tr_b8 a[140:141], v18 offset:24576\n\t"
      "ds_read_b64_tr_b8 a[142:143], v18 offset:25600\n\t"
      "ds_read_b64_tr_b8 a[148:149], v19 offset:24576\n\t"
      "ds_read_b64_tr_b8 a[150:151], v19 offset:25600\n\t"
      "ds_read_b64_tr_b8 a[156:157], v20 offset:24576\n\t"
      "ds_read_b64_tr_b8 a[158:159], v20 offset:25600\n\t"
      "ds_read_b64_tr_b8 a[164:165], v21 offset:24576\n\t"
      "ds_read_b64_tr_b8 a[166:167], v21 offset:25600\n\t"
      "ds_read_b64_tr_b8 a[172:173], v22 offset:24576\n\t"
      "ds_read_b64_tr_b8 a[174:175], v22 offset:25600\n\t"
      "ds_read_b64_tr_b8 a[180:181], v23 offset:24576\n\t"
      "ds_read_b64_tr_b8 a[182:183], v23 offset:25600\n\t"
      "s_waitcnt lgkmcnt(0)\n\t"
      "s_bfe_u32 s54, s25, 524288\n\t"
      "s_add_u32 m0, s51, 24576\n\t"
      "s_lshl_b32 s54, s54, 19\n\t"
      "s_xor_b32 s54, s54, s26\n\t"
      "buffer_load_dwordx4 v12, s[28:31], s54 offen lds\n\t"
      "buffer_load_dwordx4 v13, s[28:31], s54 offen offset:1024 lds\n\t"
      "buffer_load_dwordx4 v14, s[28:31], s54 offen offset:2048 lds\n\t"
      "buffer_load_dwordx4 v15, s[28:31], s54 offen offset:3072 lds\n\t"
      "s_waitcnt vmcnt(12)\n\t"
      "v_cmp_le_u32 vcc, s59, v24\n\t"
      "s_mov_b32 s63, s62\n\t"
      "s_cmp_lg_u32 vcc_lo, 3\n\t"
      "s_cbranch_scc1 DEEP_%=\n\t"
      "buffer_load_dwordx4 a[40:43], v7, s[32:35], s63 offen sc0 sc1\n\t"
      "buffer_load_dwordx4 a[44:47], v3, s[32:35], s63 offen sc0 sc1\n\t"
      "buffer_load_dwordx4 a[48:51], v7, s[32:35], s63 offen offset:2048 sc0 sc1\n\t"
      "buffer_load_dwordx4 a[52:55], v3, s[32:35], s63 offen offset:2048 sc0 sc1\n\t"
      "s_add_u32 s63, s63, 0x1000\n\t"
      "buffer_load_dwordx4 a[56:59], v7, s[32:35], s63 offen sc0 sc1\n\t"
      "buffer_load_dwordx4 a[60:63], v3, s[32:35], s63 offen sc0 sc1\n\t"
      "buffer_load_dwordx4 a[64:67], v7, s[32:35], s63 offen offset:2048 sc0 sc1\n\t"
      "buffer_load_dwordx4 a[68:71], v3, s[32:35], s63 offen offset:2048 sc0 sc1\n\t"
      "s_add_u32 s63, s63, 0x1000\n\t"
      "buffer_load_dwordx4 a[72:75], v8, s[32:35], s63 offen sc0 sc1\n\t"
      "buffer_load_dwordx4 a[76:79], v4, s[32:35], s63 offen sc0 sc1\n\t"
      "s_add_u32 m0, s51, 28672\n\t"
      "s_add_u32 s54, s54, 0x20000\n\t"
      "buffer_load_dwordx4 v12, s[28:31], s54 offen lds\n\t"
      "buffer_load_dwordx4 v13, s[28:31], s54 offen offset:1024 lds\n\t"
      "buffer_load_dwordx4 v14, s[28:31], s54 offen offset:2048 lds\n\t"
      "buffer_load_dwordx4 v15, s[28:31], s54 offen offset:3072 lds\n\t"
      "ds_read_b64_tr_b8 a[184:185], v16 offset:32768\n\t"
      "ds_read_b64_tr_b8 a[186:187], v16 offset:33792\n\t"
      "ds_read_b64_tr_b8 a[192:193], v17 offset:32768\n\t"
      "ds_read_b64_tr_b8 a[194:195], v17 offset:33792\n\t"
      "ds_read_b64_tr_b8 a[200:201], v18 offset:32768\n\t"
      "ds_read_b64_tr_b8 a[202:203], v18 offset:33792\n\t"
      "ds_read_b64_tr_b8 a[208:209], v19 offset:32768\n\t"
      "ds_read_b64_tr_b8 a[210:211], v19 offset:33792\n\t"
      "ds_read_b64_tr_b8 a[216:217], v20 offset:32768\n\t"
      "ds_read_b64_tr_b8 a[218:219], v20 offset:33792\n\t"
      "ds_read_b64_tr_b8 a[224:225], v21 offset:32768\n\t"
      "ds_read_b64_tr_b8 a[226:227], v21 offset:33792\n\t"
      "ds_read_b64_tr_b8 a[232:233], v22 offset:32768\n\t"
      "ds_read_b64_tr_b8 a[234:235], v22 offset:33792\n\t"
      "ds_read_b64_tr_b8 a[240:241], v23 offset:32768\n\t"
      "ds_read_b64_tr_b8 a[242:243], v23 offset:33792\n\t"
      "s_waitcnt vmcnt(12)\n\t"
      "v_mfma_f32_16x16x128_f8f6f4 v[32:35], a[40:47], a[120:127], v[32:35]\n\t"
      "ds_read_b64_tr_b8 a[188:189], v16 offset:0\n\t"
      "ds_read_b64_tr_b8 a[190:191], v16 offset:1024\n\t"
      "s_waitcnt lgkmcnt(2)\n\t"
      "s_bfe_u32 s54, s25, 524288\n\t"
      "s_add_u32 m0, s51, 32768\n\t"
      "s_lshl_b32 s54, s54, 19\n\t"
      "s_xor_b32 s54, s54, s26\n\t"
      "s_xor_b32 s54, s54, 0x40000\n\t"
      "buffer_load_dwordx4 v12, s[28:31], s54 offen lds\n\t"
      "v_mfma_f32_16x16x128_f8f6f4 v[36:39], a[40:47], a[128:135], v[36:39]\n\t"
      "ds_read_b64_tr_b8 a[196:197], v17 offset:0\n\t"
      "ds_read_b64_tr_b8 a[198:199], v17 offset:1024\n\t"
      "buffer_load_dwordx4 v13, s[28:31], s54 offen offset:1024 lds\n\t"
      "v_mfma_f32_16x16x128_f8f6f4 v[40:43], a[40:47], a[136:143], v[40:43]\n\t"
      "ds_read_b64_tr_b8 a[204:205], v18 offset:0\n\t"
      "ds_read_b64_tr_b8 a[206:207], v18 offset:1024\n\t"
      "v_mfma_f32_16x16x128_f8f6f4 v[44:47], a[40:47], a[144:151], v[44:47]\n\t"
      "s_bfe_u32 s62, s24, 131088\n\t"
      "s_mul_i32 s62, s62, 10240\n\t"
      "s_bfe_u32 s55, s24, 196626\n\t"
      "s_lshl_b32 s55, s55, 20\n\t"
      "s_add_u32 s62, s62, s60\n\t"
      "s_add_u32 s62, s62, s55\n\t"
      "buffer_load_dword v24, v9, s[32:35], s62 offen sc0 sc1\n\t"
      "ds_read_b64_tr_b8 a[212:213], v19 offset:0\n\t"
      "ds_read_b64_tr_b8 a[214:215], v19 offset:1024\n\t"
      "buffer_load_dwordx4 v14, s[28:31], s54 offen offset:2048 lds\n\t"
      "v_mfma_f32_16x16x128_f8f6f4 v[48:51], a[40:47], a[152:159], v[48:51]\n\t"
      "ds_read_b64_tr_b8 a[220:221], v20 offset:0\n\t"
      "ds_read_b64_tr_b8 a[222:223], v20 offset:1024\n\t"
      "v_mfma_f32_16x16x128_f8f6f4 v[52:55], a[40:47], a[160:167], v[52:55]\n\t"
      "ds_read_b64_tr_b8 a[228:229], v21 offset:0\n\t"
      "ds_read_b64_tr_b8 a[230:231], v21 offset:1024\n\t"
      "buffer_load_dwordx4 v15, s[28:31], s54 offen offset:3072 lds\n\t"
      "v_mfma_f32_16x16x128_f8f6f4 v[56:59], a[40:47], a[168:175], v[56:59]\n\t"
      "ds_read_b64_tr_b8 a[236:237], v22 offset:0\n\t"
      "ds_read_b64_tr_b8 a[238:239], v22 offset:1024\n\t"
      "v_mfma_f32_16x16x128_f8f6f4 v[60:63], a[40:47], a[176:183], v[60:63]\n\t"
      "ds_read_b64_tr_b8 a[244:245], v23 offset:0\n\t"
      "ds_read_b64_tr_b8 a[246:247], v23 offset:1024\n\t"
      "s_add_u32 m0, s51, 36864\n\t"
      "s_add_u32 s54, s54, 0x20000\n\t"
      "buffer_load_dwordx4 v12, s[28:31], s54 offen lds\n\t"
      "s_waitcnt vmcnt(16)\n\t"
      "v_mfma_f32_16x16x128_f8f6f4 v[64:67], a[48:55], a[120:127], v[64:67]\n\t"
      "ds_read_b64_tr_b8 v[192:193], v16 offset:8192\n\t"
      "ds_read_b64_tr_b8 v[194:195], v16 offset:9216\n\t"
      "v_mfma_f32_16x16x128_f8f6f4 v[68:71], a[48:55], a[128:135], v[68:71]\n\t"
      "ds_read_b64_tr_b8 v[200:201], v17 offset:8192\n\t"
      "ds_read_b64_tr_b8 v[202:203], v17 offset:9216\n\t"
      "buffer_load_dwordx4 v13, s[28:31], s54 offen offset:1024 lds\n\t"
      "v_mfma_f32_16x16x128_f8f6f4 v[72:75], a[48:55], a[136:143], v[72:75]\n\t"
      "ds_read_b64_tr_b8 v[208:209], v18 offset:8192\n\t"
      "ds_read_b64_tr_b8 v[210:211], v18 offset:9216\n\t"
      "buffer_load_dwordx4 v14, s[28:31], s54 offen offset:2048 lds\n\t"
      "v_mfma_f32_16x16x128_f8f6f4 v[76:79], a[48:55], a[144:151], v[76:79]\n\t"
      "ds_read_b64_tr_b8 v[216:217], v19 offset:8192\n\t"
      "ds_read_b64_tr_b8 v[218:219], v19 offset:9216\n\t"
      "v_mfma_f32_16x16x128_f8f6f4 v[80:83], a[48:55], a[152:159], v[80:83]\n\t"
      "ds_read_b64_tr_b8 v[224:225], v20 offset:8192\n\t"
      "ds_read_b64_tr_b8 v[226:227], v20 offset:9216\n\t"
      "buffer_load_dwordx4 v15, s[28:31], s54 offen offset:3072 lds\n\t"
      "v_mfma_f32_16x16x128_f8f6f4 v[84:87], a[48:55], a[160:167], v[84:87]\n\t"
      "ds_read_b64_tr_b8 v[232:233], v21 offset:8192\n\t"
      "ds_read_b64_tr_b8 v[234:235], v21 offset:9216\n\t"
      "v_mfma_f32_16x16x128_f8f6f4 v[88:91], a[48:55], a[168:175], v[88:91]\n\t"
      "ds_read_b64_tr_b8 v[240:241], v22 offset:8192\n\t"
      "ds_read_b64_tr_b8 v[242:243], v22 offset:9216\n\t"
      "s_waitcnt lgkmcnt(14)\n\t"
      "s_bfe_u32 s54, s25, 524296\n\t"
      "s_add_u32 m0, s51, 0\n\t"
      "s_lshl_b32 s54, s54, 19\n\t"
      "s_xor_b32 s54, s54, s26\n\t"
      "buffer_load_dwordx4 v12, s[28:31], s54 offen lds\n\t"
      "v_mfma_f32_16x16x128_f8f6f4 v[92:95], a[48:55], a[176:183], v[92:95]\n\t"
      "ds_read_b64_tr_b8 v[248:249], v23 offset:8192\n\t"
      "ds_read_b64_tr_b8 v[250:251], v23 offset:9216\n\t"
      "s_waitcnt vmcnt(18)\n\t"
      "v_mfma_f32_16x16x128_f8f6f4 v[96:99], a[56:63], a[120:127], v[96:99]\n\t"
      "ds_read_b64_tr_b8 v[196:197], v16 offset:16384\n\t"
      "ds_read_b64_tr_b8 v[198:199], v16 offset:17408\n\t"
      "buffer_load_dwordx4 v13, s[28:31], s54 offen offset:1024 lds\n\t"
      "v_mfma_f32_16x16x128_f8f6f4 v[100:103], a[56:63], a[128:135], v[100:103]\n\t"
      "ds_read_b64_tr_b8 v[204:205], v17 offset:16384\n\t"
      "ds_read_b64_tr_b8 v[206:207], v17 offset:17408\n\t"
      "v_mfma_f32_16x16x128_f8f6f4 v[104:107], a[56:63], a[136:143], v[104:107]\n\t"
      "ds_read_b64_tr_b8 v[212:213], v18 offset:16384\n\t"
      "ds_read_b64_tr_b8 v[214:215], v18 offset:17408\n\t"
      "buffer_load_dwordx4 v14, s[28:31], s54 offen offset:2048 lds\n\t"
      "v_mfma_f32_16x16x128_f8f6f4 v[108:111], a[56:63], a[144:151], v[108:111]\n\t"
      "ds_read_b64_tr_b8 v[220:221], v19 offset:16384\n\t"
      "ds_read_b64_tr_b8 v[222:223], v19 offset:17408\n\t"
      "v_mfma_f32_16x16x128_f8f6f4 v[112:115], a[56:63], a[152:159], v[112:115]\n\t"
      "ds_read_b64_tr_b8 v[228:229], v20 offset:16384\n\t"
      "ds_read_b64_tr_b8 v[230:231], v20 offset:17408\n\t"
      "buffer_load_dwordx4 v15, s[28:31], s54 offen offset:3072 lds\n\t"
      "v_mfma_f32_16x16x128_f8f6f4 v[116:119], a[56:63], a[160:167], v[116:119]\n\t"
      "ds_read_b64_tr_b8 v[236:237], v21 offset:16384\n\t"
      "ds_read_b64_tr_b8 v[238:239], v21 offset:17408\n\t"
      "s_waitcnt vmcnt(10)\n\t"
      "v_cmp_le_u32 vcc, s59, v24\n\t"
      "s_mov_b32 s63, s62\n\t"
      "s_cmp_lg_u32 vcc_lo, 3\n\t"
      "s_cbranch_scc1 ALT2_%=\n\t"
      "buffer_load_dwordx4 a[80:83], v7, s[32:35], s63 offen sc0 sc1\n\t"
      "v_mfma_f32_16x16x128_f8f6f4 v[120:123], a[56:63], a[168:175], v[120:123]\n\t"
      "ds_read_b64_tr_b8 v[244:245], v22 offset:16384\n\t"
      "ds_read_b64_tr_b8 v[246:247], v22 offset:17408\n\t"
      "v_mfma_f32_16x16x128_f8f6f4 v[124:127], a[56:63], a[176:183], v[124:127]\n\t"
      "ds_read_b64_tr_b8 v[252:253], v23 offset:16384\n\t"
      "ds_read_b64_tr_b8 v[254:255], v23 offset:17408\n\t"
      "buffer_load_dwordx4 a[84:87], v3, s[32:35], s63 offen sc0 sc1\n\t"
      "v_mfma_f32_16x16x128_f8f6f4 v[128:131], a[64:71], a[120:127], v[128:131]\n\t"
      "v_mfma_f32_16x16x128_f8f6f4 v[132:135], a[64:71], a[128:135], v[132:135]\n\t"
      "buffer_load_dwordx4 a[88:91], v7, s[32:35], s63 offen offset:2048 sc0 sc1\n\t"
      "v_mfma_f32_16x16x128_f8f6f4 v[136:139], a[64:71], a[136:143], v[136:139]\n\t"
      "v_mfma_f32_16x16x128_f8f6f4 v[140:143], a[64:71], a[144:151], v[140:143]\n\t"
      "buffer_load_dwordx4 a[92:95], v3, s[32:35], s63 offen offset:2048 sc0 sc1\n\t"
      "v_mfma_f32_16x16x128_f8f6f4 v[144:147], a[64:71], a[152:159], v[144:147]\n\t"
      "v_mfma_f32_16x16x128_f8f6f4 v[148:151], a[64:71], a[160:167], v[148:151]\n\t"
      "s_add_u32 s63, s63, 0x1000\n\t"
      "buffer_load_dwordx4 a[96:99], v7, s[32:35], s63 offen sc0 sc1\n\t"
      "v_mfma_f32_16x16x128_f8f6f4 v[152:155], a[64:71], a[168:175], v[152:155]\n\t"
      "buffer_load_dwordx4 a[100:103], v3, s[32:35], s63 offen sc0 sc1\n\t"
      "v_mfma_f32_16x16x128_f8f6f4 v[156:159], a[64:71], a[176:183], v[156:159]\n\t"
      "v_mfma_f32_16x16x128_f8f6f4 v[160:163], a[72:79], a[120:127], v[160:163]\n\t"
      "buffer_load_dwordx4 a[104:107], v7, s[32:35], s63 offen offset:2048 sc0 sc1\n\t"
      "v_mfma_f32_16x16x128_f8f6f4 v[164:167], a[72:79], a[128:135], v[164:167]\n\t"
      "v_mfma_f32_16x16x128_f8f6f4 v[168:171], a[72:79], a[136:143], v[168:171]\n\t"
      "buffer_load_dwordx4 a[108:111], v3, s[32:35], s63 offen offset:2048 sc0 sc1\n\t"
      "v_mfma_f32_16x16x128_f8f6f4 v[172:175], a[72:79], a[144:151], v[172:175]\n\t"
      "v_mfma_f32_16x16x128_f8f6f4 v[176:179], a[72:79], a[152:159], v[176:179]\n\t"
      "s_add_u32 s63, s63, 0x1000\n\t"
      "buffer_load_dwordx4 a[112:115], v8, s[32:35], s63 offen sc0 sc1\n\t"
      "v_mfma_f32_16x16x128_f8f6f4 v[180:183], a[72:79], a[160:167], v[180:183]\n\t"
      "buffer_load_dwordx4 a[116:119], v4, s[32:35], s63 offen sc0 sc1\n\t"
      "v_mfma_f32_16x16x128_f8f6f4 v[184:187], a[72:79], a[168:175], v[184:187]\n\t"
      "v_mfma_f32_16x16x128_f8f6f4 v[188:191], a[72:79], a[176:183], v[188:191]\n\t"
      "JOIN2_%=:\n\t"
      "s_waitcnt vmcnt(8)\n\t"
      "v_mfma_f32_16x16x128_f8f6f4 v[32:35], a[80:87], a[184:191], v[32:35]\n\t"
      "ds_read_b64_tr_b8 a[120:121], v16 offset:24576\n\t"
      "ds_read_b64_tr_b8 a[122:123], v16 offset:25600\n\t"
      "s_add_u32 m0, s51, 4096\n\t"
      "s_add_u32 s54, s54, 0x20000\n\t"
      "buffer_load_dwordx4 v12, s[28:31], s54 offen lds\n\t"
      "v_mfma_f32_16x16x128_f8f6f4 v[36:39], a[80:87], a[192:199], v[36:39]\n\t"
      "ds_read_b64_tr_b8 a[128:129], v17 offset:24576\n\t"
      "ds_read_b64_tr_b8 a[130:131], v17 offset:25600\n\t"
      "buffer_load_dwordx4 v13, s[28:31], s54 offen offset:1024 lds\n\t"
      "v_mfma_f32_16x16x128_f8f6f4 v[40:43], a[80:87], a[200:207], v[40:43]\n\t"
      "ds_read_b64_tr_b8 a[136:137], v18 offset:24576\n\t"
      "ds_read_b64_tr_b8 a[138:139], v18 offset:25600\n\t"
      "v_mfma_f32_16x16x128_f8f6f4 v[44:47], a[80:87], a[208:215], v[44:47]\n\t"
      "s_bfe_u32 s62, s24, 131096\n\t"
      "s_mul_i32 s62, s62, 10240\n\t"
      "s_bfe_u32 s55, s24, 196634\n\t"
      "s_lshl_b32 s55, s55, 20\n\t"
      "s_add_u32 s62, s62, s60\n\t"
      "s_add_u32 s62, s62, s55\n\t"
      "buffer_load_dword v24, v9, s[32:35], s62 offen sc0 sc1\n\t"
      "ds_read_b64_tr_b8 a[144:145], v19 offset:24576\n\t"
      "ds_read_b64_tr_b8 a[146:147], v19 offset:25600\n\t"
      "buffer_load_dwordx4 v14, s[28:31], s54 offen offset:2048 lds\n\t"
      "v_mfma_f32_16x16x128_f8f6f4 v[48:51], a[80:87], a[216:223], v[48:51]\n\t"
      "ds_read_b64_tr_b8 a[152:153], v20 offset:24576\n\t"
      "ds_read_b64_tr_b8 a[154:155], v20 offset:25600\n\t"
      "v_mfma_f32_16x16x128_f8f6f4 v[52:55], a[80:87], a[224:231], v[52:55]\n\t"
      "ds_read_b64_tr_b8 a[160:161], v21 offset:24576\n\t"
      "ds_read_b64_tr_b8 a[162:163], v21 offset:25600\n\t"
      "buffer_load_dwordx4 v15, s[28:31], s54 offen offset:3072 lds\n\t"
      "v_mfma_f32_16x16x128_f8f6f4 v[56:59], a[80:87], a[232:239], v[56:59]\n\t"
      "ds_read_b64_tr_b8 a[168:169], v22 offset:24576\n\t"
      "ds_read_b64_tr_b8 a[170:171], v22 offset:25600\n\t"
      "v_mfma_f32_16x16x128_f8f6f4 v[60:63], a[80:87], a[240:247], v[60:63]\n\t"
      "ds_read_b64_tr_b8 a[176:177], v23 offset:24576\n\t"
      "ds_read_b64_tr_b8 a[178:179], v23 offset:25600\n\t"
      "s_waitcnt lgkmcnt(15)\n\t"
      "s_bfe_u32 s54, s25, 524296\n\t"
      "s_add_u32 m0, s51, 8192\n\t"
      "s_lshl_b32 s54, s54, 19\n\t"
      "s_xor_b32 s54, s54, s26\n\t"
      "s_xor_b32 s54, s54, 0x40000\n\t"
      "buffer_load_dwordx4 v12, s[28:31], s54 offen lds\n\t"
      "s_waitcnt vmcnt(12)\n\t"
      "v_mfma_f32_16x16x128_f8f6f4 v[64:67], a[88:95], a[184:191], v[64:67]\n\t"
      "ds_read_b64_tr_b8 a[124:125], v16 offset:32768\n\t"
      "ds_read_b64_tr_b8 a[126:127], v16 offset:33792\n\t"
      "v_mfma_f32_16x16x128_f8f6f4 v[68:71], a[88:95], a[192:199], v[68:71]\n\t"
      "ds_read_b64_tr_b8 a[132:133], v17 offset:32768\n\t"
      "ds_read_b64_tr_b8 a[134:135], v17 offset:33792\n\t"
      "buffer_load_dwordx4 v13, s[28:31], s54 offen offset:1024 lds\n\t"
      "v_mfma_f32_16x16x128_f8f6f4 v[72:75], a[88:95], a[200:207], v[72:75]\n\t"
      "ds_read_b64_tr_b8 a[140:141], v18 offset:32768\n\t"
      "ds_read_b64_tr_b8 a[142:143], v18 offset:33792\n\t"
      "buffer_load_dwordx4 v14, s[28:31], s54 offen offset:2048 lds\n\t"
      "v_mfma_f32_16x16x128_f8f6f4 v[76:79], a[88:95], a[208:215], v[76:79]\n\t"
      "ds_read_b64_tr_b8 a[148:149], v19 offset:32768\n\t"
      "ds_read_b64_tr_b8 a[150:151], v19 offset:33792\n\t"
      "v_mfma_f32_16x16x128_f8f6f4 v[80:83], a[88:95], a[216:223], v[80:83]\n\t"
      "ds_read_b64_tr_b8 a[156:157], v20 offset:32768\n\t"
      "ds_read_b64_tr_b8 a[158:159], v20 offset:33792\n\t"
      "buffer_load_dwordx4 v15, s[28:31], s54 offen offset:3072 lds\n\t"
      "v_mfma_f32_16x16x128_f8f6f4 v[84:87], a[88:95], a[224:231], v[84:87]\n\t"
      "ds_read_b64_tr_b8 a[164:165], v21 offset:32768\n\t"
      "ds_read_b64_tr_b8 a[166:167], v21 offset:33792\n\t"
      "v_mfma_f32_16x16x128_f8f6f4 v[88:91], a[88:95], a[232:239], v[88:91]\n\t"
      "ds_read_b64_tr_b8 a[172:173], v22 offset:32768\n\t"
      "ds_read_b64_tr_b8 a[174:175], v22 offset:33792\n\t"
      "s_add_u32 m0, s51, 12288\n\t"
      "s_add_u32 s54, s54, 0x20000\n\t"
      "buffer_load_dwordx4 v12, s[28:31], s54 offen lds\n\t"
      "v_mfma_f32_16x16x128_f8f6f4 v[92:95], a[88:95], a[240:247], v[92:95]\n\t"
      "ds_read_b64_tr_b8 a[180:181], v23 offset:32768\n\t"
      "ds_read_b64_tr_b8 a[182:183], v23 offset:33792\n\t"
      "s_waitcnt vmcnt(14)\n\t"
      "v_mfma_f32_16x16x128_f8f6f4 v[96:99], a[96:103], a[184:191], v[96:99]\n\t"
      "buffer_load_dwordx4 v13, s[28:31], s54 offen offset:1024 lds\n\t"
      "v_mfma_f32_16x16x128_f8f6f4 v[100:103], a[96:103], a[192:199], v[100:103]\n\t"
      "v_mfma_f32_16x16x128_f8f6f4 v[104:107], a[96:103], a[200:207], v[104:107]\n\t"
      "buffer_load_dwordx4 v14, s[28:31], s54 offen offset:2048 lds\n\t"
      "v_mfma_f32_16x16x128_f8f6f4 v[108:111], a[96:103], a[208:215], v[108:111]\n\t"
      "v_mfma_f32_16x16x128_f8f6f4 v[112:115], a[96:103], a[216:223], v[112:115]\n\t"
      "buffer_load_dwordx4 v15, s[28:31], s54 offen offset:3072 lds\n\t"
      "v_mfma_f32_16x16x128_f8f6f4 v[116:119], a[96:103], a[224:231], v[116:119]\n\t"
      "s_waitcnt vmcnt(10)\n\t"
      "v_cmp_le_u32 vcc, s59, v24\n\t"
      "s_mov_b32 s63, s62\n\t"
      "s_cmp_lg_u32 vcc_lo, 3\n\t"
      "s_cbranch_scc1 ALT3_%=\n\t"
      "buffer_load_dwordx4 a[0:3], v7, s[32:35], s63 offen sc0 sc1\n\t"
      "v_mfma_f32_16x16x128_f8f6f4 v[120:123], a[96:103], a[232:239], v[120:123]\n\t"
      "v_mfma_f32_16x16x128_f8f6f4 v[124:127], a[96:103], a[240:247], v[124:127]\n\t"
      "buffer_load_dwordx4 a[4:7], v3, s[32:35], s63 offen sc0 sc1\n\t"
      "v_mfma_f32_16x16x128_f8f6f4 v[128:131], a[104:111], a[184:191], v[128:131]\n\t"
      "v_mfma_f32_16x16x128_f8f6f4 v[132:135], a[104:111], a[192:199], v[132:135]\n\t"
      "buffer_load_dwordx4 a[8:11], v7, s[32:35], s63 offen offset:2048 sc0 sc1\n\t"
      "v_mfma_f32_16x16x128_f8f6f4 v[136:139], a[104:111], a[200:207], v[136:139]\n\t"
      "v_mfma_f32_16x16x128_f8f6f4 v[140:143], a[104:111], a[208:215], v[140:143]\n\t"
      "buffer_load_dwordx4 a[12:15], v3, s[32:35], s63 offen offset:2048 sc0 sc1\n\t"
      "v_mfma_f32_16x16x128_f8f6f4 v[144:147], a[104:111], a[216:223], v[144:147]\n\t"
      "v_mfma_f32_16x16x128_f8f6f4 v[148:151], a[104:111], a[224:231], v[148:151]\n\t"
      "s_add_u32 s63, s63, 0x1000\n\t"
      "buffer_load_dwordx4 a[16:19], v7, s[32:35], s63 offen sc0 sc1\n\t"
      "v_mfma_f32_16x16x128_f8f6f4 v[152:155], a[104:111], a[232:239], v[152:155]\n\t"
      "buffer_load_dwordx4 a[20:23], v3, s[32:35], s63 offen sc0 sc1\n\t"
      "v_mfma_f32_16x16x128_f8f6f4 v[156:159], a[104:111], a[240:247], v[156:159]\n\t"
      "v_mfma_f32_16x16x128_f8f6f4 v[160:163], a[112:119], a[184:191], v[160:163]\n\t"
      "buffer_load_dwordx4 a[24:27], v7, s[32:35], s63 offen offset:2048 sc0 sc1\n\t"
      "v_mfma_f32_16x16x128_f8f6f4 v[164:167], a[112:119], a[192:199], v[164:167]\n\t"
      "v_mfma_f32_16x16x128_f8f6f4 v[168:171], a[112:119], a[200:207], v[168:171]\n\t"
      "buffer_load_dwordx4 a[28:31], v3, s[32:35], s63 offen offset:2048 sc0 sc1\n\t"
      "v_mfma_f32_16x16x128_f8f6f4 v[172:175], a[112:119], a[208:215], v[172:175]\n\t"
      "v_mfma_f32_16x16x128_f8f6f4 v[176:179], a[112:119], a[216:223], v[176:179]\n\t"
      "s_add_u32 s63, s63, 0x1000\n\t"
      "buffer_load_dwordx4 a[32:35], v8, s[32:35], s63 offen sc0 sc1\n\t"
      "v_mfma_f32_16x16x128_f8f6f4 v[180:183], a[112:119], a[224:231], v[180:183]\n\t"
      "buffer_load_dwordx4 a[36:39], v4, s[32:35], s63 offen sc0 sc1\n\t"
      "v_mfma_f32_16x16x128_f8f6f4 v[184:187], a[112:119], a[232:239], v[184:187]\n\t"
      "v_mfma_f32_16x16x128_f8f6f4 v[188:191], a[112:119], a[240:247], v[188:191]\n\t"
      "JOIN3_%=:\n\t"
      "CONV_%=:\n\t"
      "s_waitcnt vmcnt(8)\n\t"
      "v_mfma_f32_16x16x128_f8f6f4 v[32:35], a[0:7], v[192:199], v[32:35]\n\t"
      "s_bfe_u32 s62, s25, 131072\n\t"
      "s_mul_i32 s62, s62, 10240\n\t"
      "s_bfe_u32 s55, s25, 196610\n\t"
      "s_lshl_b32 s55, s55, 20\n\t"
      "s_add_u32 s62, s62, s60\n\t"
      "s_add_u32 s62, s62, s55\n\t"
      "buffer_load_dword v24, v9, s[32:35], s62 offen sc0 sc1\n\t"
      "ds_read_b64_tr_b8 a[184:185], v16 offset:0\n\t"
      "ds_read_b64_tr_b8 a[186:187], v16 offset:1024\n\t"
      "s_bfe_u32 s54, s25, 524304\n\t"
      "s_add_u32 m0, s51, 16384\n\t"
      "s_lshl_b32 s54, s54, 19\n\t"
      "s_xor_b32 s54, s54, s26\n\t"
      "buffer_load_dwordx4 v12, s[28:31], s54 offen lds\n\t"
      "v_mfma_f32_16x16x128_f8f6f4 v[36:39], a[0:7], v[200:207], v[36:39]\n\t"
      "ds_read_b64_tr_b8 a[192:193], v17 offset:0\n\t"
      "ds_read_b64_tr_b8 a[194:195], v17 offset:1024\n\t"
      "v_mfma_f32_16x16x128_f8f6f4 v[40:43], a[0:7], v[208:215], v[40:43]\n\t"
      "ds_read_b64_tr_b8 a[200:201], v18 offset:0\n\t"
      "ds_read_b64_tr_b8 a[202:203], v18 offset:1024\n\t"
      "buffer_load_dwordx4 v13, s[28:31], s54 offen offset:1024 lds\n\t"
      "v_mfma_f32_16x16x128_f8f6f4 v[44:47], a[0:7], v[216:223], v[44:47]\n\t"
      "ds_read_b64_tr_b8 a[208:209], v19 offset:0\n\t"
      "ds_read_b64_tr_b8 a[210:211], v19 offset:1024\n\t"
      "v_mfma_f32_16x16x128_f8f6f4 v[48:51], a[0:7], v[224:231], v[48:51]\n\t"
      "ds_read_b64_tr_b8 a[216:217], v20 offset:0\n\t"
      "ds_read_b64_tr_b8 a[218:219], v20 offset:1024\n\t"
      "buffer_load_dwordx4 v14, s[28:31], s54 offen offset:2048 lds\n\t"
      "v_mfma_f32_16x16x128_f8f6f4 v[52:55], a[0:7], v[232:239], v[52:55]\n\t"
      "ds_read_b64_tr_b8 a[224:225], v21 offset:0\n\t"
      "ds_read_b64_tr_b8 a[226:227], v21 offset:1024\n\t"
      "v_mfma_f32_16x16x128_f8f6f4 v[56:59], a[0:7], v[240:247], v[56:59]\n\t"
      "ds_read_b64_tr_b8 a[232:233], v22 offset:0\n\t"
      "ds_read_b64_tr_b8 a[234:235], v22 offset:1024\n\t"
      "buffer_load_dwordx4 v15, s[28:31], s54 offen offset:3072 lds\n\t"
      "v_mfma_f32_16x16x128_f8f6f4 v[60:63], a[0:7], v[248:255], v[60:63]\n\t"
      "ds_read_b64_tr_b8 a[240:241], v23 offset:0\n\t"
      "ds_read_b64_tr_b8 a[242:243], v23 offset:1024\n\t"
      "s_waitcnt vmcnt(11)\n\t"
      "v_mfma_f32_16x16x128_f8f6f4 v[64:67], a[8:15], v[192:199], v[64:67]\n\t"
      "ds_read_b64_tr_b8 a[188:189], v16 offset:8192\n\t"
      "ds_read_b64_tr_b8 a[190:191], v16 offset:9216\n\t"
      "v_mfma_f32_16x16x128_f8f6f4 v[68:71], a[8:15], v[200:207], v[68:71]\n\t"
      "ds_read_b64_tr_b8 a[196:197], v17 offset:8192\n\t"
      "ds_read_b64_tr_b8 a[198:199], v17 offset:9216\n\t"
      "s_add_u32 m0, s51, 20480\n\t"
      "s_add_u32 s54, s54, 0x20000\n\t"
      "buffer_load_dwordx4 v12, s[28:31], s54 offen lds\n\t"
      "v_mfma_f32_16x16x128_f8f6f4 v[72:75], a[8:15], v[208:215], v[72:75]\n\t"
      "ds_read_b64_tr_b8 a[204:205], v18 offset:8192\n\t"
      "ds_read_b64_tr_b8 a[206:207], v18 offset:9216\n\t"
      "v_mfma_f32_16x16x128_f8f6f4 v[76:79], a[8:15], v[216:223], v[76:79]\n\t"
      "ds_read_b64_tr_b8 a[212:213], v19 offset:8192\n\t"
      "ds_read_b64_tr_b8 a[214:215], v19 offset:9216\n\t"
      "buffer_load_dwordx4 v13, s[28:31], s54 offen offset:1024 lds\n\t"
      "v_mfma_f32_16x16x128_f8f6f4 v[80:83], a[8:15], v[224:231], v[80:83]\n\t"
      "ds_read_b64_tr_b8 a[220:221], v20 offset:8192\n\t"
      "ds_read_b64_tr_b8 a[222:223], v20 offset:9216\n\t"
      "v_mfma_f32_16x16x128_f8f6f4 v[84:87], a[8:15], v[232:239], v[84:87]\n\t"
      "ds_read_b64_tr_b8 a[228:229], v21 offset:8192\n\t"
      "ds_read_b64_tr_b8 a[230:231], v21 offset:9216\n\t"
      "buffer_load_dwordx4 v14, s[28:31], s54 offen offset:2048 lds\n\t"
      "v_mfma_f32_16x16x128_f8f6f4 v[88:91], a[8:15], v[240:247], v[88:91]\n\t"
      "ds_read_b64_tr_b8 a[236:237], v22 offset:8192\n\t"
      "ds_read_b64_tr_b8 a[238:239], v22 offset:9216\n\t"
      "v_mfma_f32_16x16x128_f8f6f4 v[92:95], a[8:15], v[248:255], v[92:95]\n\t"
      "ds_read_b64_tr_b8 a[244:245], v23 offset:8192\n\t"
      "ds_read_b64_tr_b8 a[246:247], v23 offset:9216\n\t"
      "buffer_load_dwordx4 v15, s[28:31], s54 offen offset:3072 lds\n\t"
      "s_waitcnt vmcnt(13)\n\t"
      "v_mfma_f32_16x16x128_f8f6f4 v[96:99], a[16:23], v[192:199], v[96:99]\n\t"
      "v_mfma_f32_16x16x128_f8f6f4 v[100:103], a[16:23], v[200:207], v[100:103]\n\t"
      "v_mfma_f32_16x16x128_f8f6f4 v[104:107], a[16:23], v[208:215], v[104:107]\n\t"
      "s_waitcnt vmcnt(8)\n\t"
      "v_cmp_le_u32 vcc, s59, v24\n\t"
      "s_mov_b32 s63, s62\n\t"
      "s_cmp_lg_u32 vcc_lo, 3\n\t"
      "s_cbranch_scc1 ALT4_%=\n\t"
      "buffer_load_dwordx4 a[40:43], v7, s[32:35], s63 offen sc0 sc1\n\t"
      "v_mfma_f32_16x16x128_f8f6f4 v[108:111], a[16:23], v[216:223], v[108:111]\n\t"
      "v_mfma_f32_16x16x128_f8f6f4 v[112:115], a[16:23], v[224:231], v[112:115]\n\t"
      "buffer_load_dwordx4 a[44:47], v3, s[32:35], s63 offen sc0 sc1\n\t"
      "v_mfma_f32_16x16x128_f8f6f4 v[116:119], a[16:23], v[232:239], v[116:119]\n\t"
      "v_mfma_f32_16x16x128_f8f6f4 v[120:123], a[16:23], v[240:247], v[120:123]\n\t"
      "buffer_load_dwordx4 a[48:51], v7, s[32:35], s63 offen offset:2048 sc0 sc1\n\t"
      "v_mfma_f32_16x16x128_f8f6f4 v[124:127], a[16:23], v[248:255], v[124:127]\n\t"
      "v_mfma_f32_16x16x128_f8f6f4 v[128:131], a[24:31], v[192:199], v[128:131]\n\t"
      "v_mfma_f32_16x16x128_f8f6f4 v[132:135], a[24:31], v[200:207], v[132:135]\n\t"
      "buffer_load_dwordx4 a[52:55], v3, s[32:35], s63 offen offset:2048 sc0 sc1\n\t"
      "v_mfma_f32_16x16x128_f8f6f4 v[136:139], a[24:31], v[208:215], v[136:139]\n\t"
      "v_mfma_f32_16x16x128_f8f6f4 v[140:143], a[24:31], v[216:223], v[140:143]\n\t"
      "s_add_u32 s63, s63, 0x1000\n\t"
      "buffer_load_dwordx4 a[56:59], v7, s[32:35], s63 offen sc0 sc1\n\t"
      "v_mfma_f32_16x16x128_f8f6f4 v[144:147], a[24:31], v[224:231], v[144:147]\n\t"
      "v_mfma_f32_16x16x128_f8f6f4 v[148:151], a[24:31], v[232:239], v[148:151]\n\t"
      "buffer_load_dwordx4 a[60:63], v3, s[32:35], s63 offen sc0 sc1\n\t"
      "v_mfma_f32_16x16x128_f8f6f4 v[152:155], a[24:31], v[240:247], v[152:155]\n\t"
      "v_mfma_f32_16x16x128_f8f6f4 v[156:159], a[24:31], v[248:255], v[156:159]\n\t"
      "buffer_load_dwordx4 a[64:67], v7, s[32:35], s63 offen offset:2048 sc0 sc1\n\t"
      "v_mfma_f32_16x16x128_f8f6f4 v[160:163], a[32:39], v[192:199], v[160:163]\n\t"
      "v_mfma_f32_16x16x128_f8f6f4 v[164:167], a[32:39], v[200:207], v[164:167]\n\t"
      "v_mfma_f32_16x16x128_f8f6f4 v[168:171], a[32:39], v[208:215], v[168:171]\n\t"
      "buffer_load_dwordx4 a[68:71], v3, s[32:35], s63 offen offset:2048 sc0 sc1\n\t"
      "v_mfma_f32_16x16x128_f8f6f4 v[172:175], a[32:39], v[216:223], v[172:175]\n\t"
      "v_mfma_f32_16x16x128_f8f6f4 v[176:179], a[32:39], v[224:231], v[176:179]\n\t"
      "s_add_u32 s63, s63, 0x1000\n\t"
      "buffer_load_dwordx4 a[72:75], v8, s[32:35], s63 offen sc0 sc1\n\t"
      "v_mfma_f32_16x16x128_f8f6f4 v[180:183], a[32:39], v[232:239], v[180:183]\n\t"
      "buffer_load_dwordx4 a[76:79], v4, s[32:35], s63 offen sc0 sc1\n\t"
      "v_mfma_f32_16x16x128_f8f6f4 v[184:187], a[32:39], v[240:247], v[184:187]\n\t"
      "v_mfma_f32_16x16x128_f8f6f4 v[188:191], a[32:39], v[248:255], v[188:191]\n\t"
      "JOIN4_%=:\n\t"
      "s_waitcnt lgkmcnt(15)\n\t"
      "s_waitcnt vmcnt(8)\n\t"
      "v_mfma_f32_16x16x128_f8f6f4 v[32:35], a[40:47], a[120:127], v[32:35]\n\t"
      "s_bfe_u32 s62, s25, 131080\n\t"
      "s_mul_i32 s62, s62, 10240\n\t"
      "s_bfe_u32 s55, s25, 196618\n\t"
      "s_lshl_b32 s55, s55, 20\n\t"
      "s_add_u32 s62, s62, s60\n\t"
      "s_add_u32 s62, s62, s55\n\t"
      "buffer_load_dword v24, v9, s[32:35], s62 offen sc0 sc1\n\t"
      "s_bfe_u32 s54, s25, 524304\n\t"
      "s_add_u32 m0, s51, 24576\n\t"
      "s_lshl_b32 s54, s54, 19\n\t"
      "s_xor_b32 s54, s54, s26\n\t"
      "s_xor_b32 s54, s54, 0x40000\n\t"
      "buffer_load_dwordx4 v12, s[28:31], s54 offen lds\n\t"
      "v_mfma_f32_16x16x128_f8f6f4 v[36:39], a[40:47], a[128:135], v[36:39]\n\t"
      "ds_read_b64_tr_b8 v[192:193], v16 offset:16384\n\t"
      "ds_read_b64_tr_b8 v[194:195], v16 offset:17408\n\t"
      "v_mfma_f32_16x16x128_f8f6f4 v[40:43], a[40:47], a[136:143], v[40:43]\n\t"
      "ds_read_b64_tr_b8 v[200:201], v17 offset:16384\n\t"
      "ds_read_b64_tr_b8 v[202:203], v17 offset:17408\n\t"
      "buffer_load_dwordx4 v13, s[28:31], s54 offen offset:1024 lds\n\t"
      "v_mfma_f32_16x16x128_f8f6f4 v[44:47], a[40:47], a[144:151], v[44:47]\n\t"
      "ds_read_b64_tr_b8 v[208:209], v18 offset:16384\n\t"
      "ds_read_b64_tr_b8 v[210:211], v18 offset:17408\n\t"
      "v_mfma_f32_16x16x128_f8f6f4 v[48:51], a[40:47], a[152:159], v[48:51]\n\t"
      "ds_read_b64_tr_b8 v[216:217], v19 offset:16384\n\t"
      "ds_read_b64_tr_b8 v[218:219], v19 offset:17408\n\t"
      "buffer_load_dwordx4 v14, s[28:31], s54 offen offset:2048 lds\n\t"
      "v_mfma_f32_16x16x128_f8f6f4 v[52:55], a[40:47], a[160:167], v[52:55]\n\t"
      "ds_read_b64_tr_b8 v[224:225], v20 offset:16384\n\t"
      "ds_read_b64_tr_b8 v[226:227], v20 offset:17408\n\t"
      "v_mfma_f32_16x16x128_f8f6f4 v[56:59], a[40:47], a[168:175], v[56:59]\n\t"
      "ds_read_b64_tr_b8 v[232:233], v21 offset:16384\n\t"
      "ds_read_b64_tr_b8 v[234:235], v21 offset:17408\n\t"
      "buffer_load_dwordx4 v15, s[28:31], s54 offen offset:3072 lds\n\t"
      "v_mfma_f32_16x16x128_f8f6f4 v[60:63], a[40:47], a[176:183], v[60:63]\n\t"
      "ds_read_b64_tr_b8 v[240:241], v22 offset:16384\n\t"
      "ds_read_b64_tr_b8 v[242:243], v22 offset:17408\n\t"
      "s_waitcnt vmcnt(11)\n\t"
      "v_mfma_f32_16x16x128_f8f6f4 v[64:67], a[48:55], a[120:127], v[64:67]\n\t"
      "ds_read_b64_tr_b8 v[248:249], v23 offset:16384\n\t"
      "ds_read_b64_tr_b8 v[250:251], v23 offset:17408\n\t"
      "v_mfma_f32_16x16x128_f8f6f4 v[68:71], a[48:55], a[128:135], v[68:71]\n\t"
      "s_add_u32 m0, s51, 28672\n\t"
      "s_add_u32 s54, s54, 0x20000\n\t"
      "buffer_load_dwordx4 v12, s[28:31], s54 offen lds\n\t"
      "v_mfma_f32_16x16x128_f8f6f4 v[72:75], a[48:55], a[136:143], v[72:75]\n\t"
      "v_mfma_f32_16x16x128_f8f6f4 v[76:79], a[48:55], a[144:151], v[76:79]\n\t"
      "buffer_load_dwordx4 v13, s[28:31], s54 offen offset:1024 lds\n\t"
      "v_mfma_f32_16x16x128_f8f6f4 v[80:83], a[48:55], a[152:159], v[80:83]\n\t"
      "v_mfma_f32_16x16x128_f8f6f4 v[84:87], a[48:55], a[160:167], v[84:87]\n\t"
      "buffer_load_dwordx4 v14, s[28:31], s54 offen offset:2048 lds\n\t"
      "v_mfma_f32_16x16x128_f8f6f4 v[88:91], a[48:55], a[168:175], v[88:91]\n\t"
      "v_mfma_f32_16x16x128_f8f6f4 v[92:95], a[48:55], a[176:183], v[92:95]\n\t"
      "buffer_load_dwordx4 v15, s[28:31], s54 offen offset:3072 lds\n\t"
      "s_waitcnt vmcnt(13)\n\t"
      "v_mfma_f32_16x16x128_f8f6f4 v[96:99], a[56:63], a[120:127], v[96:99]\n\t"
      "v_mfma_f32_16x16x128_f8f6f4 v[100:103], a[56:63], a[128:135], v[100:103]\n\t"
      "v_mfma_f32_16x16x128_f8f6f4 v[104:107], a[56:63], a[136:143], v[104:107]\n\t"
      "s_waitcnt vmcnt(8)\n\t"
      "v_cmp_le_u32 vcc, s59, v24\n\t"
      "s_mov_b32 s63, s62\n\t"
      "s_cmp_lg_u32 vcc_lo, 3\n\t"
      "s_cbranch_scc1 ALT5_%=\n\t"
      "buffer_load_dwordx4 a[80:83], v7, s[32:35], s63 offen sc0 sc1\n\t"
      "v_mfma_f32_16x16x128_f8f6f4 v[108:111], a[56:63], a[144:151], v[108:111]\n\t"
      "v_mfma_f32_16x16x128_f8f6f4 v[112:115], a[56:63], a[152:159], v[112:115]\n\t"
      "buffer_load_dwordx4 a[84:87], v3, s[32:35], s63 offen sc0 sc1\n\t"
      "v_mfma_f32_16x16x128_f8f6f4 v[116:119], a[56:63], a[160:167], v[116:119]\n\t"
      "v_mfma_f32_16x16x128_f8f6f4 v[120:123], a[56:63], a[168:175], v[120:123]\n\t"
      "buffer_load_dwordx4 a[88:91], v7, s[32:35], s63 offen offset:2048 sc0 sc1\n\t"
      "v_mfma_f32_16x16x128_f8f6f4 v[124:127], a[56:63], a[176:183], v[124:127]\n\t"
      "v_mfma_f32_16x16x128_f8f6f4 v[128:131], a[64:71], a[120:127], v[128:131]\n\t"
      "v_mfma_f32_16x16x128_f8f6f4 v[132:135], a[64:71], a[128:135], v[132:135]\n\t"
      "buffer_load_dwordx4 a[92:95], v3, s[32:35], s63 offen offset:2048 sc0 sc1\n\t"
      "v_mfma_f32_16x16x128_f8f6f4 v[136:139], a[64:71], a[136:143], v[136:139]\n\t"
      "v_mfma_f32_16x16x128_f8f6f4 v[140:143], a[64:71], a[144:151], v[140:143]\n\t"
      "s_add_u32 s63, s63, 0x1000\n\t"
      "buffer_load_dwordx4 a[96:99], v7, s[32:35], s63 offen sc0 sc1\n\t"
      "v_mfma_f32_16x16x128_f8f6f4 v[144:147], a[64:71], a[152:159], v[144:147]\n\t"
      "v_mfma_f32_16x16x128_f8f6f4 v[148:151], a[64:71], a[160:167], v[148:151]\n\t"
      "buffer_load_dwordx4 a[100:103], v3, s[32:35], s63 offen sc0 sc1\n\t"
      "v_mfma_f32_16x16x128_f8f6f4 v[152:155], a[64:71], a[168:175], v[152:155]\n\t"
      "v_mfma_f32_16x16x128_f8f6f4 v[156:159], a[64:71], a[176:183], v[156:159]\n\t"
      "buffer_load_dwordx4 a[104:107], v7, s[32:35], s63 offen offset:2048 sc0 sc1\n\t"
      "v_mfma_f32_16x16x128_f8f6f4 v[160:163], a[72:79], a[120:127], v[160:163]\n\t"
      "v_mfma_f32_16x16x128_f8f6f4 v[164:167], a[72:79], a[128:135], v[164:167]\n\t"
      "v_mfma_f32_16x16x128_f8f6f4 v[168:171], a[72:79], a[136:143], v[168:171]\n\t"
      "buffer_load_dwordx4 a[108:111], v3, s[32:35], s63 offen offset:2048 sc0 sc1\n\t"
      "v_mfma_f32_16x16x128_f8f6f4 v[172:175], a[72:79], a[144:151], v[172:175]\n\t"
      "v_mfma_f32_16x16x128_f8f6f4 v[176:179], a[72:79], a[152:159], v[176:179]\n\t"
      "s_add_u32 s63, s63, 0x1000\n\t"
      "buffer_load_dwordx4 a[112:115], v8, s[32:35], s63 offen sc0 sc1\n\t"
      "v_mfma_f32_16x16x128_f8f6f4 v[180:183], a[72:79], a[160:167], v[180:183]\n\t"
      "buffer_load_dwordx4 a[116:119], v4, s[32:35], s63 offen sc0 sc1\n\t"
      "v_mfma_f32_16x16x128_f8f6f4 v[184:187], a[72:79], a[168:175], v[184:187]\n\t"
      "v_mfma_f32_16x16x128_f8f6f4 v[188:191], a[72:79], a[176:183], v[188:191]\n\t"
      "JOIN5_%=:\n\t"
      "s_waitcnt lgkmcnt(15)\n\t"
      "s_waitcnt vmcnt(8)\n\t"
      "v_mfma_f32_16x16x128_f8f6f4 v[32:35], a[80:87], a[184:191], v[32:35]\n\t"
      "s_bfe_u32 s62, s25, 131088\n\t"
      "s_mul_i32 s62, s62, 10240\n\t"
      "s_bfe_u32 s55, s25, 196626\n\t"
      "s_lshl_b32 s55, s55, 20\n\t"
      "s_add_u32 s62, s62, s60\n\t"
      "s_add_u32 s62, s62, s55\n\t"
      "buffer_load_dword v24, v9, s[32:35], s62 offen sc0 sc1\n\t"
      "s_bfe_u32 s54, s25, 524312\n\t"
      "s_add_u32 m0, s51, 32768\n\t"
      "s_lshl_b32 s54, s54, 19\n\t"
      "s_xor_b32 s54, s54, s26\n\t"
      "buffer_load_dwordx4 v12, s[28:31], s54 offen lds\n\t"
      "v_mfma_f32_16x16x128_f8f6f4 v[36:39], a[80:87], a[192:199], v[36:39]\n\t"
      "ds_read_b64_tr_b8 v[196:197], v16 offset:24576\n\t"
      "ds_read_b64_tr_b8 v[198:199], v16 offset:25600\n\t"
      "v_mfma_f32_16x16x128_f8f6f4 v[40:43], a[80:87], a[200:207], v[40:43]\n\t"
      "ds_read_b64_tr_b8 v[204:205], v17 offset:24576\n\t"
      "ds_read_b64_tr_b8 v[206:207], v17 offset:25600\n\t"
      "buffer_load_dwordx4 v13, s[28:31], s54 offen offset:1024 lds\n\t"
      "v_mfma_f32_16x16x128_f8f6f4 v[44:47], a[80:87], a[208:215], v[44:47]\n\t"
      "ds_read_b64_tr_b8 v[212:213], v18 offset:24576\n\t"
      "ds_read_b64_tr_b8 v[214:215], v18 offset:25600\n\t"
      "v_mfma_f32_16x16x128_f8f6f4 v[48:51], a[80:87], a[216:223], v[48:51]\n\t"
      "ds_read_b64_tr_b8 v[220:221], v19 offset:24576\n\t"
      "ds_read_b64_tr_b8 v[222:223], v19 offset:25600\n\t"
      "buffer_load_dwordx4 v14, s[28:31], s54 offen offset:2048 lds\n\t"
      "v_mfma_f32_16x16x128_f8f6f4 v[52:55], a[80:87], a[224:231], v[52:55]\n\t"
      "ds_read_b64_tr_b8 v[228:229], v20 offset:24576\n\t"
      "ds_read_b64_tr_b8 v[230:231], v20 offset:25600\n\t"
      "v_mfma_f32_16x16x128_f8f6f4 v[56:59], a[80:87], a[232:239], v[56:59]\n\t"
      "ds_read_b64_tr_b8 v[236:237], v21 offset:24576\n\t"
      "ds_read_b64_tr_b8 v[238:239], v21 offset:25600\n\t"
      "buffer_load_dwordx4 v15, s[28:31], s54 offen offset:3072 lds\n\t"
      "v_mfma_f32_16x16x128_f8f6f4 v[60:63], a[80:87], a[240:247], v[60:63]\n\t"
      "ds_read_b64_tr_b8 v[244:245], v22 offset:24576\n\t"
      "ds_read_b64_tr_b8 v[246:247], v22 offset:25600\n\t"
      "s_waitcnt vmcnt(11)\n\t"
      "v_mfma_f32_16x16x128_f8f6f4 v[64:67], a[88:95], a[184:191], v[64:67]\n\t"
      "ds_read_b64_tr_b8 v[252:253], v23 offset:24576\n\t"
      "ds_read_b64_tr_b8 v[254:255], v23 offset:25600\n\t"
      "v_mfma_f32_16x16x128_f8f6f4 v[68:71], a[88:95], a[192:199], v[68:71]\n\t"
      "s_add_u32 m0, s51, 36864\n\t"
      "s_add_u32 s54, s54, 0x20000\n\t"
      "buffer_load_dwordx4 v12, s[28:31], s54 offen lds\n\t"
      "v_mfma_f32_16x16x128_f8f6f4 v[72:75], a[88:95], a[200:207], v[72:75]\n\t"
      "v_mfma_f32_16x16x128_f8f6f4 v[76:79], a[88:95], a[208:215], v[76:79]\n\t"
      "buffer_load_dwordx4 v13, s[28:31], s54 offen offset:1024 lds\n\t"
      "v_mfma_f32_16x16x128_f8f6f4 v[80:83], a[88:95], a[216:223], v[80:83]\n\t"
      "v_mfma_f32_16x16x128_f8f6f4 v[84:87], a[88:95], a[224:231], v[84:87]\n\t"
      "buffer_load_dwordx4 v14, s[28:31], s54 offen offset:2048 lds\n\t"
      "v_mfma_f32_16x16x128_f8f6f4 v[88:91], a[88:95], a[232:239], v[88:91]\n\t"
      "v_mfma_f32_16x16x128_f8f6f4 v[92:95], a[88:95], a[240:247], v[92:95]\n\t"
      "buffer_load_dwordx4 v15, s[28:31], s54 offen offset:3072 lds\n\t"
      "s_waitcnt vmcnt(13)\n\t"
      "v_mfma_f32_16x16x128_f8f6f4 v[96:99], a[96:103], a[184:191], v[96:99]\n\t"
      "v_mfma_f32_16x16x128_f8f6f4 v[100:103], a[96:103], a[192:199], v[100:103]\n\t"
      "v_mfma_f32_16x16x128_f8f6f4 v[104:107], a[96:103], a[200:207], v[104:107]\n\t"
      "s_waitcnt vmcnt(8)\n\t"
      "v_cmp_le_u32 vcc, s59, v24\n\t"
      "s_mov_b32 s63, s62\n\t"
      "s_cmp_lg_u32 vcc_lo, 3\n\t"
      "s_cbranch_scc1 ALT6_%=\n\t"
      "buffer_load_dwordx4 a[0:3], v7, s[32:35], s63 offen sc0 sc1\n\t"
      "v_mfma_f32_16x16x128_f8f6f4 v[108:111], a[96:103], a[208:215], v[108:111]\n\t"
      "v_mfma_f32_16x16x128_f8f6f4 v[112:115], a[96:103], a[216:223], v[112:115]\n\t"
      "buffer_load_dwordx4 a[4:7], v3, s[32:35], s63 offen sc0 sc1\n\t"
      "v_mfma_f32_16x16x128_f8f6f4 v[116:119], a[96:103], a[224:231], v[116:119]\n\t"
      "v_mfma_f32_16x16x128_f8f6f4 v[120:123], a[96:103], a[232:239], v[120:123]\n\t"
      "buffer_load_dwordx4 a[8:11], v7, s[32:35], s63 offen offset:2048 sc0 sc1\n\t"
      "v_mfma_f32_16x16x128_f8f6f4 v[124:127], a[96:103], a[240:247], v[124:127]\n\t"
      "v_mfma_f32_16x16x128_f8f6f4 v[128:131], a[104:111], a[184:191], v[128:131]\n\t"
      "v_mfma_f32_16x16x128_f8f6f4 v[132:135], a[104:111], a[192:199], v[132:135]\n\t"
      "buffer_load_dwordx4 a[12:15], v3, s[32:35], s63 offen offset:2048 sc0 sc1\n\t"
      "v_mfma_f32_16x16x128_f8f6f4 v[136:139], a[104:111], a[200:207], v[136:139]\n\t"
      "v_mfma_f32_16x16x128_f8f6f4 v[140:143], a[104:111], a[208:215], v[140:143]\n\t"
      "s_add_u32 s63, s63, 0x1000\n\t"
      "buffer_load_dwordx4 a[16:19], v7, s[32:35], s63 offen sc0 sc1\n\t"
      "v_mfma_f32_16x16x128_f8f6f4 v[144:147], a[104:111], a[216:223], v[144:147]\n\t"
      "v_mfma_f32_16x16x128_f8f6f4 v[148:151], a[104:111], a[224:231], v[148:151]\n\t"
      "buffer_load_dwordx4 a[20:23], v3, s[32:35], s63 offen sc0 sc1\n\t"
      "v_mfma_f32_16x16x128_f8f6f4 v[152:155], a[104:111], a[232:239], v[152:155]\n\t"
      "v_mfma_f32_16x16x128_f8f6f4 v[156:159], a[104:111], a[240:247], v[156:159]\n\t"
      "buffer_load_dwordx4 a[24:27], v7, s[32:35], s63 offen offset:2048 sc0 sc1\n\t"
      "v_mfma_f32_16x16x128_f8f6f4 v[160:163], a[112:119], a[184:191], v[160:163]\n\t"
      "v_mfma_f32_16x16x128_f8f6f4 v[164:167], a[112:119], a[192:199], v[164:167]\n\t"
      "v_mfma_f32_16x16x128_f8f6f4 v[168:171], a[112:119], a[200:207], v[168:171]\n\t"
      "buffer_load_dwordx4 a[28:31], v3, s[32:35], s63 offen offset:2048 sc0 sc1\n\t"
      "v_mfma_f32_16x16x128_f8f6f4 v[172:175], a[112:119], a[208:215], v[172:175]\n\t"
      "v_mfma_f32_16x16x128_f8f6f4 v[176:179], a[112:119], a[216:223], v[176:179]\n\t"
      "s_add_u32 s63, s63, 0x1000\n\t"
      "buffer_load_dwordx4 a[32:35], v8, s[32:35], s63 offen sc0 sc1\n\t"
      "v_mfma_f32_16x16x128_f8f6f4 v[180:183], a[112:119], a[224:231], v[180:183]\n\t"
      "buffer_load_dwordx4 a[36:39], v4, s[32:35], s63 offen sc0 sc1\n\t"
      "v_mfma_f32_16x16x128_f8f6f4 v[184:187], a[112:119], a[232:239], v[184:187]\n\t"
      "v_mfma_f32_16x16x128_f8f6f4 v[188:191], a[112:119], a[240:247], v[188:191]\n\t"
      "JOIN6_%=:\n\t"
      "s_waitcnt lgkmcnt(15)\n\t"
      "s_waitcnt lgkmcnt(0)\n\t"
      "s_waitcnt vmcnt(8)\n\t"
      "v_mfma_f32_16x16x128_f8f6f4 v[32:35], a[0:7], v[192:199], v[32:35]\n\t"
      "s_bfe_u32 s62, s25, 131096\n\t"
      "s_mul_i32 s62, s62, 10240\n\t"
      "s_bfe_u32 s55, s25, 196634\n\t"
      "s_lshl_b32 s55, s55, 20\n\t"
      "s_add_u32 s62, s62, s60\n\t"
      "s_add_u32 s62, s62, s55\n\t"
      "buffer_load_dword v24, v9, s[32:35], s62 offen sc0 sc1\n\t"
      "s_bfe_u32 s54, s25, 524312\n\t"
      "s_add_u32 m0, s51, 0\n\t"
      "s_lshl_b32 s54, s54, 19\n\t"
      "s_xor_b32 s54, s54, s26\n\t"
      "s_xor_b32 s54, s54, 0x40000\n\t"
      "buffer_load_dwordx4 v12, s[28:31], s54 offen lds\n\t"
      "v_mfma_f32_16x16x128_f8f6f4 v[36:39], a[0:7], v[200:207], v[36:39]\n\t"
      "ds_read_b64_tr_b8 a[120:121], v16 offset:32768\n\t"
      "ds_read_b64_tr_b8 a[122:123], v16 offset:33792\n\t"
      "v_mfma_f32_16x16x128_f8f6f4 v[40:43], a[0:7], v[208:215], v[40:43]\n\t"
      "ds_read_b64_tr_b8 a[128:129], v17 offset:32768\n\t"
      "ds_read_b64_tr_b8 a[130:131], v17 offset:33792\n\t"
      "buffer_load_dwordx4 v13, s[28:31], s54 offen offset:1024 lds\n\t"
      "v_mfma_f32_16x16x128_f8f6f4 v[44:47], a[0:7], v[216:223], v[44:47]\n\t"
      "ds_read_b64_tr_b8 a[136:137], v18 offset:32768\n\t"
      "ds_read_b64_tr_b8 a[138:139], v18 offset:33792\n\t"
      "v_mfma_f32_16x16x128_f8f6f4 v[48:51], a[0:7], v[224:231], v[48:51]\n\t"
      "ds_read_b64_tr_b8 a[144:145], v19 offset:32768\n\t"
      "ds_read_b64_tr_b8 a[146:147], v19 offset:33792\n\t"
      "buffer_load_dwordx4 v14, s[28:31], s54 offen offset:2048 lds\n\t"
      "v_mfma_f32_16x16x128_f8f6f4 v[52:55], a[0:7], v[232:239], v[52:55]\n\t"
      "ds_read_b64_tr_b8 a[152:153], v20 offset:32768\n\t"
      "ds_read_b64_tr_b8 a[154:155], v20 offset:33792\n\t"
      "v_mfma_f32_16x16x128_f8f6f4 v[56:59], a[0:7], v[240:247], v[56:59]\n\t"
      "ds_read_b64_tr_b8 a[160:161], v21 offset:32768\n\t"
      "ds_read_b64_tr_b8 a[162:163], v21 offset:33792\n\t"
      "buffer_load_dwordx4 v15, s[28:31], s54 offen offset:3072 lds\n\t"
      "v_mfma_f32_16x16x128_f8f6f4 v[60:63], a[0:7], v[248:255], v[60:63]\n\t"
      "ds_read_b64_tr_b8 a[168:169], v22 offset:32768\n\t"
      "ds_read_b64_tr_b8 a[170:171], v22 offset:33792\n\t"
      "s_waitcnt vmcnt(11)\n\t"
      "v_mfma_f32_16x16x128_f8f6f4 v[64:67], a[8:15], v[192:199], v[64:67]\n\t"
      "ds_read_b64_tr_b8 a[176:177], v23 offset:32768\n\t"
      "ds_read_b64_tr_b8 a[178:179], v23 offset:33792\n\t"
      "v_mfma_f32_16x16x128_f8f6f4 v[68:71], a[8:15], v[200:207], v[68:71]\n\t"
      "s_add_u32 m0, s51, 4096\n\t"
      "s_add_u32 s54, s54, 0x20000\n\t"
      "buffer_load_dwordx4 v12, s[28:31], s54 offen lds\n\t"
      "v_mfma_f32_16x16x128_f8f6f4 v[72:75], a[8:15], v[208:215], v[72:75]\n\t"
      "v_mfma_f32_16x16x128_f8f6f4 v[76:79], a[8:15], v[216:223], v[76:79]\n\t"
      "buffer_load_dwordx4 v13, s[28:31], s54 offen offset:1024 lds\n\t"
      "v_mfma_f32_16x16x128_f8f6f4 v[80:83], a[8:15], v[224:231], v[80:83]\n\t"
      "v_mfma_f32_16x16x128_f8f6f4 v[84:87], a[8:15], v[232:239], v[84:87]\n\t"
      "buffer_load_dwordx4 v14, s[28:31], s54 offen offset:2048 lds\n\t"
      "v_mfma_f32_16x16x128_f8f6f4 v[88:91], a[8:15], v[240:247], v[88:91]\n\t"
      "v_mfma_f32_16x16x128_f8f6f4 v[92:95], a[8:15], v[248:255], v[92:95]\n\t"
      "buffer_load_dwordx4 v15, s[28:31], s54 offen offset:3072 lds\n\t"
      "s_waitcnt vmcnt(13)\n\t"
      "v_mfma_f32_16x16x128_f8f6f4 v[96:99], a[16:23], v[192:199], v[96:99]\n\t"
      "v_mfma_f32_16x16x128_f8f6f4 v[100:103], a[16:23], v[200:207], v[100:103]\n\t"
      "v_mfma_f32_16x16x128_f8f6f4 v[104:107], a[16:23], v[208:215], v[104:107]\n\t"
      "s_waitcnt vmcnt(8)\n\t"
      "v_cmp_le_u32 vcc, s59, v24\n\t"
      "s_mov_b32 s63, s62\n\t"
      "s_cmp_lg_u32 vcc_lo, 3\n\t"
      "s_cbranch_scc1 ALT7_%=\n\t"
      "buffer_load_dword v29, off, s[44:47], 0 offset:0 sc1\n\t"
      "buffer_load_dwordx4 a[40:43], v7, s[32:35], s63 offen sc0 sc1\n\t"
      "v_mfma_f32_16x16x128_f8f6f4 v[108:111], a[16:23], v[216:223], v[108:111]\n\t"
      "v_mfma_f32_16x16x128_f8f6f4 v[112:115], a[16:23], v[224:231], v[112:115]\n\t"
      "buffer_load_dwordx4 a[44:47], v3, s[32:35], s63 offen sc0 sc1\n\t"
      "v_mfma_f32_16x16x128_f8f6f4 v[116:119], a[16:23], v[232:239], v[116:119]\n\t"
      "v_mfma_f32_16x16x128_f8f6f4 v[120:123], a[16:23], v[240:247], v[120:123]\n\t"
      "buffer_load_dwordx4 a[48:51], v7, s[32:35], s63 offen offset:2048 sc0 sc1\n\t"
      "v_mfma_f32_16x16x128_f8f6f4 v[124:127], a[16:23], v[248:255], v[124:127]\n\t"
      "v_mfma_f32_16x16x128_f8f6f4 v[128:131], a[24:31], v[192:199], v[128:131]\n\t"
      "v_mfma_f32_16x16x128_f8f6f4 v[132:135], a[24:31], v[200:207], v[132:135]\n\t"
      "buffer_load_dwordx4 a[52:55], v3, s[32:35], s63 offen offset:2048 sc0 sc1\n\t"
      "v_mfma_f32_16x16x128_f8f6f4 v[136:139], a[24:31], v[208:215], v[136:139]\n\t"
      "v_mfma_f32_16x16x128_f8f6f4 v[140:143], a[24:31], v[216:223], v[140:143]\n\t"
      "s_add_u32 s63, s63, 0x1000\n\t"
      "buffer_load_dwordx4 a[56:59], v7, s[32:35], s63 offen sc0 sc1\n\t"
      "v_mfma_f32_16x16x128_f8f6f4 v[144:147], a[24:31], v[224:231], v[144:147]\n\t"
      "v_mfma_f32_16x16x128_f8f6f4 v[148:151], a[24:31], v[232:239], v[148:151]\n\t"
      "buffer_load_dwordx4 a[60:63], v3, s[32:35], s63 offen sc0 sc1\n\t"
      "v_mfma_f32_16x16x128_f8f6f4 v[152:155], a[24:31], v[240:247], v[152:155]\n\t"
      "s_waitcnt vmcnt(7)\n\t"
      "ds_read_b64_tr_b8 a[124:125], v16 offset:0\n\t"
      "ds_read_b64_tr_b8 a[126:127], v16 offset:1024\n\t"
      "v_mfma_f32_16x16x128_f8f6f4 v[156:159], a[24:31], v[248:255], v[156:159]\n\t"
      "ds_read_b64_tr_b8 a[132:133], v17 offset:0\n\t"
      "ds_read_b64_tr_b8 a[134:135], v17 offset:1024\n\t"
      "buffer_load_dwordx4 a[64:67], v7, s[32:35], s63 offen offset:2048 sc0 sc1\n\t"
      "v_mfma_f32_16x16x128_f8f6f4 v[160:163], a[32:39], v[192:199], v[160:163]\n\t"
      "ds_read_b64_tr_b8 a[140:141], v18 offset:0\n\t"
      "ds_read_b64_tr_b8 a[142:143], v18 offset:1024\n\t"
      "v_mfma_f32_16x16x128_f8f6f4 v[164:167], a[32:39], v[200:207], v[164:167]\n\t"
      "ds_read_b64_tr_b8 a[148:149], v19 offset:0\n\t"
      "ds_read_b64_tr_b8 a[150:151], v19 offset:1024\n\t"
      "v_mfma_f32_16x16x128_f8f6f4 v[168:171], a[32:39], v[208:215], v[168:171]\n\t"
      "ds_read_b64_tr_b8 a[156:157], v20 offset:0\n\t"
      "ds_read_b64_tr_b8 a[158:159], v20 offset:1024\n\t"
      "buffer_load_dwordx4 a[68:71], v3, s[32:35], s63 offen offset:2048 sc0 sc1\n\t"
      "v_mfma_f32_16x16x128_f8f6f4 v[172:175], a[32:39], v[216:223], v[172:175]\n\t"
      "ds_read_b64_tr_b8 a[164:165], v21 offset:0\n\t"
      "ds_read_b64_tr_b8 a[166:167], v21 offset:1024\n\t"
      "v_mfma_f32_16x16x128_f8f6f4 v[176:179], a[32:39], v[224:231], v[176:179]\n\t"
      "ds_read_b64_tr_b8 a[172:173], v22 offset:0\n\t"
      "ds_read_b64_tr_b8 a[174:175], v22 offset:1024\n\t"
      "s_add_u32 s63, s63, 0x1000\n\t"
      "buffer_load_dwordx4 a[72:75], v8, s[32:35], s63 offen sc0 sc1\n\t"
      "v_mfma_f32_16x16x128_f8f6f4 v[180:183], a[32:39], v[232:239], v[180:183]\n\t"
      "ds_read_b64_tr_b8 a[180:181], v23 offset:0\n\t"
      "ds_read_b64_tr_b8 a[182:183], v23 offset:1024\n\t"
      "buffer_load_dwordx4 a[76:79], v4, s[32:35], s63 offen sc0 sc1\n\t"
      "v_mfma_f32_16x16x128_f8f6f4 v[184:187], a[32:39], v[240:247], v[184:187]\n\t"
      "v_mfma_f32_16x16x128_f8f6f4 v[188:191], a[32:39], v[248:255], v[188:191]\n\t"
      "JOIN7_%=:\n\t"
      "s_cmp_eq_u32 s61, 0\n\t"
      "s_cbranch_scc1 NOPOIS_%=\n\t"
      "s_waitcnt lgkmcnt(0)\n\t"
      "v_accvgpr_write_b32 a120, -1\n\t"
      "v_accvgpr_write_b32 a128, -1\n\t"
      "v_accvgpr_write_b32 a136, -1\n\t"
      "v_accvgpr_write_b32 a144, -1\n\t"
      "v_accvgpr_write_b32 a152, -1\n\t"
      "v_accvgpr_write_b32 a160, -1\n\t"
      "v_accvgpr_write_b32 a168, -1\n\t"
      "v_accvgpr_write_b32 a176, -1\n\t"
      "s_nop 1\n\t"
      "NOPOIS_%=:\n\t"
      "s_waitcnt lgkmcnt(0)\n\t"
      "s_mov_b32 m0, s51\n\t"
      "s_waitcnt vmcnt(8)\n\t"
      "v_mfma_f32_16x16x128_f8f6f4 v[32:35], a[40:47], a[120:127], v[32:35]\n\t"
      "v_and_b32 v11, 3, v2\n\t"
      "v_bfe_u32 v26, v2, 3, 2\n\t"
      "v_bfe_u32 v27, v2, 2, 1\n\t"
      "v_mfma_f32_16x16x128_f8f6f4 v[36:39], a[40:47], a[128:135], v[36:39]\n\t"
      "v_lshrrev_b32 v24, 5, v2\n\t"
      "v_lshl_or_b32 v27, v24, 1, v27\n\t"
      "v_lshlrev_b32 v11, 4, v11\n\t"
      "v_mfma_f32_16x16x128_f8f6f4 v[40:43], a[40:47], a[136:143], v[40:43]\n\t"
      "v_lshl_or_b32 v16, v26, 6, v11\n\t"
      "v_lshl_or_b32 v16, v27, 10, v16\n\t"
      "s_lshl_b32 s55, s48, 8\n\t"
      "v_mfma_f32_16x16x128_f8f6f4 v[44:47], a[40:47], a[144:151], v[44:47]\n\t"
      "s_nop 0\n\t"
      "ds_write_addtid_b32 v32 offset:0\n\t"
      "ds_write_addtid_b32 v33 offset:256\n\t"
      "ds_write_addtid_b32 v34 offset:512\n\t"
      "ds_write_addtid_b32 v35 offset:768\n\t"
      "v_add_u32 v16, s55, v16\n\t"
      "v_add_u32 v17, 40960, v16\n\t"
      "v_add_u32 v18, 81920, v16\n\t"
      "v_mfma_f32_16x16x128_f8f6f4 v[48:51], a[40:47], a[152:159], v[48:51]\n\t"
      "ds_write_addtid_b32 v36 offset:1024\n\t"
      "ds_write_addtid_b32 v37 offset:1280\n\t"
      "ds_write_addtid_b32 v38 offset:1536\n\t"
      "ds_write_addtid_b32 v39 offset:1792\n\t"
      "v_add_u32 v19, 122880, v16\n\t"
      "v_lshl_or_b32 v10, v27, 6, v11\n\t"
      "v_mfma_f32_16x16x128_f8f6f4 v[52:55], a[40:47], a[160:167], v[52:55]\n\t"
      "ds_write_addtid_b32 v40 offset:2048\n\t"
      "ds_write_addtid_b32 v41 offset:2304\n\t"
      "ds_write_addtid_b32 v42 offset:2560\n\t"
      "ds_write_addtid_b32 v43 offset:2816\n\t"
      "v_lshl_add_u32 v26, v26, 2, s48\n\t"
      "v_lshl_or_b32 v10, v26, 14, v10\n\t"
      "s_cmp_eq_u32 s49, 6\n\t"
      "v_mfma_f32_16x16x128_f8f6f4 v[56:59], a[40:47], a[168:175], v[56:59]\n\t"
      "ds_write_addtid_b32 v44 offset:3072\n\t"
      "ds_write_addtid_b32 v45 offset:3328\n\t"
      "ds_write_addtid_b32 v46 offset:3584\n\t"
      "ds_write_addtid_b32 v47 offset:3840\n\t"
      "s_cselect_b32 s55, 10, 9\n\t"
      "v_cmp_gt_u32 vcc, s55, v26\n\t"
      "v_bfrev_b32 v20, 1\n\t"
      "v_mfma_f32_16x16x128_f8f6f4 v[60:63], a[40:47], a[176:183], v[60:63]\n\t"
      "ds_write_addtid_b32 v48 offset:4096\n\t"
      "ds_write_addtid_b32 v49 offset:4352\n\t"
      "ds_write_addtid_b32 v50 offset:4608\n\t"
      "ds_write_addtid_b32 v51 offset:4864\n\t"
      "v_mov_b32 v28, s64\n\t"
      "v_cndmask_b32 v20, v20, v10, vcc\n\t"
      "v_mul_f32 v28, s65, v28\n\t"
      "s_waitcnt vmcnt(6)\n\t"
      "v_mfma_f32_16x16x128_f8f6f4 v[64:67], a[48:55], a[120:127], v[64:67]\n\t"
      "ds_write_addtid_b32 v52 offset:5120\n\t"
      "ds_write_addtid_b32 v53 offset:5376\n\t"
      "ds_write_addtid_b32 v54 offset:5632\n\t"
      "ds_write_addtid_b32 v55 offset:5888\n\t"
      "v_cmp_ne_u32 vcc, s59, v29\n\t"
      "v_mov_b32 v25, s59\n\t"
      "v_lshlrev_b32 v24, 20, %2\n\t"
      "v_mfma_f32_16x16x128_f8f6f4 v[68:71], a[48:55], a[128:135], v[68:71]\n\t"
      "ds_write_addtid_b32 v56 offset:6144\n\t"
      "ds_write_addtid_b32 v57 offset:6400\n\t"
      "ds_write_addtid_b32 v58 offset:6656\n\t"
      "ds_write_addtid_b32 v59 offset:6912\n\t"
      "v_cndmask_b32 v28, -1, v28, vcc\n\t"
      "s_mov_b32 s54, 0\n\t"
      "v_mfma_f32_16x16x128_f8f6f4 v[72:75], a[48:55], a[136:143], v[72:75]\n\t"
      "ds_write_addtid_b32 v60 offset:7168\n\t"
      "ds_write_addtid_b32 v61 offset:7424\n\t"
      "ds_write_addtid_b32 v62 offset:7680\n\t"
      "ds_write_addtid_b32 v63 offset:7936\n\t"
      "v_mfma_f32_16x16x128_f8f6f4 v[76:79], a[48:55], a[144:151], v[76:79]\n\t"
      "ds_write_addtid_b32 v64 offset:8192\n\t"
      "ds_write_addtid_b32 v65 offset:8448\n\t"
      "ds_write_addtid_b32 v66 offset:8704\n\t"
      "ds_write_addtid_b32 v67 offset:8960\n\t"
      "v_mfma_f32_16x16x128_f8f6f4 v[80:83], a[48:55], a[152:159], v[80:83]\n\t"
      "s_waitcnt lgkmcnt(4)\n\t"
      "s_barrier\n\t"
      "buffer_store_dword v25, v24, s[44:47], s53 offen sc1\n\t"
      "ds_write_addtid_b32 v68 offset:9216\n\t"
      "ds_write_addtid_b32 v69 offset:9472\n\t"
      "ds_write_addtid_b32 v70 offset:9728\n\t"
      "ds_write_addtid_b32 v71 offset:9984\n\t"
      "ds_read_b128 v[192:195], v16 offset:0\n\t"
      "v_mfma_f32_16x16x128_f8f6f4 v[84:87], a[48:55], a[160:167], v[84:87]\n\t"
      "ds_write_addtid_b32 v72 offset:10240\n\t"
      "ds_write_addtid_b32 v73 offset:10496\n\t"
      "ds_write_addtid_b32 v74 offset:10752\n\t"
      "ds_write_addtid_b32 v75 offset:11008\n\t"
      "ds_read_b128 v[196:199], v17 offset:0\n\t"
      "v_mfma_f32_16x16x128_f8f6f4 v[88:91], a[48:55], a[168:175], v[88:91]\n\t"
      "ds_write_addtid_b32 v76 offset:11264\n\t"
      "ds_write_addtid_b32 v77 offset:11520\n\t"
      "ds_write_addtid_b32 v78 offset:11776\n\t"
      "ds_write_addtid_b32 v79 offset:12032\n\t"
      "ds_read_b128 v[200:203], v18 offset:0\n\t"
      "v_mfma_f32_16x16x128_f8f6f4 v[92:95], a[48:55], a[176:183], v[92:95]\n\t"
      "ds_write_addtid_b32 v80 offset:12288\n\t"
      "ds_write_addtid_b32 v81 offset:12544\n\t"
      "ds_write_addtid_b32 v82 offset:12800\n\t"
      "ds_write_addtid_b32 v83 offset:13056\n\t"
      "ds_read_b128 v[204:207], v19 offset:0\n\t"
      "s_waitcnt vmcnt(5)\n\t"
      "v_mfma_f32_16x16x128_f8f6f4 v[96:99], a[56:63], a[120:127], v[96:99]\n\t"
      "ds_write_addtid_b32 v84 offset:13312\n\t"
      "ds_write_addtid_b32 v85 offset:13568\n\t"
      "ds_write_addtid_b32 v86 offset:13824\n\t"
      "ds_write_addtid_b32 v87 offset:14080\n\t"
      "ds_read_b128 v[208:211], v16 offset:4096\n\t"
      "v_mfma_f32_16x16x128_f8f6f4 v[100:103], a[56:63], a[128:135], v[100:103]\n\t"
      "ds_write_addtid_b32 v88 offset:14336\n\t"
      "ds_write_addtid_b32 v89 offset:14592\n\t"
      "ds_write_addtid_b32 v90 offset:14848\n\t"
      "ds_write_addtid_b32 v91 offset:15104\n\t"
      "ds_read_b128 v[212:215], v17 offset:4096\n\t"
      "s_waitcnt lgkmcnt(10)\n\t"
      "v_add_f32 v192, v192, v196\n\t"
      "v_add_f32 v193, v193, v197\n\t"
      "v_add_f32 v194, v194, v198\n\t"
      "v_add_f32 v195, v195, v199\n\t"
      "v_add_f32 v200, v200, v204\n\t"
      "v_add_f32 v201, v201, v205\n\t"
      "v_mfma_f32_16x16x128_f8f6f4 v[104:107], a[56:63], a[136:143], v[104:107]\n\t"
      "ds_write_addtid_b32 v92 offset:15360\n\t"
      "ds_write_addtid_b32 v93 offset:15616\n\t"
      "ds_write_addtid_b32 v94 offset:15872\n\t"
      "ds_write_addtid_b32 v95 offset:16128\n\t"
      "ds_read_b128 v[216:219], v18 offset:4096\n\t"
      "v_add_f32 v202, v202, v206\n\t"
      "v_add_f32 v203, v203, v207\n\t"
      "v_add_f32 v192, v192, v200\n\t"
      "v_add_f32 v193, v193, v201\n\t"
      "v_add_f32 v194, v194, v202\n\t"
      "v_add_f32 v195, v195, v203\n\t"
      "v_mfma_f32_16x16x128_f8f6f4 v[108:111], a[56:63], a[144:151], v[108:111]\n\t"
      "ds_write_addtid_b32 v96 offset:16384\n\t"
      "ds_write_addtid_b32 v97 offset:16640\n\t"
      "ds_write_addtid_b32 v98 offset:16896\n\t"
      "ds_write_addtid_b32 v99 offset:17152\n\t"
      "ds_read_b128 v[220:223], v19 offset:4096\n\t"
      "v_mul_f32 v192, v192, v28\n\t"
      "v_mul_f32 v193, v193, v28\n\t"
      "v_mul_f32 v194, v194, v28\n\t"
      "v_mul_f32 v195, v195, v28\n\t"
      "v_maximum3_f32 v192, v192, 0, 0\n\t"
      "v_maximum3_f32 v193, v193, 0, 0\n\t"
      "v_mfma_f32_16x16x128_f8f6f4 v[112:115], a[56:63], a[152:159], v[112:115]\n\t"
      "s_waitcnt lgkmcnt(6)\n\t"
      "s_barrier\n\t"
      "ds_write_addtid_b32 v100 offset:17408\n\t"
      "ds_write_addtid_b32 v101 offset:17664\n\t"
      "ds_write_addtid_b32 v102 offset:17920\n\t"
      "ds_write_addtid_b32 v103 offset:18176\n\t"
      "ds_read_b128 v[224:227], v16 offset:8192\n\t"
      "v_maximum3_f32 v194, v194, 0, 0\n\t"
      "v_maximum3_f32 v195, v195, 0, 0\n\t"
      "buffer_store_dwordx4 v[192:195], v10, s[40:43], s54 offen sc1\n\t"
      "v_mfma_f32_16x16x128_f8f6f4 v[116:119], a[56:63], a[160:167], v[116:119]\n\t"
      "ds_write_addtid_b32 v104 offset:18432\n\t"
      "ds_write_addtid_b32 v105 offset:18688\n\t"
      "ds_write_addtid_b32 v106 offset:18944\n\t"
      "ds_write_addtid_b32 v107 offset:19200\n\t"
      "ds_read_b128 v[228:231], v17 offset:8192\n\t"
      "s_waitcnt lgkmcnt(10)\n\t"
      "v_add_f32 v208, v208, v212\n\t"
      "v_add_f32 v209, v209, v213\n\t"
      "v_add_f32 v210, v210, v214\n\t"
      "v_add_f32 v211, v211, v215\n\t"
      "v_add_f32 v216, v216, v220\n\t"
      "v_add_f32 v217, v217, v221\n\t"
      "v_mfma_f32_16x16x128_f8f6f4 v[120:123], a[56:63], a[168:175], v[120:123]\n\t"
      "ds_write_addtid_b32 v108 offset:19456\n\t"
      "ds_write_addtid_b32 v109 offset:19712\n\t"
      "ds_write_addtid_b32 v110 offset:19968\n\t"
      "ds_write_addtid_b32 v111 offset:20224\n\t"
      "ds_read_b128 v[232:235], v18 offset:8192\n\t"
      "v_add_f32 v218, v218, v222\n\t"
      "v_add_f32 v219, v219, v223\n\t"
      "v_add_f32 v208, v208, v216\n\t"
      "v_add_f32 v209, v209, v217\n\t"
      "v_add_f32 v210, v210, v218\n\t"
      "v_add_f32 v211, v211, v219\n\t"
      "v_mfma_f32_16x16x128_f8f6f4 v[124:127], a[56:63], a[176:183], v[124:127]\n\t"
      "ds_write_addtid_b32 v112 offset:20480\n\t"
      "ds_write_addtid_b32 v113 offset:20736\n\t"
      "ds_write_addtid_b32 v114 offset:20992\n\t"
      "ds_write_addtid_b32 v115 offset:21248\n\t"
      "ds_read_b128 v[236:239], v19 offset:8192\n\t"
      "v_mul_f32 v208, v208, v28\n\t"
      "v_mul_f32 v209, v209, v28\n\t"
      "v_mul_f32 v210, v210, v28\n\t"
      "v_mul_f32 v211, v211, v28\n\t"
      "v_maximum3_f32 v208, v208, 0, 0\n\t"
      "v_maximum3_f32 v209, v209, 0, 0\n\t"
      "s_waitcnt vmcnt(4)\n\t"
      "v_mfma_f32_16x16x128_f8f6f4 v[128:131], a[64:71], a[120:127], v[128:131]\n\t"
      "ds_write_addtid_b32 v116 offset:21504\n\t"
      "ds_write_addtid_b32 v117 offset:21760\n\t"
      "ds_write_addtid_b32 v118 offset:22016\n\t"
      "ds_write_addtid_b32 v119 offset:22272\n\t"
      "ds_read_b128 v[240:243], v16 offset:12288\n\t"
      "v_maximum3_f32 v210, v210, 0, 0\n\t"
      "v_maximum3_f32 v211, v211, 0, 0\n\t"
      "buffer_store_dwordx4 v[208:211], v10, s[40:43], s54 offen offset:256 sc1\n\t"
      "s_add_u32 s54, s54, 0x40000\n\t"
      "v_mfma_f32_16x16x128_f8f6f4 v[132:135], a[64:71], a[128:135], v[132:135]\n\t"
      "ds_write_addtid_b32 v120 offset:22528\n\t"
      "ds_write_addtid_b32 v121 offset:22784\n\t"
      "ds_write_addtid_b32 v122 offset:23040\n\t"
      "ds_write_addtid_b32 v123 offset:23296\n\t"
      "ds_read_b128 v[244:247], v17 offset:12288\n\t"
      "s_waitcnt lgkmcnt(10)\n\t"
      "v_add_f32 v224, v224, v228\n\t"
      "v_add_f32 v225, v225, v229\n\t"
      "v_add_f32 v226, v226, v230\n\t"
      "v_add_f32 v227, v227, v231\n\t"
      "v_add_f32 v232, v232, v236\n\t"
      "v_add_f32 v233, v233, v237\n\t"
      "v_mfma_f32_16x16x128_f8f6f4 v[136:139], a[64:71], a[136:143], v[136:139]\n\t"
      "ds_write_addtid_b32 v124 offset:23552\n\t"
      "ds_write_addtid_b32 v125 offset:23808\n\t"
      "ds_write_addtid_b32 v126 offset:24064\n\t"
      "ds_write_addtid_b32 v127 offset:24320\n\t"
      "ds_read_b128 v[248:251], v18 offset:12288\n\t"
      "v_add_f32 v234, v234, v238\n\t"
      "v_add_f32 v235, v235, v239\n\t"
      "v_add_f32 v224, v224, v232\n\t"
      "v_add_f32 v225, v225, v233\n\t"
      "v_add_f32 v226, v226, v234\n\t"
      "v_add_f32 v227, v227, v235\n\t"
      "v_mfma_f32_16x16x128_f8f6f4 v[140:143], a[64:71], a[144:151], v[140:143]\n\t"
      "ds_write_addtid_b32 v128 offset:24576\n\t"
      "ds_write_addtid_b32 v129 offset:24832\n\t"
      "ds_write_addtid_b32 v130 offset:25088\n\t"
      "ds_write_addtid_b32 v131 offset:25344\n\t"
      "ds_read_b128 v[252:255], v19 offset:12288\n\t"
      "v_mul_f32 v224, v224, v28\n\t"
      "v_mul_f32 v225, v225, v28\n\t"
      "v_mul_f32 v226, v226, v28\n\t"
      "v_mul_f32 v227, v227, v28\n\t"
      "v_maximum3_f32 v224, v224, 0, 0\n\t"
      "v_maximum3_f32 v225, v225, 0, 0\n\t"
      "v_mfma_f32_16x16x128_f8f6f4 v[144:147], a[64:71], a[152:159], v[144:147]\n\t"
      "s_waitcnt lgkmcnt(6)\n\t"
      "s_barrier\n\t"
      "ds_write_addtid_b32 v132 offset:25600\n\t"
      "ds_write_addtid_b32 v133 offset:25856\n\t"
      "ds_write_addtid_b32 v134 offset:26112\n\t"
      "ds_write_addtid_b32 v135 offset:26368\n\t"
      "ds_read_b128 v[192:195], v16 offset:16384\n\t"
      "v_maximum3_f32 v226, v226, 0, 0\n\t"
      "v_maximum3_f32 v227, v227, 0, 0\n\t"
      "buffer_store_dwordx4 v[224:227], v10, s[40:43], s54 offen sc1\n\t"
      "v_mfma_f32_16x16x128_f8f6f4 v[148:151], a[64:71], a[160:167], v[148:151]\n\t"
      "ds_write_addtid_b32 v136 offset:26624\n\t"
      "ds_write_addtid_b32 v137 offset:26880\n\t"
      "ds_write_addtid_b32 v138 offset:27136\n\t"
      "ds_write_addtid_b32 v139 offset:27392\n\t"
      "ds_read_b128 v[196:199], v17 offset:16384\n\t"
      "s_waitcnt lgkmcnt(10)\n\t"
      "v_add_f32 v240, v240, v244\n\t"
      "v_add_f32 v241, v241, v245\n\t"
      "v_add_f32 v242, v242, v246\n\t"
      "v_add_f32 v243, v243, v247\n\t"
      "v_add_f32 v248, v248, v252\n\t"
      "v_add_f32 v249, v249, v253\n\t"
      "v_mfma_f32_16x16x128_f8f6f4 v[152:155], a[64:71], a[168:175], v[152:155]\n\t"
      "ds_write_addtid_b32 v140 offset:27648\n\t"
      "ds_write_addtid_b32 v141 offset:27904\n\t"
      "ds_write_addtid_b32 v142 offset:28160\n\t"
      "ds_write_addtid_b32 v143 offset:28416\n\t"
      "ds_read_b128 v[200:203], v18 offset:16384\n\t"
      "v_add_f32 v250, v250, v254\n\t"
      "v_add_f32 v251, v251, v255\n\t"
      "v_add_f32 v240, v240, v248\n\t"
      "v_add_f32 v241, v241, v249\n\t"
      "v_add_f32 v242, v242, v250\n\t"
      "v_add_f32 v243, v243, v251\n\t"
      "v_mfma_f32_16x16x128_f8f6f4 v[156:159], a[64:71], a[176:183], v[156:159]\n\t"
      "ds_write_addtid_b32 v144 offset:28672\n\t"
      "ds_write_addtid_b32 v145 offset:28928\n\t"
      "ds_write_addtid_b32 v146 offset:29184\n\t"
      "ds_write_addtid_b32 v147 offset:29440\n\t"
      "ds_read_b128 v[204:207], v19 offset:16384\n\t"
      "v_mul_f32 v240, v240, v28\n\t"
      "v_mul_f32 v241, v241, v28\n\t"
      "v_mul_f32 v242, v242, v28\n\t"
      "v_mul_f32 v243, v243, v28\n\t"
      "v_maximum3_f32 v240, v240, 0, 0\n\t"
      "v_maximum3_f32 v241, v241, 0, 0\n\t"
      "s_waitcnt vmcnt(4)\n\t"
      "v_mfma_f32_16x16x128_f8f6f4 v[160:163], a[72:79], a[120:127], v[160:163]\n\t"
      "ds_write_addtid_b32 v148 offset:29696\n\t"
      "ds_write_addtid_b32 v149 offset:29952\n\t"
      "ds_write_addtid_b32 v150 offset:30208\n\t"
      "ds_write_addtid_b32 v151 offset:30464\n\t"
      "ds_read_b128 v[208:211], v16 offset:20480\n\t"
      "v_maximum3_f32 v242, v242, 0, 0\n\t"
      "v_maximum3_f32 v243, v243, 0, 0\n\t"
      "buffer_store_dwordx4 v[240:243], v10, s[40:43], s54 offen offset:256 sc1\n\t"
      "s_add_u32 s54, s54, 0x40000\n\t"
      "v_mfma_f32_16x16x128_f8f6f4 v[164:167], a[72:79], a[128:135], v[164:167]\n\t"
      "ds_write_addtid_b32 v152 offset:30720\n\t"
      "ds_write_addtid_b32 v153 offset:30976\n\t"
      "ds_write_addtid_b32 v154 offset:31232\n\t"
      "ds_write_addtid_b32 v155 offset:31488\n\t"
      "ds_read_b128 v[212:215], v17 offset:20480\n\t"
      "s_waitcnt lgkmcnt(10)\n\t"
      "v_add_f32 v192, v192, v196\n\t"
      "v_add_f32 v193, v193, v197\n\t"
      "v_add_f32 v194, v194, v198\n\t"
      "v_add_f32 v195, v195, v199\n\t"
      "v_add_f32 v200, v200, v204\n\t"
      "v_add_f32 v201, v201, v205\n\t"
      "v_mfma_f32_16x16x128_f8f6f4 v[168:171], a[72:79], a[136:143], v[168:171]\n\t"
      "ds_write_addtid_b32 v156 offset:31744\n\t"
      "ds_write_addtid_b32 v157 offset:32000\n\t"
      "ds_write_addtid_b32 v158 offset:32256\n\t"
      "ds_write_addtid_b32 v159 offset:32512\n\t"
      "ds_read_b128 v[216:219], v18 offset:20480\n\t"
      "v_add_f32 v202, v202, v206\n\t"
      "v_add_f32 v203, v203, v207\n\t"
      "v_add_f32 v192, v192, v200\n\t"
      "v_add_f32 v193, v193, v201\n\t"
      "v_add_f32 v194, v194, v202\n\t"
      "v_add_f32 v195, v195, v203\n\t"
      "v_mfma_f32_16x16x128_f8f6f4 v[172:175], a[72:79], a[144:151], v[172:175]\n\t"
      "ds_write_addtid_b32 v160 offset:32768\n\t"
      "ds_write_addtid_b32 v161 offset:33024\n\t"
      "ds_write_addtid_b32 v162 offset:33280\n\t"
      "ds_write_addtid_b32 v163 offset:33536\n\t"
      "ds_read_b128 v[220:223], v19 offset:20480\n\t"
      "v_mul_f32 v192, v192, v28\n\t"
      "v_mul_f32 v193, v193, v28\n\t"
      "v_mul_f32 v194, v194, v28\n\t"
      "v_mul_f32 v195, v195, v28\n\t"
      "v_maximum3_f32 v192, v192, 0, 0\n\t"
      "v_maximum3_f32 v193, v193, 0, 0\n\t"
      "v_mfma_f32_16x16x128_f8f6f4 v[176:179], a[72:79], a[152:159], v[176:179]\n\t"
      "ds_write_addtid_b32 v164 offset:33792\n\t"
      "ds_write_addtid_b32 v165 offset:34048\n\t"
      "ds_write_addtid_b32 v166 offset:34304\n\t"
      "ds_write_addtid_b32 v167 offset:34560\n\t"
      "v_maximum3_f32 v194, v194, 0, 0\n\t"
      "v_maximum3_f32 v195, v195, 0, 0\n\t"
      "buffer_store_dwordx4 v[192:195], v10, s[40:43], s54 offen sc1\n\t"
      "v_mfma_f32_16x16x128_f8f6f4 v[180:183], a[72:79], a[160:167], v[180:183]\n\t"
      "ds_write_addtid_b32 v168 offset:34816\n\t"
      "ds_write_addtid_b32 v169 offset:35072\n\t"
      "ds_write_addtid_b32 v170 offset:35328\n\t"
      "ds_write_addtid_b32 v171 offset:35584\n\t"
      "s_waitcnt lgkmcnt(8)\n\t"
      "v_add_f32 v208, v208, v212\n\t"
      "v_add_f32 v209, v209, v213\n\t"
      "v_add_f32 v210, v210, v214\n\t"
      "v_add_f32 v211, v211, v215\n\t"
      "v_add_f32 v216, v216, v220\n\t"
      "v_add_f32 v217, v217, v221\n\t"
      "v_mfma_f32_16x16x128_f8f6f4 v[184:187], a[72:79], a[168:175], v[184:187]\n\t"
      "ds_write_addtid_b32 v172 offset:35840\n\t"
      "ds_write_addtid_b32 v173 offset:36096\n\t"
      "ds_write_addtid_b32 v174 offset:36352\n\t"
      "ds_write_addtid_b32 v175 offset:36608\n\t"
      "v_add_f32 v218, v218, v222\n\t"
      "v_add_f32 v219, v219, v223\n\t"
      "v_add_f32 v208, v208, v216\n\t"
      "v_add_f32 v209, v209, v217\n\t"
      "v_add_f32 v210, v210, v218\n\t"
      "v_add_f32 v211, v211, v219\n\t"
      "v_mfma_f32_16x16x128_f8f6f4 v[188:191], a[72:79], a[176:183], v[188:191]\n\t"
      "ds_write_addtid_b32 v176 offset:36864\n\t"
      "ds_write_addtid_b32 v177 offset:37120\n\t"
      "ds_write_addtid_b32 v178 offset:37376\n\t"
      "ds_write_addtid_b32 v179 offset:37632\n\t"
      "v_mul_f32 v208, v208, v28\n\t"
      "v_mul_f32 v209, v209, v28\n\t"
      "v_mul_f32 v210, v210, v28\n\t"
      "v_mul_f32 v211, v211, v28\n\t"
      "v_maximum3_f32 v208, v208, 0, 0\n\t"
      "v_maximum3_f32 v209, v209, 0, 0\n\t"
      "s_nop 3\n\t"
      "ds_write_addtid_b32 v180 offset:37888\n\t"
      "ds_write_addtid_b32 v181 offset:38144\n\t"
      "ds_write_addtid_b32 v182 offset:38400\n\t"
      "ds_write_addtid_b32 v183 offset:38656\n\t"
      "ds_write_addtid_b32 v184 offset:38912\n\t"
      "ds_write_addtid_b32 v185 offset:39168\n\t"
      "ds_write_addtid_b32 v186 offset:39424\n\t"
      "ds_write_addtid_b32 v187 offset:39680\n\t"
      "ds_write_addtid_b32 v188 offset:39936\n\t"
      "ds_write_addtid_b32 v189 offset:40192\n\t"
      "ds_write_addtid_b32 v190 offset:40448\n\t"
      "ds_write_addtid_b32 v191 offset:40704\n\t"
      "v_maximum3_f32 v210, v210, 0, 0\n\t"
      "v_maximum3_f32 v211, v211, 0, 0\n\t"
      "buffer_store_dwordx4 v[208:211], v10, s[40:43], s54 offen offset:256 sc1\n\t"
      "s_add_u32 s54, s54, 0x40000\n\t"
      "s_waitcnt lgkmcnt(0)\n\t"
      "s_barrier\n\t"
      "ds_read_b128 v[224:227], v16 offset:24576\n\t"
      "ds_read_b128 v[228:231], v17 offset:24576\n\t"
      "ds_read_b128 v[232:235], v18 offset:24576\n\t"
      "ds_read_b128 v[236:239], v19 offset:24576\n\t"
      "ds_read_b128 v[240:243], v16 offset:28672\n\t"
      "ds_read_b128 v[244:247], v17 offset:28672\n\t"
      "ds_read_b128 v[248:251], v18 offset:28672\n\t"
      "ds_read_b128 v[252:255], v19 offset:28672\n\t"
      "ds_read_b128 v[192:195], v16 offset:32768\n\t"
      "ds_read_b128 v[196:199], v17 offset:32768\n\t"
      "ds_read_b128 v[200:203], v18 offset:32768\n\t"
      "ds_read_b128 v[204:207], v19 offset:32768\n\t"
      "ds_read_b128 v[208:211], v16 offset:36864\n\t"
      "ds_read_b128 v[212:215], v17 offset:36864\n\t"
      "ds_read_b128 v[216:219], v18 offset:36864\n\t"
      "ds_read_b128 v[220:223], v19 offset:36864\n\t"
      "s_waitcnt lgkmcnt(12)\n\t"
      "v_pk_add_f32 v[224:225], v[224:225], v[228:229]\n\t"
      "v_pk_add_f32 v[232:233], v[232:233], v[236:237]\n\t"
      "v_pk_add_f32 v[226:227], v[226:227], v[230:231]\n\t"
      "v_pk_add_f32 v[234:235], v[234:235], v[238:239]\n\t"
      "v_pk_add_f32 v[224:225], v[224:225], v[232:233]\n\t"
      "v_pk_add_f32 v[226:227], v[226:227], v[234:235]\n\t"
      "v_pk_mul_f32 v[224:225], v[224:225], v[28:29] op_sel_hi:[1,0]\n\t"
      "v_pk_mul_f32 v[226:227], v[226:227], v[28:29] op_sel_hi:[1,0]\n\t"
      "v_maximum3_f32 v224, v224, 0, 0\n\t"
      "v_maximum3_f32 v225, v225, 0, 0\n\t"
      "v_maximum3_f32 v226, v226, 0, 0\n\t"
      "v_maximum3_f32 v227, v227, 0, 0\n\t"
      "buffer_store_dwordx4 v[224:227], v10, s[40:43], s54 offen sc1\n\t"
      "s_waitcnt lgkmcnt(8)\n\t"
      "v_pk_add_f32 v[240:241], v[240:241], v[244:245]\n\t"
      "v_pk_add_f32 v[248:249], v[248:249], v[252:253]\n\t"
      "v_pk_add_f32 v[242:243], v[242:243], v[246:247]\n\t"
      "v_pk_add_f32 v[250:251], v[250:251], v[254:255]\n\t"
      "v_pk_add_f32 v[240:241], v[240:241], v[248:249]\n\t"
      "v_pk_add_f32 v[242:243], v[242:243], v[250:251]\n\t"
      "v_pk_mul_f32 v[240:241], v[240:241], v[28:29] op_sel_hi:[1,0]\n\t"
      "v_pk_mul_f32 v[242:243], v[242:243], v[28:29] op_sel_hi:[1,0]\n\t"
      "v_maximum3_f32 v240, v240, 0, 0\n\t"
      "v_maximum3_f32 v241, v241, 0, 0\n\t"
      "v_maximum3_f32 v242, v242, 0, 0\n\t"
      "v_maximum3_f32 v243, v243, 0, 0\n\t"
      "buffer_store_dwordx4 v[240:243], v10, s[40:43], s54 offen offset:256 sc1\n\t"
      "s_add_u32 s54, s54, 0x40000\n\t"
      "s_waitcnt lgkmcnt(4)\n\t"
      "v_pk_add_f32 v[192:193], v[192:193], v[196:197]\n\t"
      "v_pk_add_f32 v[200:201], v[200:201], v[204:205]\n\t"
      "v_pk_add_f32 v[194:195], v[194:195], v[198:199]\n\t"
      "v_pk_add_f32 v[202:203], v[202:203], v[206:207]\n\t"
      "v_pk_add_f32 v[192:193], v[192:193], v[200:201]\n\t"
      "v_pk_add_f32 v[194:195], v[194:195], v[202:203]\n\t"
      "v_pk_mul_f32 v[192:193], v[192:193], v[28:29] op_sel_hi:[1,0]\n\t"
      "v_pk_mul_f32 v[194:195], v[194:195], v[28:29] op_sel_hi:[1,0]\n\t"
      "v_maximum3_f32 v192, v192, 0, 0\n\t"
      "v_maximum3_f32 v193, v193, 0, 0\n\t"
      "v_maximum3_f32 v194, v194, 0, 0\n\t"
      "v_maximum3_f32 v195, v195, 0, 0\n\t"
      "buffer_store_dwordx4 v[192:195], v20, s[40:43], s54 offen sc1\n\t"
      "s_waitcnt lgkmcnt(0)\n\t"
      "v_pk_add_f32 v[208:209], v[208:209], v[212:213]\n\t"
      "v_pk_add_f32 v[216:217], v[216:217], v[220:221]\n\t"
      "v_pk_add_f32 v[210:211], v[210:211], v[214:215]\n\t"
      "v_pk_add_f32 v[218:219], v[218:219], v[222:223]\n\t"
      "v_pk_add_f32 v[208:209], v[208:209], v[216:217]\n\t"
      "v_pk_add_f32 v[210:211], v[210:211], v[218:219]\n\t"
      "v_pk_mul_f32 v[208:209], v[208:209], v[28:29] op_sel_hi:[1,0]\n\t"
      "v_pk_mul_f32 v[210:211], v[210:211], v[28:29] op_sel_hi:[1,0]\n\t"
      "v_maximum3_f32 v208, v208, 0, 0\n\t"
      "v_maximum3_f32 v209, v209, 0, 0\n\t"
      "v_maximum3_f32 v210, v210, 0, 0\n\t"
      "v_maximum3_f32 v211, v211, 0, 0\n\t"
      "buffer_store_dwordx4 v[208:211], v20, s[40:43], s54 offen offset:256 sc1\n\t"
      "s_add_u32 s54, s54, 0x40000\n\t"
      "DONE_%=:\n\t"
      "s_endpgm\n\t"
      "DEEP_%=:\n\t"
      "s_add_u32 m0, s51, 28672\n\t"
      "s_add_u32 s54, s54, 0x20000\n\t"
      "buffer_load_dwordx4 v12, s[28:31], s54 offen lds\n\t"
      "buffer_load_dwordx4 v13, s[28:31], s54 offen offset:1024 lds\n\t"
      "buffer_load_dwordx4 v14, s[28:31], s54 offen offset:2048 lds\n\t"
      "buffer_load_dwordx4 v15, s[28:31], s54 offen offset:3072 lds\n\t"
      "ds_read_b64_tr_b8 a[184:185], v16 offset:32768\n\t"
      "ds_read_b64_tr_b8 a[186:187], v16 offset:33792\n\t"
      "ds_read_b64_tr_b8 a[192:193], v17 offset:32768\n\t"
      "ds_read_b64_tr_b8 a[194:195], v17 offset:33792\n\t"
      "ds_read_b64_tr_b8 a[200:201], v18 offset:32768\n\t"
      "ds_read_b64_tr_b8 a[202:203], v18 offset:33792\n\t"
      "ds_read_b64_tr_b8 a[208:209], v19 offset:32768\n\t"
      "ds_read_b64_tr_b8 a[210:211], v19 offset:33792\n\t"
      "ds_read_b64_tr_b8 a[216:217], v20 offset:32768\n\t"
      "ds_read_b64_tr_b8 a[218:219], v20 offset:33792\n\t"
      "ds_read_b64_tr_b8 a[224:225], v21 offset:32768\n\t"
      "ds_read_b64_tr_b8 a[226:227], v21 offset:33792\n\t"
      "ds_read_b64_tr_b8 a[232:233], v22 offset:32768\n\t"
      "ds_read_b64_tr_b8 a[234:235], v22 offset:33792\n\t"
      "ds_read_b64_tr_b8 a[240:241], v23 offset:32768\n\t"
      "ds_read_b64_tr_b8 a[242:243], v23 offset:33792\n\t"
      "s_bfe_u32 s62, s24, 131080\n\t"
      "s_mul_i32 s62, s62, 10240\n\t"
      "s_bfe_u32 s55, s24, 196618\n\t"
      "s_lshl_b32 s55, s55, 20\n\t"
      "s_add_u32 s62, s62, s60\n\t"
      "s_add_u32 s62, s62, s55\n\t"
      "buffer_load_dword v24, v9, s[32:35], s62 offen sc0 sc1\n\t"
      "s_waitcnt lgkmcnt(0)\n\t"
      "s_bfe_u32 s54, s25, 524288\n\t"
      "s_add_u32 m0, s51, 32768\n\t"
      "s_lshl_b32 s54, s54, 19\n\t"
      "s_xor_b32 s54, s54, s26\n\t"
      "s_xor_b32 s54, s54, 0x40000\n\t"
      "buffer_load_dwordx4 v12, s[28:31], s54 offen lds\n\t"
      "buffer_load_dwordx4 v13, s[28:31], s54 offen offset:1024 lds\n\t"
      "buffer_load_dwordx4 v14, s[28:31], s54 offen offset:2048 lds\n\t"
      "buffer_load_dwordx4 v15, s[28:31], s54 offen offset:3072 lds\n\t"
      "s_add_u32 m0, s51, 36864\n\t"
      "s_add_u32 s54, s54, 0x20000\n\t"
      "buffer_load_dwordx4 v12, s[28:31], s54 offen lds\n\t"
      "buffer_load_dwordx4 v13, s[28:31], s54 offen offset:1024 lds\n\t"
      "buffer_load_dwordx4 v14, s[28:31], s54 offen offset:2048 lds\n\t"
      "buffer_load_dwordx4 v15, s[28:31], s54 offen offset:3072 lds\n\t"
      "ds_read_b64_tr_b8 a[188:189], v16 offset:0\n\t"
      "ds_read_b64_tr_b8 a[190:191], v16 offset:1024\n\t"
      "ds_read_b64_tr_b8 a[196:197], v17 offset:0\n\t"
      "ds_read_b64_tr_b8 a[198:199], v17 offset:1024\n\t"
      "ds_read_b64_tr_b8 a[204:205], v18 offset:0\n\t"
      "ds_read_b64_tr_b8 a[206:207], v18 offset:1024\n\t"
      "ds_read_b64_tr_b8 a[212:213], v19 offset:0\n\t"
      "ds_read_b64_tr_b8 a[214:215], v19 offset:1024\n\t"
      "ds_read_b64_tr_b8 a[220:221], v20 offset:0\n\t"
      "ds_read_b64_tr_b8 a[222:223], v20 offset:1024\n\t"
      "ds_read_b64_tr_b8 a[228:229], v21 offset:0\n\t"
      "ds_read_b64_tr_b8 a[230:231], v21 offset:1024\n\t"
      "ds_read_b64_tr_b8 a[236:237], v22 offset:0\n\t"
      "ds_read_b64_tr_b8 a[238:239], v22 offset:1024\n\t"
      "ds_read_b64_tr_b8 a[244:245], v23 offset:0\n\t"
      "ds_read_b64_tr_b8 a[246:247], v23 offset:1024\n\t"
      "s_waitcnt lgkmcnt(0)\n\t"
      "s_bfe_u32 s54, s25, 524296\n\t"
      "s_add_u32 m0, s51, 0\n\t"
      "s_lshl_b32 s54, s54, 19\n\t"
      "s_xor_b32 s54, s54, s26\n\t"
      "buffer_load_dwordx4 v12, s[28:31], s54 offen lds\n\t"
      "buffer_load_dwordx4 v13, s[28:31], s54 offen offset:1024 lds\n\t"
      "buffer_load_dwordx4 v14, s[28:31], s54 offen offset:2048 lds\n\t"
      "buffer_load_dwordx4 v15, s[28:31], s54 offen offset:3072 lds\n\t"
      "s_waitcnt vmcnt(12)\n\t"
      "v_cmp_le_u32 vcc, s59, v24\n\t"
      "s_mov_b32 s63, s62\n\t"
      "s_cmp_lg_u32 vcc_lo, 3\n\t"
      "s_cbranch_scc1 ALT1D_%=\n\t"
      "buffer_load_dwordx4 a[40:43], v7, s[32:35], s63 offen sc0 sc1\n\t"
      "buffer_load_dwordx4 a[44:47], v3, s[32:35], s63 offen sc0 sc1\n\t"
      "buffer_load_dwordx4 a[48:51], v7, s[32:35], s63 offen offset:2048 sc0 sc1\n\t"
      "buffer_load_dwordx4 a[52:55], v3, s[32:35], s63 offen offset:2048 sc0 sc1\n\t"
      "s_add_u32 s63, s63, 0x1000\n\t"
      "buffer_load_dwordx4 a[56:59], v7, s[32:35], s63 offen sc0 sc1\n\t"
      "buffer_load_dwordx4 a[60:63], v3, s[32:35], s63 offen sc0 sc1\n\t"
      "buffer_load_dwordx4 a[64:67], v7, s[32:35], s63 offen offset:2048 sc0 sc1\n\t"
      "buffer_load_dwordx4 a[68:71], v3, s[32:35], s63 offen offset:2048 sc0 sc1\n\t"
      "s_add_u32 s63, s63, 0x1000\n\t"
      "buffer_load_dwordx4 a[72:75], v8, s[32:35], s63 offen sc0 sc1\n\t"
      "buffer_load_dwordx4 a[76:79], v4, s[32:35], s63 offen sc0 sc1\n\t"
      "s_add_u32 m0, s51, 4096\n\t"
      "s_add_u32 s54, s54, 0x20000\n\t"
      "buffer_load_dwordx4 v12, s[28:31], s54 offen lds\n\t"
      "buffer_load_dwordx4 v13, s[28:31], s54 offen offset:1024 lds\n\t"
      "buffer_load_dwordx4 v14, s[28:31], s54 offen offset:2048 lds\n\t"
      "buffer_load_dwordx4 v15, s[28:31], s54 offen offset:3072 lds\n\t"
      "ds_read_b64_tr_b8 v[192:193], v16 offset:8192\n\t"
      "ds_read_b64_tr_b8 v[194:195], v16 offset:9216\n\t"
      "ds_read_b64_tr_b8 v[200:201], v17 offset:8192\n\t"
      "ds_read_b64_tr_b8 v[202:203], v17 offset:9216\n\t"
      "ds_read_b64_tr_b8 v[208:209], v18 offset:8192\n\t"
      "ds_read_b64_tr_b8 v[210:211], v18 offset:9216\n\t"
      "ds_read_b64_tr_b8 v[216:217], v19 offset:8192\n\t"
      "ds_read_b64_tr_b8 v[218:219], v19 offset:9216\n\t"
      "ds_read_b64_tr_b8 v[224:225], v20 offset:8192\n\t"
      "ds_read_b64_tr_b8 v[226:227], v20 offset:9216\n\t"
      "ds_read_b64_tr_b8 v[232:233], v21 offset:8192\n\t"
      "ds_read_b64_tr_b8 v[234:235], v21 offset:9216\n\t"
      "ds_read_b64_tr_b8 v[240:241], v22 offset:8192\n\t"
      "ds_read_b64_tr_b8 v[242:243], v22 offset:9216\n\t"
      "ds_read_b64_tr_b8 v[248:249], v23 offset:8192\n\t"
      "ds_read_b64_tr_b8 v[250:251], v23 offset:9216\n\t"
      "JOIN1D_%=:\n\t"
      "s_waitcnt vmcnt(12)\n\t"
      "v_mfma_f32_16x16x128_f8f6f4 v[32:35], a[40:47], a[120:127], v[32:35]\n\t"
      "s_bfe_u32 s62, s24, 131088\n\t"
      "s_mul_i32 s62, s62, 10240\n\t"
      "s_bfe_u32 s55, s24, 196626\n\t"
      "s_lshl_b32 s55, s55, 20\n\t"
      "s_add_u32 s62, s62, s60\n\t"
      "s_add_u32 s62, s62, s55\n\t"
      "buffer_load_dword v24, v9, s[32:35], s62 offen sc0 sc1\n\t"
      "ds_read_b64_tr_b8 v[196:197], v16 offset:16384\n\t"
      "ds_read_b64_tr_b8 v[198:199], v16 offset:17408\n\t"
      "v_mfma_f32_16x16x128_f8f6f4 v[36:39], a[40:47], a[128:135], v[36:39]\n\t"
      "ds_read_b64_tr_b8 v[204:205], v17 offset:16384\n\t"
      "ds_read_b64_tr_b8 v[206:207], v17 offset:17408\n\t"
      "v_mfma_f32_16x16x128_f8f6f4 v[40:43], a[40:47], a[136:143], v[40:43]\n\t"
      "ds_read_b64_tr_b8 v[212:213], v18 offset:16384\n\t"
      "ds_read_b64_tr_b8 v[214:215], v18 offset:17408\n\t"
      "v_mfma_f32_16x16x128_f8f6f4 v[44:47], a[40:47], a[144:151], v[44:47]\n\t"
      "ds_read_b64_tr_b8 v[220:221], v19 offset:16384\n\t"
      "ds_read_b64_tr_b8 v[222:223], v19 offset:17408\n\t"
      "v_mfma_f32_16x16x128_f8f6f4 v[48:51], a[40:47], a[152:159], v[48:51]\n\t"
      "ds_read_b64_tr_b8 v[228:229], v20 offset:16384\n\t"
      "ds_read_b64_tr_b8 v[230:231], v20 offset:17408\n\t"
      "v_mfma_f32_16x16x128_f8f6f4 v[52:55], a[40:47], a[160:167], v[52:55]\n\t"
      "ds_read_b64_tr_b8 v[236:237], v21 offset:16384\n\t"
      "ds_read_b64_tr_b8 v[238:239], v21 offset:17408\n\t"
      "v_mfma_f32_16x16x128_f8f6f4 v[56:59], a[40:47], a[168:175], v[56:59]\n\t"
      "ds_read_b64_tr_b8 v[244:245], v22 offset:16384\n\t"
      "ds_read_b64_tr_b8 v[246:247], v22 offset:17408\n\t"
      "v_mfma_f32_16x16x128_f8f6f4 v[60:63], a[40:47], a[176:183], v[60:63]\n\t"
      "ds_read_b64_tr_b8 v[252:253], v23 offset:16384\n\t"
      "ds_read_b64_tr_b8 v[254:255], v23 offset:17408\n\t"
      "s_waitcnt vmcnt(11)\n\t"
      "v_mfma_f32_16x16x128_f8f6f4 v[64:67], a[48:55], a[120:127], v[64:67]\n\t"
      "v_mfma_f32_16x16x128_f8f6f4 v[68:71], a[48:55], a[128:135], v[68:71]\n\t"
      "v_mfma_f32_16x16x128_f8f6f4 v[72:75], a[48:55], a[136:143], v[72:75]\n\t"
      "v_mfma_f32_16x16x128_f8f6f4 v[76:79], a[48:55], a[144:151], v[76:79]\n\t"
      "v_mfma_f32_16x16x128_f8f6f4 v[80:83], a[48:55], a[152:159], v[80:83]\n\t"
      "v_mfma_f32_16x16x128_f8f6f4 v[84:87], a[48:55], a[160:167], v[84:87]\n\t"
      "v_mfma_f32_16x16x128_f8f6f4 v[88:91], a[48:55], a[168:175], v[88:91]\n\t"
      "v_mfma_f32_16x16x128_f8f6f4 v[92:95], a[48:55], a[176:183], v[92:95]\n\t"
      "s_waitcnt vmcnt(9)\n\t"
      "v_mfma_f32_16x16x128_f8f6f4 v[96:99], a[56:63], a[120:127], v[96:99]\n\t"
      "v_mfma_f32_16x16x128_f8f6f4 v[100:103], a[56:63], a[128:135], v[100:103]\n\t"
      "v_mfma_f32_16x16x128_f8f6f4 v[104:107], a[56:63], a[136:143], v[104:107]\n\t"
      "s_waitcnt vmcnt(0)\n\t"
      "v_cmp_le_u32 vcc, s59, v24\n\t"
      "s_mov_b32 s63, s62\n\t"
      "s_cmp_lg_u32 vcc_lo, 3\n\t"
      "s_cbranch_scc1 ALT2D_%=\n\t"
      "buffer_load_dwordx4 a[80:83], v7, s[32:35], s63 offen sc0 sc1\n\t"
      "v_mfma_f32_16x16x128_f8f6f4 v[108:111], a[56:63], a[144:151], v[108:111]\n\t"
      "v_mfma_f32_16x16x128_f8f6f4 v[112:115], a[56:63], a[152:159], v[112:115]\n\t"
      "buffer_load_dwordx4 a[84:87], v3, s[32:35], s63 offen sc0 sc1\n\t"
      "v_mfma_f32_16x16x128_f8f6f4 v[116:119], a[56:63], a[160:167], v[116:119]\n\t"
      "v_mfma_f32_16x16x128_f8f6f4 v[120:123], a[56:63], a[168:175], v[120:123]\n\t"
      "buffer_load_dwordx4 a[88:91], v7, s[32:35], s63 offen offset:2048 sc0 sc1\n\t"
      "v_mfma_f32_16x16x128_f8f6f4 v[124:127], a[56:63], a[176:183], v[124:127]\n\t"
      "v_mfma_f32_16x16x128_f8f6f4 v[128:131], a[64:71], a[120:127], v[128:131]\n\t"
      "buffer_load_dwordx4 a[92:95], v3, s[32:35], s63 offen offset:2048 sc0 sc1\n\t"
      "v_mfma_f32_16x16x128_f8f6f4 v[132:135], a[64:71], a[128:135], v[132:135]\n\t"
      "v_mfma_f32_16x16x128_f8f6f4 v[136:139], a[64:71], a[136:143], v[136:139]\n\t"
      "v_mfma_f32_16x16x128_f8f6f4 v[140:143], a[64:71], a[144:151], v[140:143]\n\t"
      "s_add_u32 s63, s63, 0x1000\n\t"
      "buffer_load_dwordx4 a[96:99], v7, s[32:35], s63 offen sc0 sc1\n\t"
      "v_mfma_f32_16x16x128_f8f6f4 v[144:147], a[64:71], a[152:159], v[144:147]\n\t"
      "v_mfma_f32_16x16x128_f8f6f4 v[148:151], a[64:71], a[160:167], v[148:151]\n\t"
      "buffer_load_dwordx4 a[100:103], v3, s[32:35], s63 offen sc0 sc1\n\t"
      "v_mfma_f32_16x16x128_f8f6f4 v[152:155], a[64:71], a[168:175], v[152:155]\n\t"
      "v_mfma_f32_16x16x128_f8f6f4 v[156:159], a[64:71], a[176:183], v[156:159]\n\t"
      "buffer_load_dwordx4 a[104:107], v7, s[32:35], s63 offen offset:2048 sc0 sc1\n\t"
      "v_mfma_f32_16x16x128_f8f6f4 v[160:163], a[72:79], a[120:127], v[160:163]\n\t"
      "v_mfma_f32_16x16x128_f8f6f4 v[164:167], a[72:79], a[128:135], v[164:167]\n\t"
      "buffer_load_dwordx4 a[108:111], v3, s[32:35], s63 offen offset:2048 sc0 sc1\n\t"
      "v_mfma_f32_16x16x128_f8f6f4 v[168:171], a[72:79], a[136:143], v[168:171]\n\t"
      "v_mfma_f32_16x16x128_f8f6f4 v[172:175], a[72:79], a[144:151], v[172:175]\n\t"
      "v_mfma_f32_16x16x128_f8f6f4 v[176:179], a[72:79], a[152:159], v[176:179]\n\t"
      "s_add_u32 s63, s63, 0x1000\n\t"
      "buffer_load_dwordx4 a[112:115], v8, s[32:35], s63 offen sc0 sc1\n\t"
      "v_mfma_f32_16x16x128_f8f6f4 v[180:183], a[72:79], a[160:167], v[180:183]\n\t"
      "buffer_load_dwordx4 a[116:119], v4, s[32:35], s63 offen sc0 sc1\n\t"
      "v_mfma_f32_16x16x128_f8f6f4 v[184:187], a[72:79], a[168:175], v[184:187]\n\t"
      "v_mfma_f32_16x16x128_f8f6f4 v[188:191], a[72:79], a[176:183], v[188:191]\n\t"
      "JOIN2D_%=:\n\t"
      "s_waitcnt vmcnt(8)\n\t"
      "v_mfma_f32_16x16x128_f8f6f4 v[32:35], a[80:87], a[184:191], v[32:35]\n\t"
      "s_bfe_u32 s62, s24, 131096\n\t"
      "s_mul_i32 s62, s62, 10240\n\t"
      "s_bfe_u32 s55, s24, 196634\n\t"
      "s_lshl_b32 s55, s55, 20\n\t"
      "s_add_u32 s62, s62, s60\n\t"
      "s_add_u32 s62, s62, s55\n\t"
      "buffer_load_dword v24, v9, s[32:35], s62 offen sc0 sc1\n\t"
      "ds_read_b64_tr_b8 a[120:121], v16 offset:24576\n\t"
      "ds_read_b64_tr_b8 a[122:123], v16 offset:25600\n\t"
      "s_waitcnt lgkmcnt(15)\n\t"
      "s_bfe_u32 s54, s25, 524296\n\t"
      "s_add_u32 m0, s51, 8192\n\t"
      "s_lshl_b32 s54, s54, 19\n\t"
      "s_xor_b32 s54, s54, s26\n\t"
      "s_xor_b32 s54, s54, 0x40000\n\t"
      "buffer_load_dwordx4 v12, s[28:31], s54 offen lds\n\t"
      "v_mfma_f32_16x16x128_f8f6f4 v[36:39], a[80:87], a[192:199], v[36:39]\n\t"
      "ds_read_b64_tr_b8 a[128:129], v17 offset:24576\n\t"
      "ds_read_b64_tr_b8 a[130:131], v17 offset:25600\n\t"
      "v_mfma_f32_16x16x128_f8f6f4 v[40:43], a[80:87], a[200:207], v[40:43]\n\t"
      "ds_read_b64_tr_b8 a[136:137], v18 offset:24576\n\t"
      "ds_read_b64_tr_b8 a[138:139], v18 offset:25600\n\t"
      "buffer_load_dwordx4 v13, s[28:31], s54 offen offset:1024 lds\n\t"
      "v_mfma_f32_16x16x128_f8f6f4 v[44:47], a[80:87], a[208:215], v[44:47]\n\t"
      "ds_read_b64_tr_b8 a[144:145], v19 offset:24576\n\t"
      "ds_read_b64_tr_b8 a[146:147], v19 offset:25600\n\t"
      "v_mfma_f32_16x16x128_f8f6f4 v[48:51], a[80:87], a[216:223], v[48:51]\n\t"
      "ds_read_b64_tr_b8 a[152:153], v20 offset:24576\n\t"
      "ds_read_b64_tr_b8 a[154:155], v20 offset:25600\n\t"
      "buffer_load_dwordx4 v14, s[28:31], s54 offen offset:2048 lds\n\t"
      "v_mfma_f32_16x16x128_f8f6f4 v[52:55], a[80:87], a[224:231], v[52:55]\n\t"
      "ds_read_b64_tr_b8 a[160:161], v21 offset:24576\n\t"
      "ds_read_b64_tr_b8 a[162:163], v21 offset:25600\n\t"
      "v_mfma_f32_16x16x128_f8f6f4 v[56:59], a[80:87], a[232:239], v[56:59]\n\t"
      "ds_read_b64_tr_b8 a[168:169], v22 offset:24576\n\t"
      "ds_read_b64_tr_b8 a[170:171], v22 offset:25600\n\t"
      "buffer_load_dwordx4 v15, s[28:31], s54 offen offset:3072 lds\n\t"
      "v_mfma_f32_16x16x128_f8f6f4 v[60:63], a[80:87], a[240:247], v[60:63]\n\t"
      "ds_read_b64_tr_b8 a[176:177], v23 offset:24576\n\t"
      "ds_read_b64_tr_b8 a[178:179], v23 offset:25600\n\t"
      "s_waitcnt vmcnt(11)\n\t"
      "v_mfma_f32_16x16x128_f8f6f4 v[64:67], a[88:95], a[184:191], v[64:67]\n\t"
      "ds_read_b64_tr_b8 a[124:125], v16 offset:32768\n\t"
      "ds_read_b64_tr_b8 a[126:127], v16 offset:33792\n\t"
      "v_mfma_f32_16x16x128_f8f6f4 v[68:71], a[88:95], a[192:199], v[68:71]\n\t"
      "ds_read_b64_tr_b8 a[132:133], v17 offset:32768\n\t"
      "ds_read_b64_tr_b8 a[134:135], v17 offset:33792\n\t"
      "s_add_u32 m0, s51, 12288\n\t"
      "s_add_u32 s54, s54, 0x20000\n\t"
      "buffer_load_dwordx4 v12, s[28:31], s54 offen lds\n\t"
      "v_mfma_f32_16x16x128_f8f6f4 v[72:75], a[88:95], a[200:207], v[72:75]\n\t"
      "ds_read_b64_tr_b8 a[140:141], v18 offset:32768\n\t"
      "ds_read_b64_tr_b8 a[142:143], v18 offset:33792\n\t"
      "v_mfma_f32_16x16x128_f8f6f4 v[76:79], a[88:95], a[208:215], v[76:79]\n\t"
      "ds_read_b64_tr_b8 a[148:149], v19 offset:32768\n\t"
      "ds_read_b64_tr_b8 a[150:151], v19 offset:33792\n\t"
      "buffer_load_dwordx4 v13, s[28:31], s54 offen offset:1024 lds\n\t"
      "v_mfma_f32_16x16x128_f8f6f4 v[80:83], a[88:95], a[216:223], v[80:83]\n\t"
      "ds_read_b64_tr_b8 a[156:157], v20 offset:32768\n\t"
      "ds_read_b64_tr_b8 a[158:159], v20 offset:33792\n\t"
      "v_mfma_f32_16x16x128_f8f6f4 v[84:87], a[88:95], a[224:231], v[84:87]\n\t"
      "ds_read_b64_tr_b8 a[164:165], v21 offset:32768\n\t"
      "ds_read_b64_tr_b8 a[166:167], v21 offset:33792\n\t"
      "buffer_load_dwordx4 v14, s[28:31], s54 offen offset:2048 lds\n\t"
      "v_mfma_f32_16x16x128_f8f6f4 v[88:91], a[88:95], a[232:239], v[88:91]\n\t"
      "ds_read_b64_tr_b8 a[172:173], v22 offset:32768\n\t"
      "ds_read_b64_tr_b8 a[174:175], v22 offset:33792\n\t"
      "v_mfma_f32_16x16x128_f8f6f4 v[92:95], a[88:95], a[240:247], v[92:95]\n\t"
      "ds_read_b64_tr_b8 a[180:181], v23 offset:32768\n\t"
      "ds_read_b64_tr_b8 a[182:183], v23 offset:33792\n\t"
      "buffer_load_dwordx4 v15, s[28:31], s54 offen offset:3072 lds\n\t"
      "s_waitcnt vmcnt(13)\n\t"
      "v_mfma_f32_16x16x128_f8f6f4 v[96:99], a[96:103], a[184:191], v[96:99]\n\t"
      "v_mfma_f32_16x16x128_f8f6f4 v[100:103], a[96:103], a[192:199], v[100:103]\n\t"
      "v_mfma_f32_16x16x128_f8f6f4 v[104:107], a[96:103], a[200:207], v[104:107]\n\t"
      "s_waitcnt vmcnt(8)\n\t"
      "v_cmp_le_u32 vcc, s59, v24\n\t"
      "s_mov_b32 s63, s62\n\t"
      "s_cmp_lg_u32 vcc_lo, 3\n\t"
      "s_cbranch_scc1 ALT3D_%=\n\t"
      "buffer_load_dwordx4 a[0:3], v7, s[32:35], s63 offen sc0 sc1\n\t"
      "v_mfma_f32_16x16x128_f8f6f4 v[108:111], a[96:103], a[208:215], v[108:111]\n\t"
      "v_mfma_f32_16x16x128_f8f6f4 v[112:115], a[96:103], a[216:223], v[112:115]\n\t"
      "buffer_load_dwordx4 a[4:7], v3, s[32:35], s63 offen sc0 sc1\n\t"
      "v_mfma_f32_16x16x128_f8f6f4 v[116:119], a[96:103], a[224:231], v[116:119]\n\t"
      "v_mfma_f32_16x16x128_f8f6f4 v[120:123], a[96:103], a[232:239], v[120:123]\n\t"
      "buffer_load_dwordx4 a[8:11], v7, s[32:35], s63 offen offset:2048 sc0 sc1\n\t"
      "v_mfma_f32_16x16x128_f8f6f4 v[124:127], a[96:103], a[240:247], v[124:127]\n\t"
      "v_mfma_f32_16x16x128_f8f6f4 v[128:131], a[104:111], a[184:191], v[128:131]\n\t"
      "v_mfma_f32_16x16x128_f8f6f4 v[132:135], a[104:111], a[192:199], v[132:135]\n\t"
      "buffer_load_dwordx4 a[12:15], v3, s[32:35], s63 offen offset:2048 sc0 sc1\n\t"
      "v_mfma_f32_16x16x128_f8f6f4 v[136:139], a[104:111], a[200:207], v[136:139]\n\t"
      "v_mfma_f32_16x16x128_f8f6f4 v[140:143], a[104:111], a[208:215], v[140:143]\n\t"
      "s_add_u32 s63, s63, 0x1000\n\t"
      "buffer_load_dwordx4 a[16:19], v7, s[32:35], s63 offen sc0 sc1\n\t"
      "v_mfma_f32_16x16x128_f8f6f4 v[144:147], a[104:111], a[216:223], v[144:147]\n\t"
      "v_mfma_f32_16x16x128_f8f6f4 v[148:151], a[104:111], a[224:231], v[148:151]\n\t"
      "buffer_load_dwordx4 a[20:23], v3, s[32:35], s63 offen sc0 sc1\n\t"
      "v_mfma_f32_16x16x128_f8f6f4 v[152:155], a[104:111], a[232:239], v[152:155]\n\t"
      "v_mfma_f32_16x16x128_f8f6f4 v[156:159], a[104:111], a[240:247], v[156:159]\n\t"
      "buffer_load_dwordx4 a[24:27], v7, s[32:35], s63 offen offset:2048 sc0 sc1\n\t"
      "v_mfma_f32_16x16x128_f8f6f4 v[160:163], a[112:119], a[184:191], v[160:163]\n\t"
      "v_mfma_f32_16x16x128_f8f6f4 v[164:167], a[112:119], a[192:199], v[164:167]\n\t"
      "v_mfma_f32_16x16x128_f8f6f4 v[168:171], a[112:119], a[200:207], v[168:171]\n\t"
      "buffer_load_dwordx4 a[28:31], v3, s[32:35], s63 offen offset:2048 sc0 sc1\n\t"
      "v_mfma_f32_16x16x128_f8f6f4 v[172:175], a[112:119], a[208:215], v[172:175]\n\t"
      "v_mfma_f32_16x16x128_f8f6f4 v[176:179], a[112:119], a[216:223], v[176:179]\n\t"
      "s_add_u32 s63, s63, 0x1000\n\t"
      "buffer_load_dwordx4 a[32:35], v8, s[32:35], s63 offen sc0 sc1\n\t"
      "v_mfma_f32_16x16x128_f8f6f4 v[180:183], a[112:119], a[224:231], v[180:183]\n\t"
      "buffer_load_dwordx4 a[36:39], v4, s[32:35], s63 offen sc0 sc1\n\t"
      "v_mfma_f32_16x16x128_f8f6f4 v[184:187], a[112:119], a[232:239], v[184:187]\n\t"
      "v_mfma_f32_16x16x128_f8f6f4 v[188:191], a[112:119], a[240:247], v[188:191]\n\t"
      "JOIN3D_%=:\n\t"
      "s_waitcnt lgkmcnt(0)\n\t"
      "s_branch CONV_%=\n\t"
      "ALT1D_%=:\n\t"
      "s_mov_b32 s21, 20496\n\t"
      "s_mov_b32 s58, 1\n\t"
      "s_call_b64 s[66:67], SETC_%=\n\t"
      "ds_read_b64_tr_b8 v[192:193], v16 offset:8192\n\t"
      "ds_read_b64_tr_b8 v[194:195], v16 offset:9216\n\t"
      "ds_read_b64_tr_b8 v[200:201], v17 offset:8192\n\t"
      "ds_read_b64_tr_b8 v[202:203], v17 offset:9216\n\t"
      "ds_read_b64_tr_b8 v[208:209], v18 offset:8192\n\t"
      "ds_read_b64_tr_b8 v[210:211], v18 offset:9216\n\t"
      "ds_read_b64_tr_b8 v[216:217], v19 offset:8192\n\t"
      "ds_read_b64_tr_b8 v[218:219], v19 offset:9216\n\t"
      "ds_read_b64_tr_b8 v[224:225], v20 offset:8192\n\t"
      "ds_read_b64_tr_b8 v[226:227], v20 offset:9216\n\t"
      "ds_read_b64_tr_b8 v[232:233], v21 offset:8192\n\t"
      "ds_read_b64_tr_b8 v[234:235], v21 offset:9216\n\t"
      "ds_read_b64_tr_b8 v[240:241], v22 offset:8192\n\t"
      "ds_read_b64_tr_b8 v[242:243], v22 offset:9216\n\t"
      "ds_read_b64_tr_b8 v[248:249], v23 offset:8192\n\t"
      "ds_read_b64_tr_b8 v[250:251], v23 offset:9216\n\t"
      "s_call_b64 s[66:67], SPIN1_%=\n\t"
      "s_cmp_lg_u32 s21, 0\n\t"
      "s_cbranch_scc1 ALS1D_%=\n\t"
      "buffer_load_dwordx4 a[40:43], v7, s[32:35], s63 offen sc0 sc1\n\t"
      "buffer_load_dwordx4 a[44:47], v3, s[32:35], s63 offen sc0 sc1\n\t"
      "buffer_load_dwordx4 a[48:51], v7, s[32:35], s63 offen offset:2048 sc0 sc1\n\t"
      "buffer_load_dwordx4 a[52:55], v3, s[32:35], s63 offen offset:2048 sc0 sc1\n\t"
      "s_and_b32 vcc_lo, vcc_lo, 2\n\t"
      "s_cbranch_scc0 ALB1D_%=\n\t"
      "ALC1D_%=:\n\t"
      "s_add_u32 s63, s63, 0x1000\n\t"
      "buffer_load_dwordx4 a[56:59], v7, s[32:35], s63 offen sc0 sc1\n\t"
      "buffer_load_dwordx4 a[60:63], v3, s[32:35], s63 offen sc0 sc1\n\t"
      "buffer_load_dwordx4 a[64:67], v7, s[32:35], s63 offen offset:2048 sc0 sc1\n\t"
      "buffer_load_dwordx4 a[68:71], v3, s[32:35], s63 offen offset:2048 sc0 sc1\n\t"
      "s_add_u32 s63, s63, 0x1000\n\t"
      "buffer_load_dwordx4 a[72:75], v8, s[32:35], s63 offen sc0 sc1\n\t"
      "buffer_load_dwordx4 a[76:79], v4, s[32:35], s63 offen sc0 sc1\n\t"
      "ALD1D_%=:\n\t"
      "s_add_u32 m0, s51, 4096\n\t"
      "s_add_u32 s54, s54, 0x20000\n\t"
      "buffer_load_dwordx4 v12, s[28:31], s54 offen lds\n\t"
      "buffer_load_dwordx4 v13, s[28:31], s54 offen offset:1024 lds\n\t"
      "buffer_load_dwordx4 v14, s[28:31], s54 offen offset:2048 lds\n\t"
      "buffer_load_dwordx4 v15, s[28:31], s54 offen offset:3072 lds\n\t"
      "s_branch JOIN1D_%=\n\t"
      "ALB1D_%=:\n\t"
      "s_call_b64 s[66:67], SPINB_%=\n\t"
      "s_branch ALC1D_%=\n\t"
      "ALS1D_%=:\n\t"
      "s_mov_b32 s56, 8\n\t"
      "s_call_b64 s[66:67], SWAP_%=\n\t"
      "s_waitcnt lgkmcnt(0)\n\t"
      "s_bitcmp0_b32 s21, 4\n\t"
      "s_cbranch_scc1 ALN12D_%=\n\t"
      "buffer_load_dwordx4 a[40:43], v7, s[32:35], s63 offen sc0 sc1\n\t"
      "buffer_load_dwordx4 a[44:47], v3, s[32:35], s63 offen sc0 sc1\n\t"
      "buffer_load_dwordx4 a[48:51], v7, s[32:35], s63 offen offset:2048 sc0 sc1\n\t"
      "buffer_load_dwordx4 a[52:55], v3, s[32:35], s63 offen offset:2048 sc0 sc1\n\t"
      "s_add_u32 s63, s63, 0x1000\n\t"
      "buffer_load_dwordx4 a[56:59], v7, s[32:35], s63 offen sc0 sc1\n\t"
      "buffer_load_dwordx4 a[60:63], v3, s[32:35], s63 offen sc0 sc1\n\t"
      "buffer_load_dwordx4 a[64:67], v7, s[32:35], s63 offen offset:2048 sc0 sc1\n\t"
      "buffer_load_dwordx4 a[68:71], v3, s[32:35], s63 offen offset:2048 sc0 sc1\n\t"
      "s_add_u32 s63, s63, 0x1000\n\t"
      "buffer_load_dwordx4 a[72:75], v8, s[32:35], s63 offen sc0 sc1\n\t"
      "buffer_load_dwordx4 a[76:79], v4, s[32:35], s63 offen sc0 sc1\n\t"
      "v_accvgpr_read_b32 v11, a120\n\t"
      "v_accvgpr_read_b32 v26, a121\n\t"
      "v_accvgpr_mov_b32 a120, a184\n\t"
      "v_accvgpr_mov_b32 a121, a185\n\t"
      "v_accvgpr_write_b32 a184, v11\n\t"
      "v_accvgpr_write_b32 a185, v26\n\t"
      "v_accvgpr_read_b32 v11, a122\n\t"
      "v_accvgpr_read_b32 v26, a123\n\t"
      "v_accvgpr_mov_b32 a122, a186\n\t"
      "v_accvgpr_mov_b32 a123, a187\n\t"
      "v_accvgpr_write_b32 a186, v11\n\t"
      "v_accvgpr_write_b32 a187, v26\n\t"
      "v_accvgpr_read_b32 v11, a128\n\t"
      "v_accvgpr_read_b32 v26, a129\n\t"
      "v_accvgpr_mov_b32 a128, a192\n\t"
      "v_accvgpr_mov_b32 a129, a193\n\t"
      "v_accvgpr_write_b32 a192, v11\n\t"
      "v_accvgpr_write_b32 a193, v26\n\t"
      "v_accvgpr_read_b32 v11, a130\n\t"
      "v_accvgpr_read_b32 v26, a131\n\t"
      "v_accvgpr_mov_b32 a130, a194\n\t"
      "v_accvgpr_mov_b32 a131, a195\n\t"
      "v_accvgpr_write_b32 a194, v11\n\t"
      "v_accvgpr_write_b32 a195, v26\n\t"
      "v_accvgpr_read_b32 v11, a136\n\t"
      "v_accvgpr_read_b32 v26, a137\n\t"
      "v_accvgpr_mov_b32 a136, a200\n\t"
      "v_accvgpr_mov_b32 a137, a201\n\t"
      "v_accvgpr_write_b32 a200, v11\n\t"
      "v_accvgpr_write_b32 a201, v26\n\t"
      "v_accvgpr_read_b32 v11, a138\n\t"
      "v_accvgpr_read_b32 v26, a139\n\t"
      "v_accvgpr_mov_b32 a138, a202\n\t"
      "v_accvgpr_mov_b32 a139, a203\n\t"
      "v_accvgpr_write_b32 a202, v11\n\t"
      "v_accvgpr_write_b32 a203, v26\n\t"
      "v_accvgpr_read_b32 v11, a144\n\t"
      "v_accvgpr_read_b32 v26, a145\n\t"
      "v_accvgpr_mov_b32 a144, a208\n\t"
      "v_accvgpr_mov_b32 a145, a209\n\t"
      "v_accvgpr_write_b32 a208, v11\n\t"
      "v_accvgpr_write_b32 a209, v26\n\t"
      "v_accvgpr_read_b32 v11, a146\n\t"
      "v_accvgpr_read_b32 v26, a147\n\t"
      "v_accvgpr_mov_b32 a146, a210\n\t"
      "v_accvgpr_mov_b32 a147, a211\n\t"
      "v_accvgpr_write_b32 a210, v11\n\t"
      "v_accvgpr_write_b32 a211, v26\n\t"
      "v_accvgpr_read_b32 v11, a152\n\t"
      "v_accvgpr_read_b32 v26, a153\n\t"
      "v_accvgpr_mov_b32 a152, a216\n\t"
      "v_accvgpr_mov_b32 a153, a217\n\t"
      "v_accvgpr_write_b32 a216, v11\n\t"
      "v_accvgpr_write_b32 a217, v26\n\t"
      "v_accvgpr_read_b32 v11, a154\n\t"
      "v_accvgpr_read_b32 v26, a155\n\t"
      "v_accvgpr_mov_b32 a154, a218\n\t"
      "v_accvgpr_mov_b32 a155, a219\n\t"
      "v_accvgpr_write_b32 a218, v11\n\t"
      "v_accvgpr_write_b32 a219, v26\n\t"
      "v_accvgpr_read_b32 v11, a160\n\t"
      "v_accvgpr_read_b32 v26, a161\n\t"
      "v_accvgpr_mov_b32 a160, a224\n\t"
      "v_accvgpr_mov_b32 a161, a225\n\t"
      "v_accvgpr_write_b32 a224, v11\n\t"
      "v_accvgpr_write_b32 a225, v26\n\t"
      "v_accvgpr_read_b32 v11, a162\n\t"
      "v_accvgpr_read_b32 v26, a163\n\t"
      "v_accvgpr_mov_b32 a162, a226\n\t"
      "v_accvgpr_mov_b32 a163, a227\n\t"
      "v_accvgpr_write_b32 a226, v11\n\t"
      "v_accvgpr_write_b32 a227, v26\n\t"
      "v_accvgpr_read_b32 v11, a168\n\t"
      "v_accvgpr_read_b32 v26, a169\n\t"
      "v_accvgpr_mov_b32 a168, a232\n\t"
      "v_accvgpr_mov_b32 a169, a233\n\t"
      "v_accvgpr_write_b32 a232, v11\n\t"
      "v_accvgpr_write_b32 a233, v26\n\t"
      "v_accvgpr_read_b32 v11, a170\n\t"
      "v_accvgpr_read_b32 v26, a171\n\t"
      "v_accvgpr_mov_b32 a170, a234\n\t"
      "v_accvgpr_mov_b32 a171, a235\n\t"
      "v_accvgpr_write_b32 a234, v11\n\t"
      "v_accvgpr_write_b32 a235, v26\n\t"
      "v_accvgpr_read_b32 v11, a176\n\t"
      "v_accvgpr_read_b32 v26, a177\n\t"
      "v_accvgpr_mov_b32 a176, a240\n\t"
      "v_accvgpr_mov_b32 a177, a241\n\t"
      "v_accvgpr_write_b32 a240, v11\n\t"
      "v_accvgpr_write_b32 a241, v26\n\t"
      "v_accvgpr_read_b32 v11, a178\n\t"
      "v_accvgpr_read_b32 v26, a179\n\t"
      "v_accvgpr_mov_b32 a178, a242\n\t"
      "v_accvgpr_mov_b32 a179, a243\n\t"
      "v_accvgpr_write_b32 a242, v11\n\t"
      "v_accvgpr_write_b32 a243, v26\n\t"
      "v_accvgpr_read_b32 v11, a124\n\t"
      "v_accvgpr_read_b32 v26, a125\n\t"
      "v_accvgpr_mov_b32 a124, a188\n\t"
      "v_accvgpr_mov_b32 a125, a189\n\t"
      "v_accvgpr_write_b32 a188, v11\n\t"
      "v_accvgpr_write_b32 a189, v26\n\t"
      "v_accvgpr_read_b32 v11, a126\n\t"
      "v_accvgpr_read_b32 v26, a127\n\t"
      "v_accvgpr_mov_b32 a126, a190\n\t"
      "v_accvgpr_mov_b32 a127, a191\n\t"
      "v_accvgpr_write_b32 a190, v11\n\t"
      "v_accvgpr_write_b32 a191, v26\n\t"
      "v_accvgpr_read_b32 v11, a132\n\t"
      "v_accvgpr_read_b32 v26, a133\n\t"
      "v_accvgpr_mov_b32 a132, a196\n\t"
      "v_accvgpr_mov_b32 a133, a197\n\t"
      "v_accvgpr_write_b32 a196, v11\n\t"
      "v_accvgpr_write_b32 a197, v26\n\t"
      "v_accvgpr_read_b32 v11, a134\n\t"
      "v_accvgpr_read_b32 v26, a135\n\t"
      "v_accvgpr_mov_b32 a134, a198\n\t"
      "v_accvgpr_mov_b32 a135, a199\n\t"
      "v_accvgpr_write_b32 a198, v11\n\t"
      "v_accvgpr_write_b32 a199, v26\n\t"
      "v_accvgpr_read_b32 v11, a140\n\t"
      "v_accvgpr_read_b32 v26, a141\n\t"
      "v_accvgpr_mov_b32 a140, a204\n\t"
      "v_accvgpr_mov_b32 a141, a205\n\t"
      "v_accvgpr_write_b32 a204, v11\n\t"
      "v_accvgpr_write_b32 a205, v26\n\t"
      "v_accvgpr_read_b32 v11, a142\n\t"
      "v_accvgpr_read_b32 v26, a143\n\t"
      "v_accvgpr_mov_b32 a142, a206\n\t"
      "v_accvgpr_mov_b32 a143, a207\n\t"
      "v_accvgpr_write_b32 a206, v11\n\t"
      "v_accvgpr_write_b32 a207, v26\n\t"
      "v_accvgpr_read_b32 v11, a148\n\t"
      "v_accvgpr_read_b32 v26, a149\n\t"
      "v_accvgpr_mov_b32 a148, a212\n\t"
      "v_accvgpr_mov_b32 a149, a213\n\t"
      "v_accvgpr_write_b32 a212, v11\n\t"
      "v_accvgpr_write_b32 a213, v26\n\t"
      "v_accvgpr_read_b32 v11, a150\n\t"
      "v_accvgpr_read_b32 v26, a151\n\t"
      "v_accvgpr_mov_b32 a150, a214\n\t"
      "v_accvgpr_mov_b32 a151, a215\n\t"
      "v_accvgpr_write_b32 a214, v11\n\t"
      "v_accvgpr_write_b32 a215, v26\n\t"
      "v_accvgpr_read_b32 v11, a156\n\t"
      "v_accvgpr_read_b32 v26, a157\n\t"
      "v_accvgpr_mov_b32 a156, a220\n\t"
      "v_accvgpr_mov_b32 a157, a221\n\t"
      "v_accvgpr_write_b32 a220, v11\n\t"
      "v_accvgpr_write_b32 a221, v26\n\t"
      "v_accvgpr_read_b32 v11, a158\n\t"
      "v_accvgpr_read_b32 v26, a159\n\t"
      "v_accvgpr_mov_b32 a158, a222\n\t"
      "v_accvgpr_mov_b32 a159, a223\n\t"
      "v_accvgpr_write_b32 a222, v11\n\t"
      "v_accvgpr_write_b32 a223, v26\n\t"
      "v_accvgpr_read_b32 v11, a164\n\t"
      "v_accvgpr_read_b32 v26, a165\n\t"
      "v_accvgpr_mov_b32 a164, a228\n\t"
      "v_accvgpr_mov_b32 a165, a229\n\t"
      "v_accvgpr_write_b32 a228, v11\n\t"
      "v_accvgpr_write_b32 a229, v26\n\t"
      "v_accvgpr_read_b32 v11, a166\n\t"
      "v_accvgpr_read_b32 v26, a167\n\t"
      "v_accvgpr_mov_b32 a166, a230\n\t"
      "v_accvgpr_mov_b32 a167, a231\n\t"
      "v_accvgpr_write_b32 a230, v11\n\t"
      "v_accvgpr_write_b32 a231, v26\n\t"
      "v_accvgpr_read_b32 v11, a172\n\t"
      "v_accvgpr_read_b32 v26, a173\n\t"
      "v_accvgpr_mov_b32 a172, a236\n\t"
      "v_accvgpr_mov_b32 a173, a237\n\t"
      "v_accvgpr_write_b32 a236, v11\n\t"
      "v_accvgpr_write_b32 a237, v26\n\t"
      "v_accvgpr_read_b32 v11, a174\n\t"
      "v_accvgpr_read_b32 v26, a175\n\t"
      "v_accvgpr_mov_b32 a174, a238\n\t"
      "v_accvgpr_mov_b32 a175, a239\n\t"
      "v_accvgpr_write_b32 a238, v11\n\t"
      "v_accvgpr_write_b32 a239, v26\n\t"
      "v_accvgpr_read_b32 v11, a180\n\t"
      "v_accvgpr_read_b32 v26, a181\n\t"
      "v_accvgpr_mov_b32 a180, a244\n\t"
      "v_accvgpr_mov_b32 a181, a245\n\t"
      "v_accvgpr_write_b32 a244, v11\n\t"
      "v_accvgpr_write_b32 a245, v26\n\t"
      "v_accvgpr_read_b32 v11, a182\n\t"
      "v_accvgpr_read_b32 v26, a183\n\t"
      "v_accvgpr_mov_b32 a182, a246\n\t"
      "v_accvgpr_mov_b32 a183, a247\n\t"
      "v_accvgpr_write_b32 a246, v11\n\t"
      "v_accvgpr_write_b32 a247, v26\n\t"
      "s_waitcnt lgkmcnt(0)\n\t"
      "s_nop 1\n\t"
      "s_branch ALD1D_%=\n\t"
      "ALN12D_%=:\n\t"
      "s_add_u32 m0, s51, 8192\n\t"
      "s_xor_b32 s56, s57, 0x0\n\t"
      "buffer_load_dwordx4 v12, s[28:31], s56 offen lds\n\t"
      "buffer_load_dwordx4 v13, s[28:31], s56 offen offset:1024 lds\n\t"
      "buffer_load_dwordx4 v14, s[28:31], s56 offen offset:2048 lds\n\t"
      "buffer_load_dwordx4 v15, s[28:31], s56 offen offset:3072 lds\n\t"
      "s_add_u32 m0, s51, 12288\n\t"
      "s_xor_b32 s56, s57, 0x20000\n\t"
      "buffer_load_dwordx4 v12, s[28:31], s56 offen lds\n\t"
      "buffer_load_dwordx4 v13, s[28:31], s56 offen offset:1024 lds\n\t"
      "buffer_load_dwordx4 v14, s[28:31], s56 offen offset:2048 lds\n\t"
      "buffer_load_dwordx4 v15, s[28:31], s56 offen offset:3072 lds\n\t"
      "buffer_load_dwordx4 a[40:43], v7, s[32:35], s63 offen sc0 sc1\n\t"
      "buffer_load_dwordx4 a[44:47], v3, s[32:35], s63 offen sc0 sc1\n\t"
      "buffer_load_dwordx4 a[48:51], v7, s[32:35], s63 offen offset:2048 sc0 sc1\n\t"
      "buffer_load_dwordx4 a[52:55], v3, s[32:35], s63 offen offset:2048 sc0 sc1\n\t"
      "s_add_u32 s63, s63, 0x1000\n\t"
      "buffer_load_dwordx4 a[56:59], v7, s[32:35], s63 offen sc0 sc1\n\t"
      "buffer_load_dwordx4 a[60:63], v3, s[32:35], s63 offen sc0 sc1\n\t"
      "buffer_load_dwordx4 a[64:67], v7, s[32:35], s63 offen offset:2048 sc0 sc1\n\t"
      "buffer_load_dwordx4 a[68:71], v3, s[32:35], s63 offen offset:2048 sc0 sc1\n\t"
      "s_add_u32 s63, s63, 0x1000\n\t"
      "buffer_load_dwordx4 a[72:75], v8, s[32:35], s63 offen sc0 sc1\n\t"
      "buffer_load_dwordx4 a[76:79], v4, s[32:35], s63 offen sc0 sc1\n\t"
      "s_waitcnt vmcnt(10)\n\t"
      "ds_read_b64_tr_b8 a[120:121], v16 offset:8192\n\t"
      "ds_read_b64_tr_b8 a[122:123], v16 offset:9216\n\t"
      "ds_read_b64_tr_b8 a[128:129], v17 offset:8192\n\t"
      "ds_read_b64_tr_b8 a[130:131], v17 offset:9216\n\t"
      "ds_read_b64_tr_b8 a[136:137], v18 offset:8192\n\t"
      "ds_read_b64_tr_b8 a[138:139], v18 offset:9216\n\t"
      "ds_read_b64_tr_b8 a[144:145], v19 offset:8192\n\t"
      "ds_read_b64_tr_b8 a[146:147], v19 offset:9216\n\t"
      "ds_read_b64_tr_b8 a[152:153], v20 offset:8192\n\t"
      "ds_read_b64_tr_b8 a[154:155], v20 offset:9216\n\t"
      "ds_read_b64_tr_b8 a[160:161], v21 offset:8192\n\t"
      "ds_read_b64_tr_b8 a[162:163], v21 offset:9216\n\t"
      "ds_read_b64_tr_b8 a[168:169], v22 offset:8192\n\t"
      "ds_read_b64_tr_b8 a[170:171], v22 offset:9216\n\t"
      "ds_read_b64_tr_b8 a[176:177], v23 offset:8192\n\t"
      "ds_read_b64_tr_b8 a[178:179], v23 offset:9216\n\t"
      "s_waitcnt lgkmcnt(0)\n\t"
      "s_add_u32 m0, s51, 8192\n\t"
      "s_xor_b32 s56, s57, 0x40000\n\t"
      "buffer_load_dwordx4 v12, s[28:31], s56 offen lds\n\t"
      "buffer_load_dwordx4 v13, s[28:31], s56 offen offset:1024 lds\n\t"
      "buffer_load_dwordx4 v14, s[28:31], s56 offen offset:2048 lds\n\t"
      "buffer_load_dwordx4 v15, s[28:31], s56 offen offset:3072 lds\n\t"
      "s_add_u32 m0, s51, 12288\n\t"
      "s_xor_b32 s56, s57, 0x60000\n\t"
      "buffer_load_dwordx4 v12, s[28:31], s56 offen lds\n\t"
      "buffer_load_dwordx4 v13, s[28:31], s56 offen offset:1024 lds\n\t"
      "buffer_load_dwordx4 v14, s[28:31], s56 offen offset:2048 lds\n\t"
      "buffer_load_dwordx4 v15, s[28:31], s56 offen offset:3072 lds\n\t"
      "s_waitcnt vmcnt(0)\n\t"
      "ds_read_b64_tr_b8 a[124:125], v16 offset:8192\n\t"
      "ds_read_b64_tr_b8 a[126:127], v16 offset:9216\n\t"
      "ds_read_b64_tr_b8 a[132:133], v17 offset:8192\n\t"
      "ds_read_b64_tr_b8 a[134:135], v17 offset:9216\n\t"
      "ds_read_b64_tr_b8 a[140:141], v18 offset:8192\n\t"
      "ds_read_b64_tr_b8 a[142:143], v18 offset:9216\n\t"
      "ds_read_b64_tr_b8 a[148:149], v19 offset:8192\n\t"
      "ds_read_b64_tr_b8 a[150:151], v19 offset:9216\n\t"
      "ds_read_b64_tr_b8 a[156:157], v20 offset:8192\n\t"
      "ds_read_b64_tr_b8 a[158:159], v20 offset:9216\n\t"
      "ds_read_b64_tr_b8 a[164:165], v21 offset:8192\n\t"
      "ds_read_b64_tr_b8 a[166:167], v21 offset:9216\n\t"
      "ds_read_b64_tr_b8 a[172:173], v22 offset:8192\n\t"
      "ds_read_b64_tr_b8 a[174:175], v22 offset:9216\n\t"
      "ds_read_b64_tr_b8 a[180:181], v23 offset:8192\n\t"
      "ds_read_b64_tr_b8 a[182:183], v23 offset:9216\n\t"
      "s_waitcnt lgkmcnt(0)\n\t"
      "s_branch ALD1D_%=\n\t"
      "ALT2D_%=:\n\t"
      "s_mov_b32 s21, 20544\n\t"
      "s_mov_b32 s58, 1\n\t"
      "v_mfma_f32_16x16x128_f8f6f4 v[108:111], a[56:63], a[144:151], v[108:111]\n\t"
      "v_mfma_f32_16x16x128_f8f6f4 v[112:115], a[56:63], a[152:159], v[112:115]\n\t"
      "s_call_b64 s[66:67], SETC_%=\n\t"
      "v_mfma_f32_16x16x128_f8f6f4 v[116:119], a[56:63], a[160:167], v[116:119]\n\t"
      "v_mfma_f32_16x16x128_f8f6f4 v[120:123], a[56:63], a[168:175], v[120:123]\n\t"
      "v_mfma_f32_16x16x128_f8f6f4 v[124:127], a[56:63], a[176:183], v[124:127]\n\t"
      "v_mfma_f32_16x16x128_f8f6f4 v[128:131], a[64:71], a[120:127], v[128:131]\n\t"
      "v_mfma_f32_16x16x128_f8f6f4 v[132:135], a[64:71], a[128:135], v[132:135]\n\t"
      "v_mfma_f32_16x16x128_f8f6f4 v[136:139], a[64:71], a[136:143], v[136:139]\n\t"
      "v_mfma_f32_16x16x128_f8f6f4 v[140:143], a[64:71], a[144:151], v[140:143]\n\t"
      "v_mfma_f32_16x16x128_f8f6f4 v[144:147], a[64:71], a[152:159], v[144:147]\n\t"
      "v_mfma_f32_16x16x128_f8f6f4 v[148:151], a[64:71], a[160:167], v[148:151]\n\t"
      "v_mfma_f32_16x16x128_f8f6f4 v[152:155], a[64:71], a[168:175], v[152:155]\n\t"
      "v_mfma_f32_16x16x128_f8f6f4 v[156:159], a[64:71], a[176:183], v[156:159]\n\t"
      "v_mfma_f32_16x16x128_f8f6f4 v[160:163], a[72:79], a[120:127], v[160:163]\n\t"
      "v_mfma_f32_16x16x128_f8f6f4 v[164:167], a[72:79], a[128:135], v[164:167]\n\t"
      "v_mfma_f32_16x16x128_f8f6f4 v[168:171], a[72:79], a[136:143], v[168:171]\n\t"
      "v_mfma_f32_16x16x128_f8f6f4 v[172:175], a[72:79], a[144:151], v[172:175]\n\t"
      "v_mfma_f32_16x16x128_f8f6f4 v[176:179], a[72:79], a[152:159], v[176:179]\n\t"
      "v_mfma_f32_16x16x128_f8f6f4 v[180:183], a[72:79], a[160:167], v[180:183]\n\t"
      "v_mfma_f32_16x16x128_f8f6f4 v[184:187], a[72:79], a[168:175], v[184:187]\n\t"
      "v_mfma_f32_16x16x128_f8f6f4 v[188:191], a[72:79], a[176:183], v[188:191]\n\t"
      "s_call_b64 s[66:67], SPIN1_%=\n\t"
      "s_cmp_lg_u32 s21, 0\n\t"
      "s_cbranch_scc1 ALS2D_%=\n\t"
      "buffer_load_dwordx4 a[80:83], v7, s[32:35], s63 offen sc0 sc1\n\t"
      "buffer_load_dwordx4 a[84:87], v3, s[32:35], s63 offen sc0 sc1\n\t"
      "buffer_load_dwordx4 a[88:91], v7, s[32:35], s63 offen offset:2048 sc0 sc1\n\t"
      "buffer_load_dwordx4 a[92:95], v3, s[32:35], s63 offen offset:2048 sc0 sc1\n\t"
      "s_and_b32 vcc_lo, vcc_lo, 2\n\t"
      "s_cbranch_scc0 ALB2D_%=\n\t"
      "ALC2D_%=:\n\t"
      "s_add_u32 s63, s63, 0x1000\n\t"
      "buffer_load_dwordx4 a[96:99], v7, s[32:35], s63 offen sc0 sc1\n\t"
      "buffer_load_dwordx4 a[100:103], v3, s[32:35], s63 offen sc0 sc1\n\t"
      "buffer_load_dwordx4 a[104:107], v7, s[32:35], s63 offen offset:2048 sc0 sc1\n\t"
      "buffer_load_dwordx4 a[108:111], v3, s[32:35], s63 offen offset:2048 sc0 sc1\n\t"
      "s_add_u32 s63, s63, 0x1000\n\t"
      "buffer_load_dwordx4 a[112:115], v8, s[32:35], s63 offen sc0 sc1\n\t"
      "buffer_load_dwordx4 a[116:119], v4, s[32:35], s63 offen sc0 sc1\n\t"
      "ALD2D_%=:\n\t"
      "s_branch JOIN2D_%=\n\t"
      "ALB2D_%=:\n\t"
      "s_call_b64 s[66:67], SPINB_%=\n\t"
      "s_branch ALC2D_%=\n\t"
      "ALS2D_%=:\n\t"
      "s_mov_b32 s56, 16\n\t"
      "s_call_b64 s[66:67], SWAP_%=\n\t"
      "s_waitcnt lgkmcnt(0)\n\t"
      "s_bitcmp0_b32 s21, 6\n\t"
      "s_cbranch_scc1 ALN23D_%=\n\t"
      "buffer_load_dwordx4 a[80:83], v7, s[32:35], s63 offen sc0 sc1\n\t"
      "buffer_load_dwordx4 a[84:87], v3, s[32:35], s63 offen sc0 sc1\n\t"
      "buffer_load_dwordx4 a[88:91], v7, s[32:35], s63 offen offset:2048 sc0 sc1\n\t"
      "buffer_load_dwordx4 a[92:95], v3, s[32:35], s63 offen offset:2048 sc0 sc1\n\t"
      "s_add_u32 s63, s63, 0x1000\n\t"
      "buffer_load_dwordx4 a[96:99], v7, s[32:35], s63 offen sc0 sc1\n\t"
      "buffer_load_dwordx4 a[100:103], v3, s[32:35], s63 offen sc0 sc1\n\t"
      "buffer_load_dwordx4 a[104:107], v7, s[32:35], s63 offen offset:2048 sc0 sc1\n\t"
      "buffer_load_dwordx4 a[108:111], v3, s[32:35], s63 offen offset:2048 sc0 sc1\n\t"
      "s_add_u32 s63, s63, 0x1000\n\t"
      "buffer_load_dwordx4 a[112:115], v8, s[32:35], s63 offen sc0 sc1\n\t"
      "buffer_load_dwordx4 a[116:119], v4, s[32:35], s63 offen sc0 sc1\n\t"
      "v_accvgpr_read_b32 v11, a184\n\t"
      "v_accvgpr_read_b32 v26, a185\n\t"
      "v_accvgpr_write_b32 a184, v192\n\t"
      "v_accvgpr_write_b32 a185, v193\n\t"
      "v_mov_b32 v192, v11\n\t"
      "v_mov_b32 v193, v26\n\t"
      "v_accvgpr_read_b32 v11, a186\n\t"
      "v_accvgpr_read_b32 v26, a187\n\t"
      "v_accvgpr_write_b32 a186, v194\n\t"
      "v_accvgpr_write_b32 a187, v195\n\t"
      "v_mov_b32 v194, v11\n\t"
      "v_mov_b32 v195, v26\n\t"
      "v_accvgpr_read_b32 v11, a192\n\t"
      "v_accvgpr_read_b32 v26, a193\n\t"
      "v_accvgpr_write_b32 a192, v200\n\t"
      "v_accvgpr_write_b32 a193, v201\n\t"
      "v_mov_b32 v200, v11\n\t"
      "v_mov_b32 v201, v26\n\t"
      "v_accvgpr_read_b32 v11, a194\n\t"
      "v_accvgpr_read_b32 v26, a195\n\t"
      "v_accvgpr_write_b32 a194, v202\n\t"
      "v_accvgpr_write_b32 a195, v203\n\t"
      "v_mov_b32 v202, v11\n\t"
      "v_mov_b32 v203, v26\n\t"
      "v_accvgpr_read_b32 v11, a200\n\t"
      "v_accvgpr_read_b32 v26, a201\n\t"
      "v_accvgpr_write_b32 a200, v208\n\t"
      "v_accvgpr_write_b32 a201, v209\n\t"
      "v_mov_b32 v208, v11\n\t"
      "v_mov_b32 v209, v26\n\t"
      "v_accvgpr_read_b32 v11, a202\n\t"
      "v_accvgpr_read_b32 v26, a203\n\t"
      "v_accvgpr_write_b32 a202, v210\n\t"
      "v_accvgpr_write_b32 a203, v211\n\t"
      "v_mov_b32 v210, v11\n\t"
      "v_mov_b32 v211, v26\n\t"
      "v_accvgpr_read_b32 v11, a208\n\t"
      "v_accvgpr_read_b32 v26, a209\n\t"
      "v_accvgpr_write_b32 a208, v216\n\t"
      "v_accvgpr_write_b32 a209, v217\n\t"
      "v_mov_b32 v216, v11\n\t"
      "v_mov_b32 v217, v26\n\t"
      "v_accvgpr_read_b32 v11, a210\n\t"
      "v_accvgpr_read_b32 v26, a211\n\t"
      "v_accvgpr_write_b32 a210, v218\n\t"
      "v_accvgpr_write_b32 a211, v219\n\t"
      "v_mov_b32 v218, v11\n\t"
      "v_mov_b32 v219, v26\n\t"
      "v_accvgpr_read_b32 v11, a216\n\t"
      "v_accvgpr_read_b32 v26, a217\n\t"
      "v_accvgpr_write_b32 a216, v224\n\t"
      "v_accvgpr_write_b32 a217, v225\n\t"
      "v_mov_b32 v224, v11\n\t"
      "v_mov_b32 v225, v26\n\t"
      "v_accvgpr_read_b32 v11, a218\n\t"
      "v_accvgpr_read_b32 v26, a219\n\t"
      "v_accvgpr_write_b32 a218, v226\n\t"
      "v_accvgpr_write_b32 a219, v227\n\t"
      "v_mov_b32 v226, v11\n\t"
      "v_mov_b32 v227, v26\n\t"
      "v_accvgpr_read_b32 v11, a224\n\t"
      "v_accvgpr_read_b32 v26, a225\n\t"
      "v_accvgpr_write_b32 a224, v232\n\t"
      "v_accvgpr_write_b32 a225, v233\n\t"
      "v_mov_b32 v232, v11\n\t"
      "v_mov_b32 v233, v26\n\t"
      "v_accvgpr_read_b32 v11, a226\n\t"
      "v_accvgpr_read_b32 v26, a227\n\t"
      "v_accvgpr_write_b32 a226, v234\n\t"
      "v_accvgpr_write_b32 a227, v235\n\t"
      "v_mov_b32 v234, v11\n\t"
      "v_mov_b32 v235, v26\n\t"
      "v_accvgpr_read_b32 v11, a232\n\t"
      "v_accvgpr_read_b32 v26, a233\n\t"
      "v_accvgpr_write_b32 a232, v240\n\t"
      "v_accvgpr_write_b32 a233, v241\n\t"
      "v_mov_b32 v240, v11\n\t"
      "v_mov_b32 v241, v26\n\t"
      "v_accvgpr_read_b32 v11, a234\n\t"
      "v_accvgpr_read_b32 v26, a235\n\t"
      "v_accvgpr_write_b32 a234, v242\n\t"
      "v_accvgpr_write_b32 a235, v243\n\t"
      "v_mov_b32 v242, v11\n\t"
      "v_mov_b32 v243, v26\n\t"
      "v_accvgpr_read_b32 v11, a240\n\t"
      "v_accvgpr_read_b32 v26, a241\n\t"
      "v_accvgpr_write_b32 a240, v248\n\t"
      "v_accvgpr_write_b32 a241, v249\n\t"
      "v_mov_b32 v248, v11\n\t"
      "v_mov_b32 v249, v26\n\t"
      "v_accvgpr_read_b32 v11, a242\n\t"
      "v_accvgpr_read_b32 v26, a243\n\t"
      "v_accvgpr_write_b32 a242, v250\n\t"
      "v_accvgpr_write_b32 a243, v251\n\t"
      "v_mov_b32 v250, v11\n\t"
      "v_mov_b32 v251, v26\n\t"
      "v_accvgpr_read_b32 v11, a188\n\t"
      "v_accvgpr_read_b32 v26, a189\n\t"
      "v_accvgpr_write_b32 a188, v196\n\t"
      "v_accvgpr_write_b32 a189, v197\n\t"
      "v_mov_b32 v196, v11\n\t"
      "v_mov_b32 v197, v26\n\t"
      "v_accvgpr_read_b32 v11, a190\n\t"
      "v_accvgpr_read_b32 v26, a191\n\t"
      "v_accvgpr_write_b32 a190, v198\n\t"
      "v_accvgpr_write_b32 a191, v199\n\t"
      "v_mov_b32 v198, v11\n\t"
      "v_mov_b32 v199, v26\n\t"
      "v_accvgpr_read_b32 v11, a196\n\t"
      "v_accvgpr_read_b32 v26, a197\n\t"
      "v_accvgpr_write_b32 a196, v204\n\t"
      "v_accvgpr_write_b32 a197, v205\n\t"
      "v_mov_b32 v204, v11\n\t"
      "v_mov_b32 v205, v26\n\t"
      "v_accvgpr_read_b32 v11, a198\n\t"
      "v_accvgpr_read_b32 v26, a199\n\t"
      "v_accvgpr_write_b32 a198, v206\n\t"
      "v_accvgpr_write_b32 a199, v207\n\t"
      "v_mov_b32 v206, v11\n\t"
      "v_mov_b32 v207, v26\n\t"
      "v_accvgpr_read_b32 v11, a204\n\t"
      "v_accvgpr_read_b32 v26, a205\n\t"
      "v_accvgpr_write_b32 a204, v212\n\t"
      "v_accvgpr_write_b32 a205, v213\n\t"
      "v_mov_b32 v212, v11\n\t"
      "v_mov_b32 v213, v26\n\t"
      "v_accvgpr_read_b32 v11, a206\n\t"
      "v_accvgpr_read_b32 v26, a207\n\t"
      "v_accvgpr_write_b32 a206, v214\n\t"
      "v_accvgpr_write_b32 a207, v215\n\t"
      "v_mov_b32 v214, v11\n\t"
      "v_mov_b32 v215, v26\n\t"
      "v_accvgpr_read_b32 v11, a212\n\t"
      "v_accvgpr_read_b32 v26, a213\n\t"
      "v_accvgpr_write_b32 a212, v220\n\t"
      "v_accvgpr_write_b32 a213, v221\n\t"
      "v_mov_b32 v220, v11\n\t"
      "v_mov_b32 v221, v26\n\t"
      "v_accvgpr_read_b32 v11, a214\n\t"
      "v_accvgpr_read_b32 v26, a215\n\t"
      "v_accvgpr_write_b32 a214, v222\n\t"
      "v_accvgpr_write_b32 a215, v223\n\t"
      "v_mov_b32 v222, v11\n\t"
      "v_mov_b32 v223, v26\n\t"
      "v_accvgpr_read_b32 v11, a220\n\t"
      "v_accvgpr_read_b32 v26, a221\n\t"
      "v_accvgpr_write_b32 a220, v228\n\t"
      "v_accvgpr_write_b32 a221, v229\n\t"
      "v_mov_b32 v228, v11\n\t"
      "v_mov_b32 v229, v26\n\t"
      "v_accvgpr_read_b32 v11, a222\n\t"
      "v_accvgpr_read_b32 v26, a223\n\t"
      "v_accvgpr_write_b32 a222, v230\n\t"
      "v_accvgpr_write_b32 a223, v231\n\t"
      "v_mov_b32 v230, v11\n\t"
      "v_mov_b32 v231, v26\n\t"
      "v_accvgpr_read_b32 v11, a228\n\t"
      "v_accvgpr_read_b32 v26, a229\n\t"
      "v_accvgpr_write_b32 a228, v236\n\t"
      "v_accvgpr_write_b32 a229, v237\n\t"
      "v_mov_b32 v236, v11\n\t"
      "v_mov_b32 v237, v26\n\t"
      "v_accvgpr_read_b32 v11, a230\n\t"
      "v_accvgpr_read_b32 v26, a231\n\t"
      "v_accvgpr_write_b32 a230, v238\n\t"
      "v_accvgpr_write_b32 a231, v239\n\t"
      "v_mov_b32 v238, v11\n\t"
      "v_mov_b32 v239, v26\n\t"
      "v_accvgpr_read_b32 v11, a236\n\t"
      "v_accvgpr_read_b32 v26, a237\n\t"
      "v_accvgpr_write_b32 a236, v244\n\t"
      "v_accvgpr_write_b32 a237, v245\n\t"
      "v_mov_b32 v244, v11\n\t"
      "v_mov_b32 v245, v26\n\t"
      "v_accvgpr_read_b32 v11, a238\n\t"
      "v_accvgpr_read_b32 v26, a239\n\t"
      "v_accvgpr_write_b32 a238, v246\n\t"
      "v_accvgpr_write_b32 a239, v247\n\t"
      "v_mov_b32 v246, v11\n\t"
      "v_mov_b32 v247, v26\n\t"
      "v_accvgpr_read_b32 v11, a244\n\t"
      "v_accvgpr_read_b32 v26, a245\n\t"
      "v_accvgpr_write_b32 a244, v252\n\t"
      "v_accvgpr_write_b32 a245, v253\n\t"
      "v_mov_b32 v252, v11\n\t"
      "v_mov_b32 v253, v26\n\t"
      "v_accvgpr_read_b32 v11, a246\n\t"
      "v_accvgpr_read_b32 v26, a247\n\t"
      "v_accvgpr_write_b32 a246, v254\n\t"
      "v_accvgpr_write_b32 a247, v255\n\t"
      "v_mov_b32 v254, v11\n\t"
      "v_mov_b32 v255, v26\n\t"
      "s_waitcnt lgkmcnt(0)\n\t"
      "s_nop 1\n\t"
      "s_branch ALD2D_%=\n\t"
      "ALN23D_%=:\n\t"
      "s_add_u32 m0, s51, 8192\n\t"
      "s_xor_b32 s56, s57, 0x0\n\t"
      "buffer_load_dwordx4 v12, s[28:31], s56 offen lds\n\t"
      "buffer_load_dwordx4 v13, s[28:31], s56 offen offset:1024 lds\n\t"
      "buffer_load_dwordx4 v14, s[28:31], s56 offen offset:2048 lds\n\t"
      "buffer_load_dwordx4 v15, s[28:31], s56 offen offset:3072 lds\n\t"
      "s_add_u32 m0, s51, 12288\n\t"
      "s_xor_b32 s56, s57, 0x20000\n\t"
      "buffer_load_dwordx4 v12, s[28:31], s56 offen lds\n\t"
      "buffer_load_dwordx4 v13, s[28:31], s56 offen offset:1024 lds\n\t"
      "buffer_load_dwordx4 v14, s[28:31], s56 offen offset:2048 lds\n\t"
      "buffer_load_dwordx4 v15, s[28:31], s56 offen offset:3072 lds\n\t"
      "s_add_u32 m0, s51, 16384\n\t"
      "s_xor_b32 s56, s57, 0x40000\n\t"
      "buffer_load_dwordx4 v12, s[28:31], s56 offen lds\n\t"
      "buffer_load_dwordx4 v13, s[28:31], s56 offen offset:1024 lds\n\t"
      "buffer_load_dwordx4 v14, s[28:31], s56 offen offset:2048 lds\n\t"
      "buffer_load_dwordx4 v15, s[28:31], s56 offen offset:3072 lds\n\t"
      "s_add_u32 m0, s51, 20480\n\t"
      "s_xor_b32 s56, s57, 0x60000\n\t"
      "buffer_load_dwordx4 v12, s[28:31], s56 offen lds\n\t"
      "buffer_load_dwordx4 v13, s[28:31], s56 offen offset:1024 lds\n\t"
      "buffer_load_dwordx4 v14, s[28:31], s56 offen offset:2048 lds\n\t"
      "buffer_load_dwordx4 v15, s[28:31], s56 offen offset:3072 lds\n\t"
      "buffer_load_dwordx4 a[80:83], v7, s[32:35], s63 offen sc0 sc1\n\t"
      "buffer_load_dwordx4 a[84:87], v3, s[32:35], s63 offen sc0 sc1\n\t"
      "buffer_load_dwordx4 a[88:91], v7, s[32:35], s63 offen offset:2048 sc0 sc1\n\t"
      "buffer_load_dwordx4 a[92:95], v3, s[32:35], s63 offen offset:2048 sc0 sc1\n\t"
      "s_add_u32 s63, s63, 0x1000\n\t"
      "buffer_load_dwordx4 a[96:99], v7, s[32:35], s63 offen sc0 sc1\n\t"
      "buffer_load_dwordx4 a[100:103], v3, s[32:35], s63 offen sc0 sc1\n\t"
      "buffer_load_dwordx4 a[104:107], v7, s[32:35], s63 offen offset:2048 sc0 sc1\n\t"
      "buffer_load_dwordx4 a[108:111], v3, s[32:35], s63 offen offset:2048 sc0 sc1\n\t"
      "s_add_u32 s63, s63, 0x1000\n\t"
      "buffer_load_dwordx4 a[112:115], v8, s[32:35], s63 offen sc0 sc1\n\t"
      "buffer_load_dwordx4 a[116:119], v4, s[32:35], s63 offen sc0 sc1\n\t"
      "s_waitcnt vmcnt(18)\n\t"
      "ds_read_b64_tr_b8 a[184:185], v16 offset:8192\n\t"
      "ds_read_b64_tr_b8 a[186:187], v16 offset:9216\n\t"
      "ds_read_b64_tr_b8 a[192:193], v17 offset:8192\n\t"
      "ds_read_b64_tr_b8 a[194:195], v17 offset:9216\n\t"
      "ds_read_b64_tr_b8 a[200:201], v18 offset:8192\n\t"
      "ds_read_b64_tr_b8 a[202:203], v18 offset:9216\n\t"
      "ds_read_b64_tr_b8 a[208:209], v19 offset:8192\n\t"
      "ds_read_b64_tr_b8 a[210:211], v19 offset:9216\n\t"
      "ds_read_b64_tr_b8 a[216:217], v20 offset:8192\n\t"
      "ds_read_b64_tr_b8 a[218:219], v20 offset:9216\n\t"
      "ds_read_b64_tr_b8 a[224:225], v21 offset:8192\n\t"
      "ds_read_b64_tr_b8 a[226:227], v21 offset:9216\n\t"
      "ds_read_b64_tr_b8 a[232:233], v22 offset:8192\n\t"
      "ds_read_b64_tr_b8 a[234:235], v22 offset:9216\n\t"
      "ds_read_b64_tr_b8 a[240:241], v23 offset:8192\n\t"
      "ds_read_b64_tr_b8 a[242:243], v23 offset:9216\n\t"
      "s_waitcnt vmcnt(10)\n\t"
      "ds_read_b64_tr_b8 a[188:189], v16 offset:16384\n\t"
      "ds_read_b64_tr_b8 a[190:191], v16 offset:17408\n\t"
      "ds_read_b64_tr_b8 a[196:197], v17 offset:16384\n\t"
      "ds_read_b64_tr_b8 a[198:199], v17 offset:17408\n\t"
      "ds_read_b64_tr_b8 a[204:205], v18 offset:16384\n\t"
      "ds_read_b64_tr_b8 a[206:207], v18 offset:17408\n\t"
      "ds_read_b64_tr_b8 a[212:213], v19 offset:16384\n\t"
      "ds_read_b64_tr_b8 a[214:215], v19 offset:17408\n\t"
      "ds_read_b64_tr_b8 a[220:221], v20 offset:16384\n\t"
      "ds_read_b64_tr_b8 a[222:223], v20 offset:17408\n\t"
      "ds_read_b64_tr_b8 a[228:229], v21 offset:16384\n\t"
      "ds_read_b64_tr_b8 a[230:231], v21 offset:17408\n\t"
      "ds_read_b64_tr_b8 a[236:237], v22 offset:16384\n\t"
      "ds_read_b64_tr_b8 a[238:239], v22 offset:17408\n\t"
      "ds_read_b64_tr_b8 a[244:245], v23 offset:16384\n\t"
      "ds_read_b64_tr_b8 a[246:247], v23 offset:17408\n\t"
      "s_waitcnt lgkmcnt(0)\n\t"
      "s_branch ALD2D_%=\n\t"
      "ALT3D_%=:\n\t"
      "s_mov_b32 s21, 20736\n\t"
      "s_mov_b32 s58, 1\n\t"
      "v_mfma_f32_16x16x128_f8f6f4 v[108:111], a[96:103], a[208:215], v[108:111]\n\t"
      "v_mfma_f32_16x16x128_f8f6f4 v[112:115], a[96:103], a[216:223], v[112:115]\n\t"
      "s_call_b64 s[66:67], SETC_%=\n\t"
      "v_mfma_f32_16x16x128_f8f6f4 v[116:119], a[96:103], a[224:231], v[116:119]\n\t"
      "v_mfma_f32_16x16x128_f8f6f4 v[120:123], a[96:103], a[232:239], v[120:123]\n\t"
      "v_mfma_f32_16x16x128_f8f6f4 v[124:127], a[96:103], a[240:247], v[124:127]\n\t"
      "v_mfma_f32_16x16x128_f8f6f4 v[128:131], a[104:111], a[184:191], v[128:131]\n\t"
      "v_mfma_f32_16x16x128_f8f6f4 v[132:135], a[104:111], a[192:199], v[132:135]\n\t"
      "v_mfma_f32_16x16x128_f8f6f4 v[136:139], a[104:111], a[200:207], v[136:139]\n\t"
      "v_mfma_f32_16x16x128_f8f6f4 v[140:143], a[104:111], a[208:215], v[140:143]\n\t"
      "v_mfma_f32_16x16x128_f8f6f4 v[144:147], a[104:111], a[216:223], v[144:147]\n\t"
      "v_mfma_f32_16x16x128_f8f6f4 v[148:151], a[104:111], a[224:231], v[148:151]\n\t"
      "v_mfma_f32_16x16x128_f8f6f4 v[152:155], a[104:111], a[232:239], v[152:155]\n\t"
      "v_mfma_f32_16x16x128_f8f6f4 v[156:159], a[104:111], a[240:247], v[156:159]\n\t"
      "v_mfma_f32_16x16x128_f8f6f4 v[160:163], a[112:119], a[184:191], v[160:163]\n\t"
      "v_mfma_f32_16x16x128_f8f6f4 v[164:167], a[112:119], a[192:199], v[164:167]\n\t"
      "v_mfma_f32_16x16x128_f8f6f4 v[168:171], a[112:119], a[200:207], v[168:171]\n\t"
      "v_mfma_f32_16x16x128_f8f6f4 v[172:175], a[112:119], a[208:215], v[172:175]\n\t"
      "v_mfma_f32_16x16x128_f8f6f4 v[176:179], a[112:119], a[216:223], v[176:179]\n\t"
      "v_mfma_f32_16x16x128_f8f6f4 v[180:183], a[112:119], a[224:231], v[180:183]\n\t"
      "v_mfma_f32_16x16x128_f8f6f4 v[184:187], a[112:119], a[232:239], v[184:187]\n\t"
      "v_mfma_f32_16x16x128_f8f6f4 v[188:191], a[112:119], a[240:247], v[188:191]\n\t"
      "s_call_b64 s[66:67], SPIN1_%=\n\t"
      "s_cmp_lg_u32 s21, 0\n\t"
      "s_cbranch_scc1 ALS3D_%=\n\t"
      "buffer_load_dwordx4 a[0:3], v7, s[32:35], s63 offen sc0 sc1\n\t"
      "buffer_load_dwordx4 a[4:7], v3, s[32:35], s63 offen sc0 sc1\n\t"
      "buffer_load_dwordx4 a[8:11], v7, s[32:35], s63 offen offset:2048 sc0 sc1\n\t"
      "buffer_load_dwordx4 a[12:15], v3, s[32:35], s63 offen offset:2048 sc0 sc1\n\t"
      "s_and_b32 vcc_lo, vcc_lo, 2\n\t"
      "s_cbranch_scc0 ALB3D_%=\n\t"
      "ALC3D_%=:\n\t"
      "s_add_u32 s63, s63, 0x1000\n\t"
      "buffer_load_dwordx4 a[16:19], v7, s[32:35], s63 offen sc0 sc1\n\t"
      "buffer_load_dwordx4 a[20:23], v3, s[32:35], s63 offen sc0 sc1\n\t"
      "buffer_load_dwordx4 a[24:27], v7, s[32:35], s63 offen offset:2048 sc0 sc1\n\t"
      "buffer_load_dwordx4 a[28:31], v3, s[32:35], s63 offen offset:2048 sc0 sc1\n\t"
      "s_add_u32 s63, s63, 0x1000\n\t"
      "buffer_load_dwordx4 a[32:35], v8, s[32:35], s63 offen sc0 sc1\n\t"
      "buffer_load_dwordx4 a[36:39], v4, s[32:35], s63 offen sc0 sc1\n\t"
      "ALD3D_%=:\n\t"
      "s_branch JOIN3D_%=\n\t"
      "ALB3D_%=:\n\t"
      "s_call_b64 s[66:67], SPINB_%=\n\t"
      "s_branch ALC3D_%=\n\t"
      "ALS3D_%=:\n\t"
      "s_mov_b32 s56, 24\n\t"
      "s_call_b64 s[66:67], SWAP_%=\n\t"
      "s_waitcnt lgkmcnt(0)\n\t"
      "s_bitcmp0_b32 s21, 8\n\t"
      "s_cbranch_scc1 ALN34D_%=\n\t"
      "buffer_load_dwordx4 a[0:3], v7, s[32:35], s63 offen sc0 sc1\n\t"
      "buffer_load_dwordx4 a[4:7], v3, s[32:35], s63 offen sc0 sc1\n\t"
      "buffer_load_dwordx4 a[8:11], v7, s[32:35], s63 offen offset:2048 sc0 sc1\n\t"
      "buffer_load_dwordx4 a[12:15], v3, s[32:35], s63 offen offset:2048 sc0 sc1\n\t"
      "s_add_u32 s63, s63, 0x1000\n\t"
      "buffer_load_dwordx4 a[16:19], v7, s[32:35], s63 offen sc0 sc1\n\t"
      "buffer_load_dwordx4 a[20:23], v3, s[32:35], s63 offen sc0 sc1\n\t"
      "buffer_load_dwordx4 a[24:27], v7, s[32:35], s63 offen offset:2048 sc0 sc1\n\t"
      "buffer_load_dwordx4 a[28:31], v3, s[32:35], s63 offen offset:2048 sc0 sc1\n\t"
      "s_add_u32 s63, s63, 0x1000\n\t"
      "buffer_load_dwordx4 a[32:35], v8, s[32:35], s63 offen sc0 sc1\n\t"
      "buffer_load_dwordx4 a[36:39], v4, s[32:35], s63 offen sc0 sc1\n\t"
      "v_mov_b32 v11, v192\n\t"
      "v_mov_b32 v26, v193\n\t"
      "v_accvgpr_read_b32 v192, a120\n\t"
      "v_accvgpr_read_b32 v193, a121\n\t"
      "v_accvgpr_write_b32 a120, v11\n\t"
      "v_accvgpr_write_b32 a121, v26\n\t"
      "v_mov_b32 v11, v194\n\t"
      "v_mov_b32 v26, v195\n\t"
      "v_accvgpr_read_b32 v194, a122\n\t"
      "v_accvgpr_read_b32 v195, a123\n\t"
      "v_accvgpr_write_b32 a122, v11\n\t"
      "v_accvgpr_write_b32 a123, v26\n\t"
      "v_mov_b32 v11, v200\n\t"
      "v_mov_b32 v26, v201\n\t"
      "v_accvgpr_read_b32 v200, a128\n\t"
      "v_accvgpr_read_b32 v201, a129\n\t"
      "v_accvgpr_write_b32 a128, v11\n\t"
      "v_accvgpr_write_b32 a129, v26\n\t"
      "v_mov_b32 v11, v202\n\t"
      "v_mov_b32 v26, v203\n\t"
      "v_accvgpr_read_b32 v202, a130\n\t"
      "v_accvgpr_read_b32 v203, a131\n\t"
      "v_accvgpr_write_b32 a130, v11\n\t"
      "v_accvgpr_write_b32 a131, v26\n\t"
      "v_mov_b32 v11, v208\n\t"
      "v_mov_b32 v26, v209\n\t"
      "v_accvgpr_read_b32 v208, a136\n\t"
      "v_accvgpr_read_b32 v209, a137\n\t"
      "v_accvgpr_write_b32 a136, v11\n\t"
      "v_accvgpr_write_b32 a137, v26\n\t"
      "v_mov_b32 v11, v210\n\t"
      "v_mov_b32 v26, v211\n\t"
      "v_accvgpr_read_b32 v210, a138\n\t"
      "v_accvgpr_read_b32 v211, a139\n\t"
      "v_accvgpr_write_b32 a138, v11\n\t"
      "v_accvgpr_write_b32 a139, v26\n\t"
      "v_mov_b32 v11, v216\n\t"
      "v_mov_b32 v26, v217\n\t"
      "v_accvgpr_read_b32 v216, a144\n\t"
      "v_accvgpr_read_b32 v217, a145\n\t"
      "v_accvgpr_write_b32 a144, v11\n\t"
      "v_accvgpr_write_b32 a145, v26\n\t"
      "v_mov_b32 v11, v218\n\t"
      "v_mov_b32 v26, v219\n\t"
      "v_accvgpr_read_b32 v218, a146\n\t"
      "v_accvgpr_read_b32 v219, a147\n\t"
      "v_accvgpr_write_b32 a146, v11\n\t"
      "v_accvgpr_write_b32 a147, v26\n\t"
      "v_mov_b32 v11, v224\n\t"
      "v_mov_b32 v26, v225\n\t"
      "v_accvgpr_read_b32 v224, a152\n\t"
      "v_accvgpr_read_b32 v225, a153\n\t"
      "v_accvgpr_write_b32 a152, v11\n\t"
      "v_accvgpr_write_b32 a153, v26\n\t"
      "v_mov_b32 v11, v226\n\t"
      "v_mov_b32 v26, v227\n\t"
      "v_accvgpr_read_b32 v226, a154\n\t"
      "v_accvgpr_read_b32 v227, a155\n\t"
      "v_accvgpr_write_b32 a154, v11\n\t"
      "v_accvgpr_write_b32 a155, v26\n\t"
      "v_mov_b32 v11, v232\n\t"
      "v_mov_b32 v26, v233\n\t"
      "v_accvgpr_read_b32 v232, a160\n\t"
      "v_accvgpr_read_b32 v233, a161\n\t"
      "v_accvgpr_write_b32 a160, v11\n\t"
      "v_accvgpr_write_b32 a161, v26\n\t"
      "v_mov_b32 v11, v234\n\t"
      "v_mov_b32 v26, v235\n\t"
      "v_accvgpr_read_b32 v234, a162\n\t"
      "v_accvgpr_read_b32 v235, a163\n\t"
      "v_accvgpr_write_b32 a162, v11\n\t"
      "v_accvgpr_write_b32 a163, v26\n\t"
      "v_mov_b32 v11, v240\n\t"
      "v_mov_b32 v26, v241\n\t"
      "v_accvgpr_read_b32 v240, a168\n\t"
      "v_accvgpr_read_b32 v241, a169\n\t"
      "v_accvgpr_write_b32 a168, v11\n\t"
      "v_accvgpr_write_b32 a169, v26\n\t"
      "v_mov_b32 v11, v242\n\t"
      "v_mov_b32 v26, v243\n\t"
      "v_accvgpr_read_b32 v242, a170\n\t"
      "v_accvgpr_read_b32 v243, a171\n\t"
      "v_accvgpr_write_b32 a170, v11\n\t"
      "v_accvgpr_write_b32 a171, v26\n\t"
      "v_mov_b32 v11, v248\n\t"
      "v_mov_b32 v26, v249\n\t"
      "v_accvgpr_read_b32 v248, a176\n\t"
      "v_accvgpr_read_b32 v249, a177\n\t"
      "v_accvgpr_write_b32 a176, v11\n\t"
      "v_accvgpr_write_b32 a177, v26\n\t"
      "v_mov_b32 v11, v250\n\t"
      "v_mov_b32 v26, v251\n\t"
      "v_accvgpr_read_b32 v250, a178\n\t"
      "v_accvgpr_read_b32 v251, a179\n\t"
      "v_accvgpr_write_b32 a178, v11\n\t"
      "v_accvgpr_write_b32 a179, v26\n\t"
      "v_mov_b32 v11, v196\n\t"
      "v_mov_b32 v26, v197\n\t"
      "v_accvgpr_read_b32 v196, a124\n\t"
      "v_accvgpr_read_b32 v197, a125\n\t"
      "v_accvgpr_write_b32 a124, v11\n\t"
      "v_accvgpr_write_b32 a125, v26\n\t"
      "v_mov_b32 v11, v198\n\t"
      "v_mov_b32 v26, v199\n\t"
      "v_accvgpr_read_b32 v198, a126\n\t"
      "v_accvgpr_read_b32 v199, a127\n\t"
      "v_accvgpr_write_b32 a126, v11\n\t"
      "v_accvgpr_write_b32 a127, v26\n\t"
      "v_mov_b32 v11, v204\n\t"
      "v_mov_b32 v26, v205\n\t"
      "v_accvgpr_read_b32 v204, a132\n\t"
      "v_accvgpr_read_b32 v205, a133\n\t"
      "v_accvgpr_write_b32 a132, v11\n\t"
      "v_accvgpr_write_b32 a133, v26\n\t"
      "v_mov_b32 v11, v206\n\t"
      "v_mov_b32 v26, v207\n\t"
      "v_accvgpr_read_b32 v206, a134\n\t"
      "v_accvgpr_read_b32 v207, a135\n\t"
      "v_accvgpr_write_b32 a134, v11\n\t"
      "v_accvgpr_write_b32 a135, v26\n\t"
      "v_mov_b32 v11, v212\n\t"
      "v_mov_b32 v26, v213\n\t"
      "v_accvgpr_read_b32 v212, a140\n\t"
      "v_accvgpr_read_b32 v213, a141\n\t"
      "v_accvgpr_write_b32 a140, v11\n\t"
      "v_accvgpr_write_b32 a141, v26\n\t"
      "v_mov_b32 v11, v214\n\t"
      "v_mov_b32 v26, v215\n\t"
      "v_accvgpr_read_b32 v214, a142\n\t"
      "v_accvgpr_read_b32 v215, a143\n\t"
      "v_accvgpr_write_b32 a142, v11\n\t"
      "v_accvgpr_write_b32 a143, v26\n\t"
      "v_mov_b32 v11, v220\n\t"
      "v_mov_b32 v26, v221\n\t"
      "v_accvgpr_read_b32 v220, a148\n\t"
      "v_accvgpr_read_b32 v221, a149\n\t"
      "v_accvgpr_write_b32 a148, v11\n\t"
      "v_accvgpr_write_b32 a149, v26\n\t"
      "v_mov_b32 v11, v222\n\t"
      "v_mov_b32 v26, v223\n\t"
      "v_accvgpr_read_b32 v222, a150\n\t"
      "v_accvgpr_read_b32 v223, a151\n\t"
      "v_accvgpr_write_b32 a150, v11\n\t"
      "v_accvgpr_write_b32 a151, v26\n\t"
      "v_mov_b32 v11, v228\n\t"
      "v_mov_b32 v26, v229\n\t"
      "v_accvgpr_read_b32 v228, a156\n\t"
      "v_accvgpr_read_b32 v229, a157\n\t"
      "v_accvgpr_write_b32 a156, v11\n\t"
      "v_accvgpr_write_b32 a157, v26\n\t"
      "v_mov_b32 v11, v230\n\t"
      "v_mov_b32 v26, v231\n\t"
      "v_accvgpr_read_b32 v230, a158\n\t"
      "v_accvgpr_read_b32 v231, a159\n\t"
      "v_accvgpr_write_b32 a158, v11\n\t"
      "v_accvgpr_write_b32 a159, v26\n\t"
      "v_mov_b32 v11, v236\n\t"
      "v_mov_b32 v26, v237\n\t"
      "v_accvgpr_read_b32 v236, a164\n\t"
      "v_accvgpr_read_b32 v237, a165\n\t"
      "v_accvgpr_write_b32 a164, v11\n\t"
      "v_accvgpr_write_b32 a165, v26\n\t"
      "v_mov_b32 v11, v238\n\t"
      "v_mov_b32 v26, v239\n\t"
      "v_accvgpr_read_b32 v238, a166\n\t"
      "v_accvgpr_read_b32 v239, a167\n\t"
      "v_accvgpr_write_b32 a166, v11\n\t"
      "v_accvgpr_write_b32 a167, v26\n\t"
      "v_mov_b32 v11, v244\n\t"
      "v_mov_b32 v26, v245\n\t"
      "v_accvgpr_read_b32 v244, a172\n\t"
      "v_accvgpr_read_b32 v245, a173\n\t"
      "v_accvgpr_write_b32 a172, v11\n\t"
      "v_accvgpr_write_b32 a173, v26\n\t"
      "v_mov_b32 v11, v246\n\t"
      "v_mov_b32 v26, v247\n\t"
      "v_accvgpr_read_b32 v246, a174\n\t"
      "v_accvgpr_read_b32 v247, a175\n\t"
      "v_accvgpr_write_b32 a174, v11\n\t"
      "v_accvgpr_write_b32 a175, v26\n\t"
      "v_mov_b32 v11, v252\n\t"
      "v_mov_b32 v26, v253\n\t"
      "v_accvgpr_read_b32 v252, a180\n\t"
      "v_accvgpr_read_b32 v253, a181\n\t"
      "v_accvgpr_write_b32 a180, v11\n\t"
      "v_accvgpr_write_b32 a181, v26\n\t"
      "v_mov_b32 v11, v254\n\t"
      "v_mov_b32 v26, v255\n\t"
      "v_accvgpr_read_b32 v254, a182\n\t"
      "v_accvgpr_read_b32 v255, a183\n\t"
      "v_accvgpr_write_b32 a182, v11\n\t"
      "v_accvgpr_write_b32 a183, v26\n\t"
      "s_waitcnt lgkmcnt(0)\n\t"
      "s_nop 1\n\t"
      "s_branch ALD3D_%=\n\t"
      "ALN34D_%=:\n\t"
      "s_add_u32 m0, s51, 16384\n\t"
      "s_xor_b32 s56, s57, 0x0\n\t"
      "buffer_load_dwordx4 v12, s[28:31], s56 offen lds\n\t"
      "buffer_load_dwordx4 v13, s[28:31], s56 offen offset:1024 lds\n\t"
      "buffer_load_dwordx4 v14, s[28:31], s56 offen offset:2048 lds\n\t"
      "buffer_load_dwordx4 v15, s[28:31], s56 offen offset:3072 lds\n\t"
      "s_add_u32 m0, s51, 20480\n\t"
      "s_xor_b32 s56, s57, 0x20000\n\t"
      "buffer_load_dwordx4 v12, s[28:31], s56 offen lds\n\t"
      "buffer_load_dwordx4 v13, s[28:31], s56 offen offset:1024 lds\n\t"
      "buffer_load_dwordx4 v14, s[28:31], s56 offen offset:2048 lds\n\t"
      "buffer_load_dwordx4 v15, s[28:31], s56 offen offset:3072 lds\n\t"
      "s_add_u32 m0, s51, 24576\n\t"
      "s_xor_b32 s56, s57, 0x40000\n\t"
      "buffer_load_dwordx4 v12, s[28:31], s56 offen lds\n\t"
      "buffer_load_dwordx4 v13, s[28:31], s56 offen offset:1024 lds\n\t"
      "buffer_load_dwordx4 v14, s[28:31], s56 offen offset:2048 lds\n\t"
      "buffer_load_dwordx4 v15, s[28:31], s56 offen offset:3072 lds\n\t"
      "s_add_u32 m0, s51, 28672\n\t"
      "s_xor_b32 s56, s57, 0x60000\n\t"
      "buffer_load_dwordx4 v12, s[28:31], s56 offen lds\n\t"
      "buffer_load_dwordx4 v13, s[28:31], s56 offen offset:1024 lds\n\t"
      "buffer_load_dwordx4 v14, s[28:31], s56 offen offset:2048 lds\n\t"
      "buffer_load_dwordx4 v15, s[28:31], s56 offen offset:3072 lds\n\t"
      "buffer_load_dwordx4 a[0:3], v7, s[32:35], s63 offen sc0 sc1\n\t"
      "buffer_load_dwordx4 a[4:7], v3, s[32:35], s63 offen sc0 sc1\n\t"
      "buffer_load_dwordx4 a[8:11], v7, s[32:35], s63 offen offset:2048 sc0 sc1\n\t"
      "buffer_load_dwordx4 a[12:15], v3, s[32:35], s63 offen offset:2048 sc0 sc1\n\t"
      "s_add_u32 s63, s63, 0x1000\n\t"
      "buffer_load_dwordx4 a[16:19], v7, s[32:35], s63 offen sc0 sc1\n\t"
      "buffer_load_dwordx4 a[20:23], v3, s[32:35], s63 offen sc0 sc1\n\t"
      "buffer_load_dwordx4 a[24:27], v7, s[32:35], s63 offen offset:2048 sc0 sc1\n\t"
      "buffer_load_dwordx4 a[28:31], v3, s[32:35], s63 offen offset:2048 sc0 sc1\n\t"
      "s_add_u32 s63, s63, 0x1000\n\t"
      "buffer_load_dwordx4 a[32:35], v8, s[32:35], s63 offen sc0 sc1\n\t"
      "buffer_load_dwordx4 a[36:39], v4, s[32:35], s63 offen sc0 sc1\n\t"
      "s_waitcnt vmcnt(18)\n\t"
      "ds_read_b64_tr_b8 v[192:193], v16 offset:16384\n\t"
      "ds_read_b64_tr_b8 v[194:195], v16 offset:17408\n\t"
      "ds_read_b64_tr_b8 v[200:201], v17 offset:16384\n\t"
      "ds_read_b64_tr_b8 v[202:203], v17 offset:17408\n\t"
      "ds_read_b64_tr_b8 v[208:209], v18 offset:16384\n\t"
      "ds_read_b64_tr_b8 v[210:211], v18 offset:17408\n\t"
      "ds_read_b64_tr_b8 v[216:217], v19 offset:16384\n\t"
      "ds_read_b64_tr_b8 v[218:219], v19 offset:17408\n\t"
      "ds_read_b64_tr_b8 v[224:225], v20 offset:16384\n\t"
      "ds_read_b64_tr_b8 v[226:227], v20 offset:17408\n\t"
      "ds_read_b64_tr_b8 v[232:233], v21 offset:16384\n\t"
      "ds_read_b64_tr_b8 v[234:235], v21 offset:17408\n\t"
      "ds_read_b64_tr_b8 v[240:241], v22 offset:16384\n\t"
      "ds_read_b64_tr_b8 v[242:243], v22 offset:17408\n\t"
      "ds_read_b64_tr_b8 v[248:249], v23 offset:16384\n\t"
      "ds_read_b64_tr_b8 v[250:251], v23 offset:17408\n\t"
      "s_waitcnt vmcnt(10)\n\t"
      "ds_read_b64_tr_b8 v[196:197], v16 offset:24576\n\t"
      "ds_read_b64_tr_b8 v[198:199], v16 offset:25600\n\t"
      "ds_read_b64_tr_b8 v[204:205], v17 offset:24576\n\t"
      "ds_read_b64_tr_b8 v[206:207], v17 offset:25600\n\t"
      "ds_read_b64_tr_b8 v[212:213], v18 offset:24576\n\t"
      "ds_read_b64_tr_b8 v[214:215], v18 offset:25600\n\t"
      "ds_read_b64_tr_b8 v[220:221], v19 offset:24576\n\t"
      "ds_read_b64_tr_b8 v[222:223], v19 offset:25600\n\t"
      "ds_read_b64_tr_b8 v[228:229], v20 offset:24576\n\t"
      "ds_read_b64_tr_b8 v[230:231], v20 offset:25600\n\t"
      "ds_read_b64_tr_b8 v[236:237], v21 offset:24576\n\t"
      "ds_read_b64_tr_b8 v[238:239], v21 offset:25600\n\t"
      "ds_read_b64_tr_b8 v[244:245], v22 offset:24576\n\t"
      "ds_read_b64_tr_b8 v[246:247], v22 offset:25600\n\t"
      "ds_read_b64_tr_b8 v[252:253], v23 offset:24576\n\t"
      "ds_read_b64_tr_b8 v[254:255], v23 offset:25600\n\t"
      "s_waitcnt lgkmcnt(0)\n\t"
      "s_branch ALD3D_%=\n\t"
      "ALT2_%=:\n\t"
      "s_mov_b32 s21, 0\n\t"
      "s_mov_b32 s58, 1\n\t"
      "v_mfma_f32_16x16x128_f8f6f4 v[120:123], a[56:63], a[168:175], v[120:123]\n\t"
      "ds_read_b64_tr_b8 v[244:245], v22 offset:16384\n\t"
      "ds_read_b64_tr_b8 v[246:247], v22 offset:17408\n\t"
      "v_mfma_f32_16x16x128_f8f6f4 v[124:127], a[56:63], a[176:183], v[124:127]\n\t"
      "ds_read_b64_tr_b8 v[252:253], v23 offset:16384\n\t"
      "ds_read_b64_tr_b8 v[254:255], v23 offset:17408\n\t"
      "s_call_b64 s[66:67], SETC_%=\n\t"
      "v_mfma_f32_16x16x128_f8f6f4 v[128:131], a[64:71], a[120:127], v[128:131]\n\t"
      "v_mfma_f32_16x16x128_f8f6f4 v[132:135], a[64:71], a[128:135], v[132:135]\n\t"
      "v_mfma_f32_16x16x128_f8f6f4 v[136:139], a[64:71], a[136:143], v[136:139]\n\t"
      "v_mfma_f32_16x16x128_f8f6f4 v[140:143], a[64:71], a[144:151], v[140:143]\n\t"
      "v_mfma_f32_16x16x128_f8f6f4 v[144:147], a[64:71], a[152:159], v[144:147]\n\t"
      "v_mfma_f32_16x16x128_f8f6f4 v[148:151], a[64:71], a[160:167], v[148:151]\n\t"
      "v_mfma_f32_16x16x128_f8f6f4 v[152:155], a[64:71], a[168:175], v[152:155]\n\t"
      "v_mfma_f32_16x16x128_f8f6f4 v[156:159], a[64:71], a[176:183], v[156:159]\n\t"
      "v_mfma_f32_16x16x128_f8f6f4 v[160:163], a[72:79], a[120:127], v[160:163]\n\t"
      "v_mfma_f32_16x16x128_f8f6f4 v[164:167], a[72:79], a[128:135], v[164:167]\n\t"
      "v_mfma_f32_16x16x128_f8f6f4 v[168:171], a[72:79], a[136:143], v[168:171]\n\t"
      "v_mfma_f32_16x16x128_f8f6f4 v[172:175], a[72:79], a[144:151], v[172:175]\n\t"
      "v_mfma_f32_16x16x128_f8f6f4 v[176:179], a[72:79], a[152:159], v[176:179]\n\t"
      "v_mfma_f32_16x16x128_f8f6f4 v[180:183], a[72:79], a[160:167], v[180:183]\n\t"
      "v_mfma_f32_16x16x128_f8f6f4 v[184:187], a[72:79], a[168:175], v[184:187]\n\t"
      "v_mfma_f32_16x16x128_f8f6f4 v[188:191], a[72:79], a[176:183], v[188:191]\n\t"
      "s_call_b64 s[66:67], SPIN1_%=\n\t"
      "buffer_load_dwordx4 a[80:83], v7, s[32:35], s63 offen sc0 sc1\n\t"
      "buffer_load_dwordx4 a[84:87], v3, s[32:35], s63 offen sc0 sc1\n\t"
      "buffer_load_dwordx4 a[88:91], v7, s[32:35], s63 offen offset:2048 sc0 sc1\n\t"
      "buffer_load_dwordx4 a[92:95], v3, s[32:35], s63 offen offset:2048 sc0 sc1\n\t"
      "s_and_b32 vcc_lo, vcc_lo, 2\n\t"
      "s_cbranch_scc0 ALB2_%=\n\t"
      "ALC2_%=:\n\t"
      "s_add_u32 s63, s63, 0x1000\n\t"
      "buffer_load_dwordx4 a[96:99], v7, s[32:35], s63 offen sc0 sc1\n\t"
      "buffer_load_dwordx4 a[100:103], v3, s[32:35], s63 offen sc0 sc1\n\t"
      "buffer_load_dwordx4 a[104:107], v7, s[32:35], s63 offen offset:2048 sc0 sc1\n\t"
      "buffer_load_dwordx4 a[108:111], v3, s[32:35], s63 offen offset:2048 sc0 sc1\n\t"
      "s_add_u32 s63, s63, 0x1000\n\t"
      "buffer_load_dwordx4 a[112:115], v8, s[32:35], s63 offen sc0 sc1\n\t"
      "buffer_load_dwordx4 a[116:119], v4, s[32:35], s63 offen sc0 sc1\n\t"
      "ALD2_%=:\n\t"
      "s_branch JOIN2_%=\n\t"
      "ALB2_%=:\n\t"
      "s_call_b64 s[66:67], SPINB_%=\n\t"
      "s_branch ALC2_%=\n\t"
      "ALT3_%=:\n\t"
      "s_mov_b32 s21, 0\n\t"
      "s_mov_b32 s58, 1\n\t"
      "v_mfma_f32_16x16x128_f8f6f4 v[120:123], a[96:103], a[232:239], v[120:123]\n\t"
      "v_mfma_f32_16x16x128_f8f6f4 v[124:127], a[96:103], a[240:247], v[124:127]\n\t"
      "s_call_b64 s[66:67], SETC_%=\n\t"
      "v_mfma_f32_16x16x128_f8f6f4 v[128:131], a[104:111], a[184:191], v[128:131]\n\t"
      "v_mfma_f32_16x16x128_f8f6f4 v[132:135], a[104:111], a[192:199], v[132:135]\n\t"
      "v_mfma_f32_16x16x128_f8f6f4 v[136:139], a[104:111], a[200:207], v[136:139]\n\t"
      "v_mfma_f32_16x16x128_f8f6f4 v[140:143], a[104:111], a[208:215], v[140:143]\n\t"
      "v_mfma_f32_16x16x128_f8f6f4 v[144:147], a[104:111], a[216:223], v[144:147]\n\t"
      "v_mfma_f32_16x16x128_f8f6f4 v[148:151], a[104:111], a[224:231], v[148:151]\n\t"
      "v_mfma_f32_16x16x128_f8f6f4 v[152:155], a[104:111], a[232:239], v[152:155]\n\t"
      "v_mfma_f32_16x16x128_f8f6f4 v[156:159], a[104:111], a[240:247], v[156:159]\n\t"
      "v_mfma_f32_16x16x128_f8f6f4 v[160:163], a[112:119], a[184:191], v[160:163]\n\t"
      "v_mfma_f32_16x16x128_f8f6f4 v[164:167], a[112:119], a[192:199], v[164:167]\n\t"
      "v_mfma_f32_16x16x128_f8f6f4 v[168:171], a[112:119], a[200:207], v[168:171]\n\t"
      "v_mfma_f32_16x16x128_f8f6f4 v[172:175], a[112:119], a[208:215], v[172:175]\n\t"
      "v_mfma_f32_16x16x128_f8f6f4 v[176:179], a[112:119], a[216:223], v[176:179]\n\t"
      "v_mfma_f32_16x16x128_f8f6f4 v[180:183], a[112:119], a[224:231], v[180:183]\n\t"
      "v_mfma_f32_16x16x128_f8f6f4 v[184:187], a[112:119], a[232:239], v[184:187]\n\t"
      "v_mfma_f32_16x16x128_f8f6f4 v[188:191], a[112:119], a[240:247], v[188:191]\n\t"
      "s_call_b64 s[66:67], SPIN1_%=\n\t"
      "buffer_load_dwordx4 a[0:3], v7, s[32:35], s63 offen sc0 sc1\n\t"
      "buffer_load_dwordx4 a[4:7], v3, s[32:35], s63 offen sc0 sc1\n\t"
      "buffer_load_dwordx4 a[8:11], v7, s[32:35], s63 offen offset:2048 sc0 sc1\n\t"
      "buffer_load_dwordx4 a[12:15], v3, s[32:35], s63 offen offset:2048 sc0 sc1\n\t"
      "s_and_b32 vcc_lo, vcc_lo, 2\n\t"
      "s_cbranch_scc0 ALB3_%=\n\t"
      "ALC3_%=:\n\t"
      "s_add_u32 s63, s63, 0x1000\n\t"
      "buffer_load_dwordx4 a[16:19], v7, s[32:35], s63 offen sc0 sc1\n\t"
      "buffer_load_dwordx4 a[20:23], v3, s[32:35], s63 offen sc0 sc1\n\t"
      "buffer_load_dwordx4 a[24:27], v7, s[32:35], s63 offen offset:2048 sc0 sc1\n\t"
      "buffer_load_dwordx4 a[28:31], v3, s[32:35], s63 offen offset:2048 sc0 sc1\n\t"
      "s_add_u32 s63, s63, 0x1000\n\t"
      "buffer_load_dwordx4 a[32:35], v8, s[32:35], s63 offen sc0 sc1\n\t"
      "buffer_load_dwordx4 a[36:39], v4, s[32:35], s63 offen sc0 sc1\n\t"
      "ALD3_%=:\n\t"
      "s_branch JOIN3_%=\n\t"
      "ALB3_%=:\n\t"
      "s_call_b64 s[66:67], SPINB_%=\n\t"
      "s_branch ALC3_%=\n\t"
      "ALT4_%=:\n\t"
      "s_mov_b32 s21, 17408\n\t"
      "s_mov_b32 s58, 1\n\t"
      "v_mfma_f32_16x16x128_f8f6f4 v[108:111], a[16:23], v[216:223], v[108:111]\n\t"
      "v_mfma_f32_16x16x128_f8f6f4 v[112:115], a[16:23], v[224:231], v[112:115]\n\t"
      "s_call_b64 s[66:67], SETC_%=\n\t"
      "v_mfma_f32_16x16x128_f8f6f4 v[116:119], a[16:23], v[232:239], v[116:119]\n\t"
      "v_mfma_f32_16x16x128_f8f6f4 v[120:123], a[16:23], v[240:247], v[120:123]\n\t"
      "v_mfma_f32_16x16x128_f8f6f4 v[124:127], a[16:23], v[248:255], v[124:127]\n\t"
      "v_mfma_f32_16x16x128_f8f6f4 v[128:131], a[24:31], v[192:199], v[128:131]\n\t"
      "v_mfma_f32_16x16x128_f8f6f4 v[132:135], a[24:31], v[200:207], v[132:135]\n\t"
      "v_mfma_f32_16x16x128_f8f6f4 v[136:139], a[24:31], v[208:215], v[136:139]\n\t"
      "v_mfma_f32_16x16x128_f8f6f4 v[140:143], a[24:31], v[216:223], v[140:143]\n\t"
      "v_mfma_f32_16x16x128_f8f6f4 v[144:147], a[24:31], v[224:231], v[144:147]\n\t"
      "v_mfma_f32_16x16x128_f8f6f4 v[148:151], a[24:31], v[232:239], v[148:151]\n\t"
      "v_mfma_f32_16x16x128_f8f6f4 v[152:155], a[24:31], v[240:247], v[152:155]\n\t"
      "v_mfma_f32_16x16x128_f8f6f4 v[156:159], a[24:31], v[248:255], v[156:159]\n\t"
      "v_mfma_f32_16x16x128_f8f6f4 v[160:163], a[32:39], v[192:199], v[160:163]\n\t"
      "v_mfma_f32_16x16x128_f8f6f4 v[164:167], a[32:39], v[200:207], v[164:167]\n\t"
      "v_mfma_f32_16x16x128_f8f6f4 v[168:171], a[32:39], v[208:215], v[168:171]\n\t"
      "v_mfma_f32_16x16x128_f8f6f4 v[172:175], a[32:39], v[216:223], v[172:175]\n\t"
      "v_mfma_f32_16x16x128_f8f6f4 v[176:179], a[32:39], v[224:231], v[176:179]\n\t"
      "v_mfma_f32_16x16x128_f8f6f4 v[180:183], a[32:39], v[232:239], v[180:183]\n\t"
      "v_mfma_f32_16x16x128_f8f6f4 v[184:187], a[32:39], v[240:247], v[184:187]\n\t"
      "v_mfma_f32_16x16x128_f8f6f4 v[188:191], a[32:39], v[248:255], v[188:191]\n\t"
      "s_call_b64 s[66:67], SPIN1_%=\n\t"
      "s_cmp_lg_u32 s21, 0\n\t"
      "s_cbranch_scc1 ALS4_%=\n\t"
      "buffer_load_dwordx4 a[40:43], v7, s[32:35], s63 offen sc0 sc1\n\t"
      "buffer_load_dwordx4 a[44:47], v3, s[32:35], s63 offen sc0 sc1\n\t"
      "buffer_load_dwordx4 a[48:51], v7, s[32:35], s63 offen offset:2048 sc0 sc1\n\t"
      "buffer_load_dwordx4 a[52:55], v3, s[32:35], s63 offen offset:2048 sc0 sc1\n\t"
      "s_and_b32 vcc_lo, vcc_lo, 2\n\t"
      "s_cbranch_scc0 ALB4_%=\n\t"
      "ALC4_%=:\n\t"
      "s_add_u32 s63, s63, 0x1000\n\t"
      "buffer_load_dwordx4 a[56:59], v7, s[32:35], s63 offen sc0 sc1\n\t"
      "buffer_load_dwordx4 a[60:63], v3, s[32:35], s63 offen sc0 sc1\n\t"
      "buffer_load_dwordx4 a[64:67], v7, s[32:35], s63 offen offset:2048 sc0 sc1\n\t"
      "buffer_load_dwordx4 a[68:71], v3, s[32:35], s63 offen offset:2048 sc0 sc1\n\t"
      "s_add_u32 s63, s63, 0x1000\n\t"
      "buffer_load_dwordx4 a[72:75], v8, s[32:35], s63 offen sc0 sc1\n\t"
      "buffer_load_dwordx4 a[76:79], v4, s[32:35], s63 offen sc0 sc1\n\t"
      "ALD4_%=:\n\t"
      "s_branch JOIN4_%=\n\t"
      "ALB4_%=:\n\t"
      "s_call_b64 s[66:67], SPINB_%=\n\t"
      "s_branch ALC4_%=\n\t"
      "ALS4_%=:\n\t"
      "s_mov_b32 s56, 32\n\t"
      "s_call_b64 s[66:67], SWAP_%=\n\t"
      "s_waitcnt lgkmcnt(0)\n\t"
      "s_bitcmp0_b32 s21, 10\n\t"
      "s_cbranch_scc1 ALN45_%=\n\t"
      "buffer_load_dwordx4 a[40:43], v7, s[32:35], s63 offen sc0 sc1\n\t"
      "buffer_load_dwordx4 a[44:47], v3, s[32:35], s63 offen sc0 sc1\n\t"
      "buffer_load_dwordx4 a[48:51], v7, s[32:35], s63 offen offset:2048 sc0 sc1\n\t"
      "buffer_load_dwordx4 a[52:55], v3, s[32:35], s63 offen offset:2048 sc0 sc1\n\t"
      "s_add_u32 s63, s63, 0x1000\n\t"
      "buffer_load_dwordx4 a[56:59], v7, s[32:35], s63 offen sc0 sc1\n\t"
      "buffer_load_dwordx4 a[60:63], v3, s[32:35], s63 offen sc0 sc1\n\t"
      "buffer_load_dwordx4 a[64:67], v7, s[32:35], s63 offen offset:2048 sc0 sc1\n\t"
      "buffer_load_dwordx4 a[68:71], v3, s[32:35], s63 offen offset:2048 sc0 sc1\n\t"
      "s_add_u32 s63, s63, 0x1000\n\t"
      "buffer_load_dwordx4 a[72:75], v8, s[32:35], s63 offen sc0 sc1\n\t"
      "buffer_load_dwordx4 a[76:79], v4, s[32:35], s63 offen sc0 sc1\n\t"
      "v_accvgpr_read_b32 v11, a120\n\t"
      "v_accvgpr_read_b32 v26, a121\n\t"
      "v_accvgpr_mov_b32 a120, a184\n\t"
      "v_accvgpr_mov_b32 a121, a185\n\t"
      "v_accvgpr_write_b32 a184, v11\n\t"
      "v_accvgpr_write_b32 a185, v26\n\t"
      "v_accvgpr_read_b32 v11, a122\n\t"
      "v_accvgpr_read_b32 v26, a123\n\t"
      "v_accvgpr_mov_b32 a122, a186\n\t"
      "v_accvgpr_mov_b32 a123, a187\n\t"
      "v_accvgpr_write_b32 a186, v11\n\t"
      "v_accvgpr_write_b32 a187, v26\n\t"
      "v_accvgpr_read_b32 v11, a128\n\t"
      "v_accvgpr_read_b32 v26, a129\n\t"
      "v_accvgpr_mov_b32 a128, a192\n\t"
      "v_accvgpr_mov_b32 a129, a193\n\t"
      "v_accvgpr_write_b32 a192, v11\n\t"
      "v_accvgpr_write_b32 a193, v26\n\t"
      "v_accvgpr_read_b32 v11, a130\n\t"
      "v_accvgpr_read_b32 v26, a131\n\t"
      "v_accvgpr_mov_b32 a130, a194\n\t"
      "v_accvgpr_mov_b32 a131, a195\n\t"
      "v_accvgpr_write_b32 a194, v11\n\t"
      "v_accvgpr_write_b32 a195, v26\n\t"
      "v_accvgpr_read_b32 v11, a136\n\t"
      "v_accvgpr_read_b32 v26, a137\n\t"
      "v_accvgpr_mov_b32 a136, a200\n\t"
      "v_accvgpr_mov_b32 a137, a201\n\t"
      "v_accvgpr_write_b32 a200, v11\n\t"
      "v_accvgpr_write_b32 a201, v26\n\t"
      "v_accvgpr_read_b32 v11, a138\n\t"
      "v_accvgpr_read_b32 v26, a139\n\t"
      "v_accvgpr_mov_b32 a138, a202\n\t"
      "v_accvgpr_mov_b32 a139, a203\n\t"
      "v_accvgpr_write_b32 a202, v11\n\t"
      "v_accvgpr_write_b32 a203, v26\n\t"
      "v_accvgpr_read_b32 v11, a144\n\t"
      "v_accvgpr_read_b32 v26, a145\n\t"
      "v_accvgpr_mov_b32 a144, a208\n\t"
      "v_accvgpr_mov_b32 a145, a209\n\t"
      "v_accvgpr_write_b32 a208, v11\n\t"
      "v_accvgpr_write_b32 a209, v26\n\t"
      "v_accvgpr_read_b32 v11, a146\n\t"
      "v_accvgpr_read_b32 v26, a147\n\t"
      "v_accvgpr_mov_b32 a146, a210\n\t"
      "v_accvgpr_mov_b32 a147, a211\n\t"
      "v_accvgpr_write_b32 a210, v11\n\t"
      "v_accvgpr_write_b32 a211, v26\n\t"
      "v_accvgpr_read_b32 v11, a152\n\t"
      "v_accvgpr_read_b32 v26, a153\n\t"
      "v_accvgpr_mov_b32 a152, a216\n\t"
      "v_accvgpr_mov_b32 a153, a217\n\t"
      "v_accvgpr_write_b32 a216, v11\n\t"
      "v_accvgpr_write_b32 a217, v26\n\t"
      "v_accvgpr_read_b32 v11, a154\n\t"
      "v_accvgpr_read_b32 v26, a155\n\t"
      "v_accvgpr_mov_b32 a154, a218\n\t"
      "v_accvgpr_mov_b32 a155, a219\n\t"
      "v_accvgpr_write_b32 a218, v11\n\t"
      "v_accvgpr_write_b32 a219, v26\n\t"
      "v_accvgpr_read_b32 v11, a160\n\t"
      "v_accvgpr_read_b32 v26, a161\n\t"
      "v_accvgpr_mov_b32 a160, a224\n\t"
      "v_accvgpr_mov_b32 a161, a225\n\t"
      "v_accvgpr_write_b32 a224, v11\n\t"
      "v_accvgpr_write_b32 a225, v26\n\t"
      "v_accvgpr_read_b32 v11, a162\n\t"
      "v_accvgpr_read_b32 v26, a163\n\t"
      "v_accvgpr_mov_b32 a162, a226\n\t"
      "v_accvgpr_mov_b32 a163, a227\n\t"
      "v_accvgpr_write_b32 a226, v11\n\t"
      "v_accvgpr_write_b32 a227, v26\n\t"
      "v_accvgpr_read_b32 v11, a168\n\t"
      "v_accvgpr_read_b32 v26, a169\n\t"
      "v_accvgpr_mov_b32 a168, a232\n\t"
      "v_accvgpr_mov_b32 a169, a233\n\t"
      "v_accvgpr_write_b32 a232, v11\n\t"
      "v_accvgpr_write_b32 a233, v26\n\t"
      "v_accvgpr_read_b32 v11, a170\n\t"
      "v_accvgpr_read_b32 v26, a171\n\t"
      "v_accvgpr_mov_b32 a170, a234\n\t"
      "v_accvgpr_mov_b32 a171, a235\n\t"
      "v_accvgpr_write_b32 a234, v11\n\t"
      "v_accvgpr_write_b32 a235, v26\n\t"
      "v_accvgpr_read_b32 v11, a176\n\t"
      "v_accvgpr_read_b32 v26, a177\n\t"
      "v_accvgpr_mov_b32 a176, a240\n\t"
      "v_accvgpr_mov_b32 a177, a241\n\t"
      "v_accvgpr_write_b32 a240, v11\n\t"
      "v_accvgpr_write_b32 a241, v26\n\t"
      "v_accvgpr_read_b32 v11, a178\n\t"
      "v_accvgpr_read_b32 v26, a179\n\t"
      "v_accvgpr_mov_b32 a178, a242\n\t"
      "v_accvgpr_mov_b32 a179, a243\n\t"
      "v_accvgpr_write_b32 a242, v11\n\t"
      "v_accvgpr_write_b32 a243, v26\n\t"
      "v_accvgpr_read_b32 v11, a124\n\t"
      "v_accvgpr_read_b32 v26, a125\n\t"
      "v_accvgpr_mov_b32 a124, a188\n\t"
      "v_accvgpr_mov_b32 a125, a189\n\t"
      "v_accvgpr_write_b32 a188, v11\n\t"
      "v_accvgpr_write_b32 a189, v26\n\t"
      "v_accvgpr_read_b32 v11, a126\n\t"
      "v_accvgpr_read_b32 v26, a127\n\t"
      "v_accvgpr_mov_b32 a126, a190\n\t"
      "v_accvgpr_mov_b32 a127, a191\n\t"
      "v_accvgpr_write_b32 a190, v11\n\t"
      "v_accvgpr_write_b32 a191, v26\n\t"
      "v_accvgpr_read_b32 v11, a132\n\t"
      "v_accvgpr_read_b32 v26, a133\n\t"
      "v_accvgpr_mov_b32 a132, a196\n\t"
      "v_accvgpr_mov_b32 a133, a197\n\t"
      "v_accvgpr_write_b32 a196, v11\n\t"
      "v_accvgpr_write_b32 a197, v26\n\t"
      "v_accvgpr_read_b32 v11, a134\n\t"
      "v_accvgpr_read_b32 v26, a135\n\t"
      "v_accvgpr_mov_b32 a134, a198\n\t"
      "v_accvgpr_mov_b32 a135, a199\n\t"
      "v_accvgpr_write_b32 a198, v11\n\t"
      "v_accvgpr_write_b32 a199, v26\n\t"
      "v_accvgpr_read_b32 v11, a140\n\t"
      "v_accvgpr_read_b32 v26, a141\n\t"
      "v_accvgpr_mov_b32 a140, a204\n\t"
      "v_accvgpr_mov_b32 a141, a205\n\t"
      "v_accvgpr_write_b32 a204, v11\n\t"
      "v_accvgpr_write_b32 a205, v26\n\t"
      "v_accvgpr_read_b32 v11, a142\n\t"
      "v_accvgpr_read_b32 v26, a143\n\t"
      "v_accvgpr_mov_b32 a142, a206\n\t"
      "v_accvgpr_mov_b32 a143, a207\n\t"
      "v_accvgpr_write_b32 a206, v11\n\t"
      "v_accvgpr_write_b32 a207, v26\n\t"
      "v_accvgpr_read_b32 v11, a148\n\t"
      "v_accvgpr_read_b32 v26, a149\n\t"
      "v_accvgpr_mov_b32 a148, a212\n\t"
      "v_accvgpr_mov_b32 a149, a213\n\t"
      "v_accvgpr_write_b32 a212, v11\n\t"
      "v_accvgpr_write_b32 a213, v26\n\t"
      "v_accvgpr_read_b32 v11, a150\n\t"
      "v_accvgpr_read_b32 v26, a151\n\t"
      "v_accvgpr_mov_b32 a150, a214\n\t"
      "v_accvgpr_mov_b32 a151, a215\n\t"
      "v_accvgpr_write_b32 a214, v11\n\t"
      "v_accvgpr_write_b32 a215, v26\n\t"
      "v_accvgpr_read_b32 v11, a156\n\t"
      "v_accvgpr_read_b32 v26, a157\n\t"
      "v_accvgpr_mov_b32 a156, a220\n\t"
      "v_accvgpr_mov_b32 a157, a221\n\t"
      "v_accvgpr_write_b32 a220, v11\n\t"
      "v_accvgpr_write_b32 a221, v26\n\t"
      "v_accvgpr_read_b32 v11, a158\n\t"
      "v_accvgpr_read_b32 v26, a159\n\t"
      "v_accvgpr_mov_b32 a158, a222\n\t"
      "v_accvgpr_mov_b32 a159, a223\n\t"
      "v_accvgpr_write_b32 a222, v11\n\t"
      "v_accvgpr_write_b32 a223, v26\n\t"
      "v_accvgpr_read_b32 v11, a164\n\t"
      "v_accvgpr_read_b32 v26, a165\n\t"
      "v_accvgpr_mov_b32 a164, a228\n\t"
      "v_accvgpr_mov_b32 a165, a229\n\t"
      "v_accvgpr_write_b32 a228, v11\n\t"
      "v_accvgpr_write_b32 a229, v26\n\t"
      "v_accvgpr_read_b32 v11, a166\n\t"
      "v_accvgpr_read_b32 v26, a167\n\t"
      "v_accvgpr_mov_b32 a166, a230\n\t"
      "v_accvgpr_mov_b32 a167, a231\n\t"
      "v_accvgpr_write_b32 a230, v11\n\t"
      "v_accvgpr_write_b32 a231, v26\n\t"
      "v_accvgpr_read_b32 v11, a172\n\t"
      "v_accvgpr_read_b32 v26, a173\n\t"
      "v_accvgpr_mov_b32 a172, a236\n\t"
      "v_accvgpr_mov_b32 a173, a237\n\t"
      "v_accvgpr_write_b32 a236, v11\n\t"
      "v_accvgpr_write_b32 a237, v26\n\t"
      "v_accvgpr_read_b32 v11, a174\n\t"
      "v_accvgpr_read_b32 v26, a175\n\t"
      "v_accvgpr_mov_b32 a174, a238\n\t"
      "v_accvgpr_mov_b32 a175, a239\n\t"
      "v_accvgpr_write_b32 a238, v11\n\t"
      "v_accvgpr_write_b32 a239, v26\n\t"
      "v_accvgpr_read_b32 v11, a180\n\t"
      "v_accvgpr_read_b32 v26, a181\n\t"
      "v_accvgpr_mov_b32 a180, a244\n\t"
      "v_accvgpr_mov_b32 a181, a245\n\t"
      "v_accvgpr_write_b32 a244, v11\n\t"
      "v_accvgpr_write_b32 a245, v26\n\t"
      "v_accvgpr_read_b32 v11, a182\n\t"
      "v_accvgpr_read_b32 v26, a183\n\t"
      "v_accvgpr_mov_b32 a182, a246\n\t"
      "v_accvgpr_mov_b32 a183, a247\n\t"
      "v_accvgpr_write_b32 a246, v11\n\t"
      "v_accvgpr_write_b32 a247, v26\n\t"
      "s_waitcnt lgkmcnt(0)\n\t"
      "s_nop 1\n\t"
      "s_branch ALD4_%=\n\t"
      "ALN45_%=:\n\t"
      "s_add_u32 m0, s51, 0\n\t"
      "s_xor_b32 s56, s57, 0x0\n\t"
      "buffer_load_dwordx4 v12, s[28:31], s56 offen lds\n\t"
      "buffer_load_dwordx4 v13, s[28:31], s56 offen offset:1024 lds\n\t"
      "buffer_load_dwordx4 v14, s[28:31], s56 offen offset:2048 lds\n\t"
      "buffer_load_dwordx4 v15, s[28:31], s56 offen offset:3072 lds\n\t"
      "s_add_u32 m0, s51, 4096\n\t"
      "s_xor_b32 s56, s57, 0x20000\n\t"
      "buffer_load_dwordx4 v12, s[28:31], s56 offen lds\n\t"
      "buffer_load_dwordx4 v13, s[28:31], s56 offen offset:1024 lds\n\t"
      "buffer_load_dwordx4 v14, s[28:31], s56 offen offset:2048 lds\n\t"
      "buffer_load_dwordx4 v15, s[28:31], s56 offen offset:3072 lds\n\t"
      "s_add_u32 m0, s51, 8192\n\t"
      "s_xor_b32 s56, s57, 0x40000\n\t"
      "buffer_load_dwordx4 v12, s[28:31], s56 offen lds\n\t"
      "buffer_load_dwordx4 v13, s[28:31], s56 offen offset:1024 lds\n\t"
      "buffer_load_dwordx4 v14, s[28:31], s56 offen offset:2048 lds\n\t"
      "buffer_load_dwordx4 v15, s[28:31], s56 offen offset:3072 lds\n\t"
      "s_add_u32 m0, s51, 12288\n\t"
      "s_xor_b32 s56, s57, 0x60000\n\t"
      "buffer_load_dwordx4 v12, s[28:31], s56 offen lds\n\t"
      "buffer_load_dwordx4 v13, s[28:31], s56 offen offset:1024 lds\n\t"
      "buffer_load_dwordx4 v14, s[28:31], s56 offen offset:2048 lds\n\t"
      "buffer_load_dwordx4 v15, s[28:31], s56 offen offset:3072 lds\n\t"
      "buffer_load_dwordx4 a[40:43], v7, s[32:35], s63 offen sc0 sc1\n\t"
      "buffer_load_dwordx4 a[44:47], v3, s[32:35], s63 offen sc0 sc1\n\t"
      "buffer_load_dwordx4 a[48:51], v7, s[32:35], s63 offen offset:2048 sc0 sc1\n\t"
      "buffer_load_dwordx4 a[52:55], v3, s[32:35], s63 offen offset:2048 sc0 sc1\n\t"
      "s_add_u32 s63, s63, 0x1000\n\t"
      "buffer_load_dwordx4 a[56:59], v7, s[32:35], s63 offen sc0 sc1\n\t"
      "buffer_load_dwordx4 a[60:63], v3, s[32:35], s63 offen sc0 sc1\n\t"
      "buffer_load_dwordx4 a[64:67], v7, s[32:35], s63 offen offset:2048 sc0 sc1\n\t"
      "buffer_load_dwordx4 a[68:71], v3, s[32:35], s63 offen offset:2048 sc0 sc1\n\t"
      "s_add_u32 s63, s63, 0x1000\n\t"
      "buffer_load_dwordx4 a[72:75], v8, s[32:35], s63 offen sc0 sc1\n\t"
      "buffer_load_dwordx4 a[76:79], v4, s[32:35], s63 offen sc0 sc1\n\t"
      "s_waitcnt vmcnt(18)\n\t"
      "ds_read_b64_tr_b8 a[120:121], v16 offset:0\n\t"
      "ds_read_b64_tr_b8 a[122:123], v16 offset:1024\n\t"
      "ds_read_b64_tr_b8 a[128:129], v17 offset:0\n\t"
      "ds_read_b64_tr_b8 a[130:131], v17 offset:1024\n\t"
      "ds_read_b64_tr_b8 a[136:137], v18 offset:0\n\t"
      "ds_read_b64_tr_b8 a[138:139], v18 offset:1024\n\t"
      "ds_read_b64_tr_b8 a[144:145], v19 offset:0\n\t"
      "ds_read_b64_tr_b8 a[146:147], v19 offset:1024\n\t"
      "ds_read_b64_tr_b8 a[152:153], v20 offset:0\n\t"
      "ds_read_b64_tr_b8 a[154:155], v20 offset:1024\n\t"
      "ds_read_b64_tr_b8 a[160:161], v21 offset:0\n\t"
      "ds_read_b64_tr_b8 a[162:163], v21 offset:1024\n\t"
      "ds_read_b64_tr_b8 a[168:169], v22 offset:0\n\t"
      "ds_read_b64_tr_b8 a[170:171], v22 offset:1024\n\t"
      "ds_read_b64_tr_b8 a[176:177], v23 offset:0\n\t"
      "ds_read_b64_tr_b8 a[178:179], v23 offset:1024\n\t"
      "s_waitcnt vmcnt(10)\n\t"
      "ds_read_b64_tr_b8 a[124:125], v16 offset:8192\n\t"
      "ds_read_b64_tr_b8 a[126:127], v16 offset:9216\n\t"
      "ds_read_b64_tr_b8 a[132:133], v17 offset:8192\n\t"
      "ds_read_b64_tr_b8 a[134:135], v17 offset:9216\n\t"
      "ds_read_b64_tr_b8 a[140:141], v18 offset:8192\n\t"
      "ds_read_b64_tr_b8 a[142:143], v18 offset:9216\n\t"
      "ds_read_b64_tr_b8 a[148:149], v19 offset:8192\n\t"
      "ds_read_b64_tr_b8 a[150:151], v19 offset:9216\n\t"
      "ds_read_b64_tr_b8 a[156:157], v20 offset:8192\n\t"
      "ds_read_b64_tr_b8 a[158:159], v20 offset:9216\n\t"
      "ds_read_b64_tr_b8 a[164:165], v21 offset:8192\n\t"
      "ds_read_b64_tr_b8 a[166:167], v21 offset:9216\n\t"
      "ds_read_b64_tr_b8 a[172:173], v22 offset:8192\n\t"
      "ds_read_b64_tr_b8 a[174:175], v22 offset:9216\n\t"
      "ds_read_b64_tr_b8 a[180:181], v23 offset:8192\n\t"
      "ds_read_b64_tr_b8 a[182:183], v23 offset:9216\n\t"
      "s_waitcnt lgkmcnt(0)\n\t"
      "s_branch ALD4_%=\n\t"
      "ALT5_%=:\n\t"
      "s_mov_b32 s21, 20480\n\t"
      "s_mov_b32 s58, 1\n\t"
      "v_mfma_f32_16x16x128_f8f6f4 v[108:111], a[56:63], a[144:151], v[108:111]\n\t"
      "v_mfma_f32_16x16x128_f8f6f4 v[112:115], a[56:63], a[152:159], v[112:115]\n\t"
      "s_call_b64 s[66:67], SETC_%=\n\t"
      "v_mfma_f32_16x16x128_f8f6f4 v[116:119], a[56:63], a[160:167], v[116:119]\n\t"
      "v_mfma_f32_16x16x128_f8f6f4 v[120:123], a[56:63], a[168:175], v[120:123]\n\t"
      "v_mfma_f32_16x16x128_f8f6f4 v[124:127], a[56:63], a[176:183], v[124:127]\n\t"
      "v_mfma_f32_16x16x128_f8f6f4 v[128:131], a[64:71], a[120:127], v[128:131]\n\t"
      "v_mfma_f32_16x16x128_f8f6f4 v[132:135], a[64:71], a[128:135], v[132:135]\n\t"
      "v_mfma_f32_16x16x128_f8f6f4 v[136:139], a[64:71], a[136:143], v[136:139]\n\t"
      "v_mfma_f32_16x16x128_f8f6f4 v[140:143], a[64:71], a[144:151], v[140:143]\n\t"
      "v_mfma_f32_16x16x128_f8f6f4 v[144:147], a[64:71], a[152:159], v[144:147]\n\t"
      "v_mfma_f32_16x16x128_f8f6f4 v[148:151], a[64:71], a[160:167], v[148:151]\n\t"
      "v_mfma_f32_16x16x128_f8f6f4 v[152:155], a[64:71], a[168:175], v[152:155]\n\t"
      "v_mfma_f32_16x16x128_f8f6f4 v[156:159], a[64:71], a[176:183], v[156:159]\n\t"
      "v_mfma_f32_16x16x128_f8f6f4 v[160:163], a[72:79], a[120:127], v[160:163]\n\t"
      "v_mfma_f32_16x16x128_f8f6f4 v[164:167], a[72:79], a[128:135], v[164:167]\n\t"
      "v_mfma_f32_16x16x128_f8f6f4 v[168:171], a[72:79], a[136:143], v[168:171]\n\t"
      "v_mfma_f32_16x16x128_f8f6f4 v[172:175], a[72:79], a[144:151], v[172:175]\n\t"
      "v_mfma_f32_16x16x128_f8f6f4 v[176:179], a[72:79], a[152:159], v[176:179]\n\t"
      "v_mfma_f32_16x16x128_f8f6f4 v[180:183], a[72:79], a[160:167], v[180:183]\n\t"
      "v_mfma_f32_16x16x128_f8f6f4 v[184:187], a[72:79], a[168:175], v[184:187]\n\t"
      "v_mfma_f32_16x16x128_f8f6f4 v[188:191], a[72:79], a[176:183], v[188:191]\n\t"
      "s_call_b64 s[66:67], SPIN1_%=\n\t"
      "s_cmp_lg_u32 s21, 0\n\t"
      "s_cbranch_scc1 ALS5_%=\n\t"
      "buffer_load_dwordx4 a[80:83], v7, s[32:35], s63 offen sc0 sc1\n\t"
      "buffer_load_dwordx4 a[84:87], v3, s[32:35], s63 offen sc0 sc1\n\t"
      "buffer_load_dwordx4 a[88:91], v7, s[32:35], s63 offen offset:2048 sc0 sc1\n\t"
      "buffer_load_dwordx4 a[92:95], v3, s[32:35], s63 offen offset:2048 sc0 sc1\n\t"
      "s_and_b32 vcc_lo, vcc_lo, 2\n\t"
      "s_cbranch_scc0 ALB5_%=\n\t"
      "ALC5_%=:\n\t"
      "s_add_u32 s63, s63, 0x1000\n\t"
      "buffer_load_dwordx4 a[96:99], v7, s[32:35], s63 offen sc0 sc1\n\t"
      "buffer_load_dwordx4 a[100:103], v3, s[32:35], s63 offen sc0 sc1\n\t"
      "buffer_load_dwordx4 a[104:107], v7, s[32:35], s63 offen offset:2048 sc0 sc1\n\t"
      "buffer_load_dwordx4 a[108:111], v3, s[32:35], s63 offen offset:2048 sc0 sc1\n\t"
      "s_add_u32 s63, s63, 0x1000\n\t"
      "buffer_load_dwordx4 a[112:115], v8, s[32:35], s63 offen sc0 sc1\n\t"
      "buffer_load_dwordx4 a[116:119], v4, s[32:35], s63 offen sc0 sc1\n\t"
      "ALD5_%=:\n\t"
      "s_branch JOIN5_%=\n\t"
      "ALB5_%=:\n\t"
      "s_call_b64 s[66:67], SPINB_%=\n\t"
      "s_branch ALC5_%=\n\t"
      "ALS5_%=:\n\t"
      "s_mov_b32 s56, 40\n\t"
      "s_call_b64 s[66:67], SWAP_%=\n\t"
      "s_waitcnt lgkmcnt(0)\n\t"
      "s_bitcmp0_b32 s21, 12\n\t"
      "s_cbranch_scc1 ALN56_%=\n\t"
      "ds_read_b64_tr_b8 a[188:189], v16 offset:24576\n\t"
      "ds_read_b64_tr_b8 a[190:191], v16 offset:25600\n\t"
      "ds_read_b64_tr_b8 a[196:197], v17 offset:24576\n\t"
      "ds_read_b64_tr_b8 a[198:199], v17 offset:25600\n\t"
      "ds_read_b64_tr_b8 a[204:205], v18 offset:24576\n\t"
      "ds_read_b64_tr_b8 a[206:207], v18 offset:25600\n\t"
      "ds_read_b64_tr_b8 a[212:213], v19 offset:24576\n\t"
      "ds_read_b64_tr_b8 a[214:215], v19 offset:25600\n\t"
      "ds_read_b64_tr_b8 a[220:221], v20 offset:24576\n\t"
      "ds_read_b64_tr_b8 a[222:223], v20 offset:25600\n\t"
      "ds_read_b64_tr_b8 a[228:229], v21 offset:24576\n\t"
      "ds_read_b64_tr_b8 a[230:231], v21 offset:25600\n\t"
      "ds_read_b64_tr_b8 a[236:237], v22 offset:24576\n\t"
      "ds_read_b64_tr_b8 a[238:239], v22 offset:25600\n\t"
      "ds_read_b64_tr_b8 a[244:245], v23 offset:24576\n\t"
      "ds_read_b64_tr_b8 a[246:247], v23 offset:25600\n\t"
      "s_waitcnt lgkmcnt(0)\n\t"
      "s_add_u32 m0, s51, 24576\n\t"
      "s_xor_b32 s56, s19, 0x40000\n\t"
      "buffer_load_dwordx4 v12, s[28:31], s56 offen lds\n\t"
      "buffer_load_dwordx4 v13, s[28:31], s56 offen offset:1024 lds\n\t"
      "buffer_load_dwordx4 v14, s[28:31], s56 offen offset:2048 lds\n\t"
      "buffer_load_dwordx4 v15, s[28:31], s56 offen offset:3072 lds\n\t"
      "s_add_u32 m0, s51, 28672\n\t"
      "s_xor_b32 s56, s19, 0x60000\n\t"
      "buffer_load_dwordx4 v12, s[28:31], s56 offen lds\n\t"
      "buffer_load_dwordx4 v13, s[28:31], s56 offen offset:1024 lds\n\t"
      "buffer_load_dwordx4 v14, s[28:31], s56 offen offset:2048 lds\n\t"
      "buffer_load_dwordx4 v15, s[28:31], s56 offen offset:3072 lds\n\t"
      "buffer_load_dwordx4 a[80:83], v7, s[32:35], s63 offen sc0 sc1\n\t"
      "buffer_load_dwordx4 a[84:87], v3, s[32:35], s63 offen sc0 sc1\n\t"
      "buffer_load_dwordx4 a[88:91], v7, s[32:35], s63 offen offset:2048 sc0 sc1\n\t"
      "buffer_load_dwordx4 a[92:95], v3, s[32:35], s63 offen offset:2048 sc0 sc1\n\t"
      "s_add_u32 s63, s63, 0x1000\n\t"
      "buffer_load_dwordx4 a[96:99], v7, s[32:35], s63 offen sc0 sc1\n\t"
      "buffer_load_dwordx4 a[100:103], v3, s[32:35], s63 offen sc0 sc1\n\t"
      "buffer_load_dwordx4 a[104:107], v7, s[32:35], s63 offen offset:2048 sc0 sc1\n\t"
      "buffer_load_dwordx4 a[108:111], v3, s[32:35], s63 offen offset:2048 sc0 sc1\n\t"
      "s_add_u32 s63, s63, 0x1000\n\t"
      "buffer_load_dwordx4 a[112:115], v8, s[32:35], s63 offen sc0 sc1\n\t"
      "buffer_load_dwordx4 a[116:119], v4, s[32:35], s63 offen sc0 sc1\n\t"
      "v_accvgpr_read_b32 v11, a184\n\t"
      "v_accvgpr_read_b32 v26, a185\n\t"
      "v_accvgpr_write_b32 a184, v192\n\t"
      "v_accvgpr_write_b32 a185, v193\n\t"
      "v_mov_b32 v192, v11\n\t"
      "v_mov_b32 v193, v26\n\t"
      "v_accvgpr_read_b32 v11, a186\n\t"
      "v_accvgpr_read_b32 v26, a187\n\t"
      "v_accvgpr_write_b32 a186, v194\n\t"
      "v_accvgpr_write_b32 a187, v195\n\t"
      "v_mov_b32 v194, v11\n\t"
      "v_mov_b32 v195, v26\n\t"
      "v_accvgpr_read_b32 v11, a192\n\t"
      "v_accvgpr_read_b32 v26, a193\n\t"
      "v_accvgpr_write_b32 a192, v200\n\t"
      "v_accvgpr_write_b32 a193, v201\n\t"
      "v_mov_b32 v200, v11\n\t"
      "v_mov_b32 v201, v26\n\t"
      "v_accvgpr_read_b32 v11, a194\n\t"
      "v_accvgpr_read_b32 v26, a195\n\t"
      "v_accvgpr_write_b32 a194, v202\n\t"
      "v_accvgpr_write_b32 a195, v203\n\t"
      "v_mov_b32 v202, v11\n\t"
      "v_mov_b32 v203, v26\n\t"
      "v_accvgpr_read_b32 v11, a200\n\t"
      "v_accvgpr_read_b32 v26, a201\n\t"
      "v_accvgpr_write_b32 a200, v208\n\t"
      "v_accvgpr_write_b32 a201, v209\n\t"
      "v_mov_b32 v208, v11\n\t"
      "v_mov_b32 v209, v26\n\t"
      "v_accvgpr_read_b32 v11, a202\n\t"
      "v_accvgpr_read_b32 v26, a203\n\t"
      "v_accvgpr_write_b32 a202, v210\n\t"
      "v_accvgpr_write_b32 a203, v211\n\t"
      "v_mov_b32 v210, v11\n\t"
      "v_mov_b32 v211, v26\n\t"
      "v_accvgpr_read_b32 v11, a208\n\t"
      "v_accvgpr_read_b32 v26, a209\n\t"
      "v_accvgpr_write_b32 a208, v216\n\t"
      "v_accvgpr_write_b32 a209, v217\n\t"
      "v_mov_b32 v216, v11\n\t"
      "v_mov_b32 v217, v26\n\t"
      "v_accvgpr_read_b32 v11, a210\n\t"
      "v_accvgpr_read_b32 v26, a211\n\t"
      "v_accvgpr_write_b32 a210, v218\n\t"
      "v_accvgpr_write_b32 a211, v219\n\t"
      "v_mov_b32 v218, v11\n\t"
      "v_mov_b32 v219, v26\n\t"
      "v_accvgpr_read_b32 v11, a216\n\t"
      "v_accvgpr_read_b32 v26, a217\n\t"
      "v_accvgpr_write_b32 a216, v224\n\t"
      "v_accvgpr_write_b32 a217, v225\n\t"
      "v_mov_b32 v224, v11\n\t"
      "v_mov_b32 v225, v26\n\t"
      "v_accvgpr_read_b32 v11, a218\n\t"
      "v_accvgpr_read_b32 v26, a219\n\t"
      "v_accvgpr_write_b32 a218, v226\n\t"
      "v_accvgpr_write_b32 a219, v227\n\t"
      "v_mov_b32 v226, v11\n\t"
      "v_mov_b32 v227, v26\n\t"
      "v_accvgpr_read_b32 v11, a224\n\t"
      "v_accvgpr_read_b32 v26, a225\n\t"
      "v_accvgpr_write_b32 a224, v232\n\t"
      "v_accvgpr_write_b32 a225, v233\n\t"
      "v_mov_b32 v232, v11\n\t"
      "v_mov_b32 v233, v26\n\t"
      "v_accvgpr_read_b32 v11, a226\n\t"
      "v_accvgpr_read_b32 v26, a227\n\t"
      "v_accvgpr_write_b32 a226, v234\n\t"
      "v_accvgpr_write_b32 a227, v235\n\t"
      "v_mov_b32 v234, v11\n\t"
      "v_mov_b32 v235, v26\n\t"
      "v_accvgpr_read_b32 v11, a232\n\t"
      "v_accvgpr_read_b32 v26, a233\n\t"
      "v_accvgpr_write_b32 a232, v240\n\t"
      "v_accvgpr_write_b32 a233, v241\n\t"
      "v_mov_b32 v240, v11\n\t"
      "v_mov_b32 v241, v26\n\t"
      "v_accvgpr_read_b32 v11, a234\n\t"
      "v_accvgpr_read_b32 v26, a235\n\t"
      "v_accvgpr_write_b32 a234, v242\n\t"
      "v_accvgpr_write_b32 a235, v243\n\t"
      "v_mov_b32 v242, v11\n\t"
      "v_mov_b32 v243, v26\n\t"
      "v_accvgpr_read_b32 v11, a240\n\t"
      "v_accvgpr_read_b32 v26, a241\n\t"
      "v_accvgpr_write_b32 a240, v248\n\t"
      "v_accvgpr_write_b32 a241, v249\n\t"
      "v_mov_b32 v248, v11\n\t"
      "v_mov_b32 v249, v26\n\t"
      "v_accvgpr_read_b32 v11, a242\n\t"
      "v_accvgpr_read_b32 v26, a243\n\t"
      "v_accvgpr_write_b32 a242, v250\n\t"
      "v_accvgpr_write_b32 a243, v251\n\t"
      "v_mov_b32 v250, v11\n\t"
      "v_mov_b32 v251, v26\n\t"
      "s_waitcnt lgkmcnt(0)\n\t"
      "s_nop 1\n\t"
      "s_branch ALD5_%=\n\t"
      "ALN56_%=:\n\t"
      "s_add_u32 m0, s51, 0\n\t"
      "s_xor_b32 s56, s57, 0x0\n\t"
      "buffer_load_dwordx4 v12, s[28:31], s56 offen lds\n\t"
      "buffer_load_dwordx4 v13, s[28:31], s56 offen offset:1024 lds\n\t"
      "buffer_load_dwordx4 v14, s[28:31], s56 offen offset:2048 lds\n\t"
      "buffer_load_dwordx4 v15, s[28:31], s56 offen offset:3072 lds\n\t"
      "s_add_u32 m0, s51, 4096\n\t"
      "s_xor_b32 s56, s57, 0x20000\n\t"
      "buffer_load_dwordx4 v12, s[28:31], s56 offen lds\n\t"
      "buffer_load_dwordx4 v13, s[28:31], s56 offen offset:1024 lds\n\t"
      "buffer_load_dwordx4 v14, s[28:31], s56 offen offset:2048 lds\n\t"
      "buffer_load_dwordx4 v15, s[28:31], s56 offen offset:3072 lds\n\t"
      "s_add_u32 m0, s51, 8192\n\t"
      "s_xor_b32 s56, s57, 0x40000\n\t"
      "buffer_load_dwordx4 v12, s[28:31], s56 offen lds\n\t"
      "buffer_load_dwordx4 v13, s[28:31], s56 offen offset:1024 lds\n\t"
      "buffer_load_dwordx4 v14, s[28:31], s56 offen offset:2048 lds\n\t"
      "buffer_load_dwordx4 v15, s[28:31], s56 offen offset:3072 lds\n\t"
      "s_add_u32 m0, s51, 12288\n\t"
      "s_xor_b32 s56, s57, 0x60000\n\t"
      "buffer_load_dwordx4 v12, s[28:31], s56 offen lds\n\t"
      "buffer_load_dwordx4 v13, s[28:31], s56 offen offset:1024 lds\n\t"
      "buffer_load_dwordx4 v14, s[28:31], s56 offen offset:2048 lds\n\t"
      "buffer_load_dwordx4 v15, s[28:31], s56 offen offset:3072 lds\n\t"
      "buffer_load_dwordx4 a[80:83], v7, s[32:35], s63 offen sc0 sc1\n\t"
      "buffer_load_dwordx4 a[84:87], v3, s[32:35], s63 offen sc0 sc1\n\t"
      "buffer_load_dwordx4 a[88:91], v7, s[32:35], s63 offen offset:2048 sc0 sc1\n\t"
      "buffer_load_dwordx4 a[92:95], v3, s[32:35], s63 offen offset:2048 sc0 sc1\n\t"
      "s_add_u32 s63, s63, 0x1000\n\t"
      "buffer_load_dwordx4 a[96:99], v7, s[32:35], s63 offen sc0 sc1\n\t"
      "buffer_load_dwordx4 a[100:103], v3, s[32:35], s63 offen sc0 sc1\n\t"
      "buffer_load_dwordx4 a[104:107], v7, s[32:35], s63 offen offset:2048 sc0 sc1\n\t"
      "buffer_load_dwordx4 a[108:111], v3, s[32:35], s63 offen offset:2048 sc0 sc1\n\t"
      "s_add_u32 s63, s63, 0x1000\n\t"
      "buffer_load_dwordx4 a[112:115], v8, s[32:35], s63 offen sc0 sc1\n\t"
      "buffer_load_dwordx4 a[116:119], v4, s[32:35], s63 offen sc0 sc1\n\t"
      "s_waitcnt vmcnt(18)\n\t"
      "ds_read_b64_tr_b8 a[184:185], v16 offset:0\n\t"
      "ds_read_b64_tr_b8 a[186:187], v16 offset:1024\n\t"
      "ds_read_b64_tr_b8 a[192:193], v17 offset:0\n\t"
      "ds_read_b64_tr_b8 a[194:195], v17 offset:1024\n\t"
      "ds_read_b64_tr_b8 a[200:201], v18 offset:0\n\t"
      "ds_read_b64_tr_b8 a[202:203], v18 offset:1024\n\t"
      "ds_read_b64_tr_b8 a[208:209], v19 offset:0\n\t"
      "ds_read_b64_tr_b8 a[210:211], v19 offset:1024\n\t"
      "ds_read_b64_tr_b8 a[216:217], v20 offset:0\n\t"
      "ds_read_b64_tr_b8 a[218:219], v20 offset:1024\n\t"
      "ds_read_b64_tr_b8 a[224:225], v21 offset:0\n\t"
      "ds_read_b64_tr_b8 a[226:227], v21 offset:1024\n\t"
      "ds_read_b64_tr_b8 a[232:233], v22 offset:0\n\t"
      "ds_read_b64_tr_b8 a[234:235], v22 offset:1024\n\t"
      "ds_read_b64_tr_b8 a[240:241], v23 offset:0\n\t"
      "ds_read_b64_tr_b8 a[242:243], v23 offset:1024\n\t"
      "s_waitcnt vmcnt(10)\n\t"
      "ds_read_b64_tr_b8 a[188:189], v16 offset:8192\n\t"
      "ds_read_b64_tr_b8 a[190:191], v16 offset:9216\n\t"
      "ds_read_b64_tr_b8 a[196:197], v17 offset:8192\n\t"
      "ds_read_b64_tr_b8 a[198:199], v17 offset:9216\n\t"
      "ds_read_b64_tr_b8 a[204:205], v18 offset:8192\n\t"
      "ds_read_b64_tr_b8 a[206:207], v18 offset:9216\n\t"
      "ds_read_b64_tr_b8 a[212:213], v19 offset:8192\n\t"
      "ds_read_b64_tr_b8 a[214:215], v19 offset:9216\n\t"
      "ds_read_b64_tr_b8 a[220:221], v20 offset:8192\n\t"
      "ds_read_b64_tr_b8 a[222:223], v20 offset:9216\n\t"
      "ds_read_b64_tr_b8 a[228:229], v21 offset:8192\n\t"
      "ds_read_b64_tr_b8 a[230:231], v21 offset:9216\n\t"
      "ds_read_b64_tr_b8 a[236:237], v22 offset:8192\n\t"
      "ds_read_b64_tr_b8 a[238:239], v22 offset:9216\n\t"
      "ds_read_b64_tr_b8 a[244:245], v23 offset:8192\n\t"
      "ds_read_b64_tr_b8 a[246:247], v23 offset:9216\n\t"
      "s_waitcnt lgkmcnt(0)\n\t"
      "s_branch ALD5_%=\n\t"
      "ALT6_%=:\n\t"
      "s_mov_b32 s21, 16384\n\t"
      "s_mov_b32 s58, 1\n\t"
      "v_mfma_f32_16x16x128_f8f6f4 v[108:111], a[96:103], a[208:215], v[108:111]\n\t"
      "v_mfma_f32_16x16x128_f8f6f4 v[112:115], a[96:103], a[216:223], v[112:115]\n\t"
      "s_call_b64 s[66:67], SETC_%=\n\t"
      "v_mfma_f32_16x16x128_f8f6f4 v[116:119], a[96:103], a[224:231], v[116:119]\n\t"
      "v_mfma_f32_16x16x128_f8f6f4 v[120:123], a[96:103], a[232:239], v[120:123]\n\t"
      "v_mfma_f32_16x16x128_f8f6f4 v[124:127], a[96:103], a[240:247], v[124:127]\n\t"
      "v_mfma_f32_16x16x128_f8f6f4 v[128:131], a[104:111], a[184:191], v[128:131]\n\t"
      "v_mfma_f32_16x16x128_f8f6f4 v[132:135], a[104:111], a[192:199], v[132:135]\n\t"
      "v_mfma_f32_16x16x128_f8f6f4 v[136:139], a[104:111], a[200:207], v[136:139]\n\t"
      "v_mfma_f32_16x16x128_f8f6f4 v[140:143], a[104:111], a[208:215], v[140:143]\n\t"
      "v_mfma_f32_16x16x128_f8f6f4 v[144:147], a[104:111], a[216:223], v[144:147]\n\t"
      "v_mfma_f32_16x16x128_f8f6f4 v[148:151], a[104:111], a[224:231], v[148:151]\n\t"
      "v_mfma_f32_16x16x128_f8f6f4 v[152:155], a[104:111], a[232:239], v[152:155]\n\t"
      "v_mfma_f32_16x16x128_f8f6f4 v[156:159], a[104:111], a[240:247], v[156:159]\n\t"
      "v_mfma_f32_16x16x128_f8f6f4 v[160:163], a[112:119], a[184:191], v[160:163]\n\t"
      "v_mfma_f32_16x16x128_f8f6f4 v[164:167], a[112:119], a[192:199], v[164:167]\n\t"
      "v_mfma_f32_16x16x128_f8f6f4 v[168:171], a[112:119], a[200:207], v[168:171]\n\t"
      "v_mfma_f32_16x16x128_f8f6f4 v[172:175], a[112:119], a[208:215], v[172:175]\n\t"
      "v_mfma_f32_16x16x128_f8f6f4 v[176:179], a[112:119], a[216:223], v[176:179]\n\t"
      "v_mfma_f32_16x16x128_f8f6f4 v[180:183], a[112:119], a[224:231], v[180:183]\n\t"
      "v_mfma_f32_16x16x128_f8f6f4 v[184:187], a[112:119], a[232:239], v[184:187]\n\t"
      "v_mfma_f32_16x16x128_f8f6f4 v[188:191], a[112:119], a[240:247], v[188:191]\n\t"
      "s_call_b64 s[66:67], SPIN1_%=\n\t"
      "s_cmp_lg_u32 s21, 0\n\t"
      "s_cbranch_scc1 ALS6_%=\n\t"
      "buffer_load_dwordx4 a[0:3], v7, s[32:35], s63 offen sc0 sc1\n\t"
      "buffer_load_dwordx4 a[4:7], v3, s[32:35], s63 offen sc0 sc1\n\t"
      "buffer_load_dwordx4 a[8:11], v7, s[32:35], s63 offen offset:2048 sc0 sc1\n\t"
      "buffer_load_dwordx4 a[12:15], v3, s[32:35], s63 offen offset:2048 sc0 sc1\n\t"
      "s_and_b32 vcc_lo, vcc_lo, 2\n\t"
      "s_cbranch_scc0 ALB6_%=\n\t"
      "ALC6_%=:\n\t"
      "s_add_u32 s63, s63, 0x1000\n\t"
      "buffer_load_dwordx4 a[16:19], v7, s[32:35], s63 offen sc0 sc1\n\t"
      "buffer_load_dwordx4 a[20:23], v3, s[32:35], s63 offen sc0 sc1\n\t"
      "buffer_load_dwordx4 a[24:27], v7, s[32:35], s63 offen offset:2048 sc0 sc1\n\t"
      "buffer_load_dwordx4 a[28:31], v3, s[32:35], s63 offen offset:2048 sc0 sc1\n\t"
      "s_add_u32 s63, s63, 0x1000\n\t"
      "buffer_load_dwordx4 a[32:35], v8, s[32:35], s63 offen sc0 sc1\n\t"
      "buffer_load_dwordx4 a[36:39], v4, s[32:35], s63 offen sc0 sc1\n\t"
      "ALD6_%=:\n\t"
      "s_branch JOIN6_%=\n\t"
      "ALB6_%=:\n\t"
      "s_call_b64 s[66:67], SPINB_%=\n\t"
      "s_branch ALC6_%=\n\t"
      "ALS6_%=:\n\t"
      "s_mov_b32 s56, 48\n\t"
      "s_call_b64 s[66:67], SWAP_%=\n\t"
      "s_waitcnt lgkmcnt(0)\n\t"
      "s_add_u32 m0, s51, 0\n\t"
      "s_xor_b32 s56, s57, 0x40000\n\t"
      "buffer_load_dwordx4 v12, s[28:31], s56 offen lds\n\t"
      "buffer_load_dwordx4 v13, s[28:31], s56 offen offset:1024 lds\n\t"
      "buffer_load_dwordx4 v14, s[28:31], s56 offen offset:2048 lds\n\t"
      "buffer_load_dwordx4 v15, s[28:31], s56 offen offset:3072 lds\n\t"
      "s_add_u32 m0, s51, 4096\n\t"
      "s_xor_b32 s56, s57, 0x60000\n\t"
      "buffer_load_dwordx4 v12, s[28:31], s56 offen lds\n\t"
      "buffer_load_dwordx4 v13, s[28:31], s56 offen offset:1024 lds\n\t"
      "buffer_load_dwordx4 v14, s[28:31], s56 offen offset:2048 lds\n\t"
      "buffer_load_dwordx4 v15, s[28:31], s56 offen offset:3072 lds\n\t"
      "ds_read_b64_tr_b8 v[192:193], v16 offset:32768\n\t"
      "ds_read_b64_tr_b8 v[194:195], v16 offset:33792\n\t"
      "ds_read_b64_tr_b8 v[200:201], v17 offset:32768\n\t"
      "ds_read_b64_tr_b8 v[202:203], v17 offset:33792\n\t"
      "ds_read_b64_tr_b8 v[208:209], v18 offset:32768\n\t"
      "ds_read_b64_tr_b8 v[210:211], v18 offset:33792\n\t"
      "ds_read_b64_tr_b8 v[216:217], v19 offset:32768\n\t"
      "ds_read_b64_tr_b8 v[218:219], v19 offset:33792\n\t"
      "ds_read_b64_tr_b8 v[224:225], v20 offset:32768\n\t"
      "ds_read_b64_tr_b8 v[226:227], v20 offset:33792\n\t"
      "ds_read_b64_tr_b8 v[232:233], v21 offset:32768\n\t"
      "ds_read_b64_tr_b8 v[234:235], v21 offset:33792\n\t"
      "ds_read_b64_tr_b8 v[240:241], v22 offset:32768\n\t"
      "ds_read_b64_tr_b8 v[242:243], v22 offset:33792\n\t"
      "ds_read_b64_tr_b8 v[248:249], v23 offset:32768\n\t"
      "ds_read_b64_tr_b8 v[250:251], v23 offset:33792\n\t"
      "s_waitcnt lgkmcnt(0)\n\t"
      "s_add_u32 m0, s51, 32768\n\t"
      "s_xor_b32 s56, s19, 0x0\n\t"
      "buffer_load_dwordx4 v12, s[28:31], s56 offen lds\n\t"
      "buffer_load_dwordx4 v13, s[28:31], s56 offen offset:1024 lds\n\t"
      "buffer_load_dwordx4 v14, s[28:31], s56 offen offset:2048 lds\n\t"
      "buffer_load_dwordx4 v15, s[28:31], s56 offen offset:3072 lds\n\t"
      "s_add_u32 m0, s51, 36864\n\t"
      "s_xor_b32 s56, s19, 0x20000\n\t"
      "buffer_load_dwordx4 v12, s[28:31], s56 offen lds\n\t"
      "buffer_load_dwordx4 v13, s[28:31], s56 offen offset:1024 lds\n\t"
      "buffer_load_dwordx4 v14, s[28:31], s56 offen offset:2048 lds\n\t"
      "buffer_load_dwordx4 v15, s[28:31], s56 offen offset:3072 lds\n\t"
      "buffer_load_dwordx4 a[0:3], v7, s[32:35], s63 offen sc0 sc1\n\t"
      "buffer_load_dwordx4 a[4:7], v3, s[32:35], s63 offen sc0 sc1\n\t"
      "buffer_load_dwordx4 a[8:11], v7, s[32:35], s63 offen offset:2048 sc0 sc1\n\t"
      "buffer_load_dwordx4 a[12:15], v3, s[32:35], s63 offen offset:2048 sc0 sc1\n\t"
      "s_add_u32 s63, s63, 0x1000\n\t"
      "buffer_load_dwordx4 a[16:19], v7, s[32:35], s63 offen sc0 sc1\n\t"
      "buffer_load_dwordx4 a[20:23], v3, s[32:35], s63 offen sc0 sc1\n\t"
      "buffer_load_dwordx4 a[24:27], v7, s[32:35], s63 offen offset:2048 sc0 sc1\n\t"
      "buffer_load_dwordx4 a[28:31], v3, s[32:35], s63 offen offset:2048 sc0 sc1\n\t"
      "s_add_u32 s63, s63, 0x1000\n\t"
      "buffer_load_dwordx4 a[32:35], v8, s[32:35], s63 offen sc0 sc1\n\t"
      "buffer_load_dwordx4 a[36:39], v4, s[32:35], s63 offen sc0 sc1\n\t"
      "s_waitcnt vmcnt(10)\n\t"
      "ds_read_b64_tr_b8 v[196:197], v16 offset:0\n\t"
      "ds_read_b64_tr_b8 v[198:199], v16 offset:1024\n\t"
      "ds_read_b64_tr_b8 v[204:205], v17 offset:0\n\t"
      "ds_read_b64_tr_b8 v[206:207], v17 offset:1024\n\t"
      "ds_read_b64_tr_b8 v[212:213], v18 offset:0\n\t"
      "ds_read_b64_tr_b8 v[214:215], v18 offset:1024\n\t"
      "ds_read_b64_tr_b8 v[220:221], v19 offset:0\n\t"
      "ds_read_b64_tr_b8 v[222:223], v19 offset:1024\n\t"
      "ds_read_b64_tr_b8 v[228:229], v20 offset:0\n\t"
      "ds_read_b64_tr_b8 v[230:231], v20 offset:1024\n\t"
      "ds_read_b64_tr_b8 v[236:237], v21 offset:0\n\t"
      "ds_read_b64_tr_b8 v[238:239], v21 offset:1024\n\t"
      "ds_read_b64_tr_b8 v[244:245], v22 offset:0\n\t"
      "ds_read_b64_tr_b8 v[246:247], v22 offset:1024\n\t"
      "ds_read_b64_tr_b8 v[252:253], v23 offset:0\n\t"
      "ds_read_b64_tr_b8 v[254:255], v23 offset:1024\n\t"
      "s_waitcnt lgkmcnt(0)\n\t"
      "s_nop 1\n\t"
      "s_branch ALD6_%=\n\t"
      "ALN67_%=:\n\t"
      "ALT7_%=:\n\t"
      "s_mov_b32 s21, 0\n\t"
      "s_mov_b32 s58, 1\n\t"
      "v_mfma_f32_16x16x128_f8f6f4 v[108:111], a[16:23], v[216:223], v[108:111]\n\t"
      "v_mfma_f32_16x16x128_f8f6f4 v[112:115], a[16:23], v[224:231], v[112:115]\n\t"
      "s_call_b64 s[66:67], SETC_%=\n\t"
      "v_mfma_f32_16x16x128_f8f6f4 v[116:119], a[16:23], v[232:239], v[116:119]\n\t"
      "v_mfma_f32_16x16x128_f8f6f4 v[120:123], a[16:23], v[240:247], v[120:123]\n\t"
      "v_mfma_f32_16x16x128_f8f6f4 v[124:127], a[16:23], v[248:255], v[124:127]\n\t"
      "v_mfma_f32_16x16x128_f8f6f4 v[128:131], a[24:31], v[192:199], v[128:131]\n\t"
      "v_mfma_f32_16x16x128_f8f6f4 v[132:135], a[24:31], v[200:207], v[132:135]\n\t"
      "v_mfma_f32_16x16x128_f8f6f4 v[136:139], a[24:31], v[208:215], v[136:139]\n\t"
      "v_mfma_f32_16x16x128_f8f6f4 v[140:143], a[24:31], v[216:223], v[140:143]\n\t"
      "v_mfma_f32_16x16x128_f8f6f4 v[144:147], a[24:31], v[224:231], v[144:147]\n\t"
      "v_mfma_f32_16x16x128_f8f6f4 v[148:151], a[24:31], v[232:239], v[148:151]\n\t"
      "v_mfma_f32_16x16x128_f8f6f4 v[152:155], a[24:31], v[240:247], v[152:155]\n\t"
      "s_waitcnt vmcnt(1)\n\t"
      "ds_read_b64_tr_b8 a[124:125], v16 offset:0\n\t"
      "ds_read_b64_tr_b8 a[126:127], v16 offset:1024\n\t"
      "v_mfma_f32_16x16x128_f8f6f4 v[156:159], a[24:31], v[248:255], v[156:159]\n\t"
      "ds_read_b64_tr_b8 a[132:133], v17 offset:0\n\t"
      "ds_read_b64_tr_b8 a[134:135], v17 offset:1024\n\t"
      "v_mfma_f32_16x16x128_f8f6f4 v[160:163], a[32:39], v[192:199], v[160:163]\n\t"
      "ds_read_b64_tr_b8 a[140:141], v18 offset:0\n\t"
      "ds_read_b64_tr_b8 a[142:143], v18 offset:1024\n\t"
      "v_mfma_f32_16x16x128_f8f6f4 v[164:167], a[32:39], v[200:207], v[164:167]\n\t"
      "ds_read_b64_tr_b8 a[148:149], v19 offset:0\n\t"
      "ds_read_b64_tr_b8 a[150:151], v19 offset:1024\n\t"
      "v_mfma_f32_16x16x128_f8f6f4 v[168:171], a[32:39], v[208:215], v[168:171]\n\t"
      "ds_read_b64_tr_b8 a[156:157], v20 offset:0\n\t"
      "ds_read_b64_tr_b8 a[158:159], v20 offset:1024\n\t"
      "v_mfma_f32_16x16x128_f8f6f4 v[172:175], a[32:39], v[216:223], v[172:175]\n\t"
      "ds_read_b64_tr_b8 a[164:165], v21 offset:0\n\t"
      "ds_read_b64_tr_b8 a[166:167], v21 offset:1024\n\t"
      "v_mfma_f32_16x16x128_f8f6f4 v[176:179], a[32:39], v[224:231], v[176:179]\n\t"
      "ds_read_b64_tr_b8 a[172:173], v22 offset:0\n\t"
      "ds_read_b64_tr_b8 a[174:175], v22 offset:1024\n\t"
      "v_mfma_f32_16x16x128_f8f6f4 v[180:183], a[32:39], v[232:239], v[180:183]\n\t"
      "ds_read_b64_tr_b8 a[180:181], v23 offset:0\n\t"
      "ds_read_b64_tr_b8 a[182:183], v23 offset:1024\n\t"
      "v_mfma_f32_16x16x128_f8f6f4 v[184:187], a[32:39], v[240:247], v[184:187]\n\t"
      "v_mfma_f32_16x16x128_f8f6f4 v[188:191], a[32:39], v[248:255], v[188:191]\n\t"
      "s_call_b64 s[66:67], SPIN1_%=\n\t"
      "buffer_load_dword v29, off, s[44:47], 0 offset:0 sc1\n\t"
      "buffer_load_dwordx4 a[40:43], v7, s[32:35], s63 offen sc0 sc1\n\t"
      "buffer_load_dwordx4 a[44:47], v3, s[32:35], s63 offen sc0 sc1\n\t"
      "buffer_load_dwordx4 a[48:51], v7, s[32:35], s63 offen offset:2048 sc0 sc1\n\t"
      "buffer_load_dwordx4 a[52:55], v3, s[32:35], s63 offen offset:2048 sc0 sc1\n\t"
      "s_and_b32 vcc_lo, vcc_lo, 2\n\t"
      "s_cbranch_scc0 ALB7_%=\n\t"
      "ALC7_%=:\n\t"
      "s_add_u32 s63, s63, 0x1000\n\t"
      "buffer_load_dwordx4 a[56:59], v7, s[32:35], s63 offen sc0 sc1\n\t"
      "buffer_load_dwordx4 a[60:63], v3, s[32:35], s63 offen sc0 sc1\n\t"
      "buffer_load_dwordx4 a[64:67], v7, s[32:35], s63 offen offset:2048 sc0 sc1\n\t"
      "buffer_load_dwordx4 a[68:71], v3, s[32:35], s63 offen offset:2048 sc0 sc1\n\t"
      "s_add_u32 s63, s63, 0x1000\n\t"
      "buffer_load_dwordx4 a[72:75], v8, s[32:35], s63 offen sc0 sc1\n\t"
      "buffer_load_dwordx4 a[76:79], v4, s[32:35], s63 offen sc0 sc1\n\t"
      "ALD7_%=:\n\t"
      "s_branch JOIN7_%=\n\t"
      "ALB7_%=:\n\t"
      "s_call_b64 s[66:67], SPINB_%=\n\t"
      "s_branch ALC7_%=\n\t"
      "SETC_%=:\n\t"
      "v_bfrev_b32 v30, 1\n\t"
      "s_lshl_b32 s56, s21, 1\n\t"
      "s_or_b32 exec_lo, s56, s21\n\t"
      "s_mov_b32 exec_hi, 0\n\t"
      "v_lshlrev_b32 v11, 2, v2\n\t"
      "v_and_b32 v11, 0x38, v11\n\t"
      "v_lshrrev_b64 v[28:29], v11, s[24:25]\n\t"
      "v_and_b32 v11, 3, v28\n\t"
      "v_mul_u32_u24 v11, 10240, v11\n\t"
      "v_bfe_u32 v26, v28, 2, 3\n\t"
      "v_lshl_add_u32 v11, v26, 20, v11\n\t"
      "v_add_u32 v11, s60, v11\n\t"
      "v_and_b32 v26, 1, v2\n\t"
      "v_lshl_add_u32 v11, v26, 7, v11\n\t"
      "v_add_u32 v30, 8960, v11\n\t"
      "s_mov_b32 exec_lo, 3\n\t"
      "v_add_u32 v30, s62, v9\n\t"
      "s_mov_b64 exec, -1\n\t"
      "buffer_load_dword v31, v30, s[32:35], 0 offen sc0 sc1\n\t"
      "s_setpc_b64 s[66:67]\n\t"
      "SPINB_%=:\n\t"
      "s_mov_b32 s58, 2\n\t"
      "s_branch SPINW_%=\n\t"
      "SPIN1_%=:\n\t"
      "s_waitcnt vmcnt(0)\n\t"
      "v_cmp_le_u32 vcc, s59, v31\n\t"
      "s_nop 0\n\t"
      "s_andn2_b32 s57, s58, vcc_lo\n\t"
      "s_cbranch_scc0 SPINR_%=\n\t"
      "s_lshr_b32 s57, vcc_lo, 1\n\t"
      "s_and_b32 s57, s57, vcc_lo\n\t"
      "s_and_b32 s57, s57, s21\n\t"
      "s_cbranch_scc1 SPINR_%=\n\t"
      "SPINW_%=:\n\t"
      "s_cmp_lg_u32 s61, 0\n\t"
      "s_cbranch_scc1 SPIND_%=\n\t"
      "SPINL_%=:\n\t"
      "s_mov_b32 s56, 8\n\t"
      "buffer_load_dword v31, v30, s[32:35], 0 offen sc0 sc1\n\t"
      "s_sleep 1\n\t"
      "buffer_load_dword v5, v30, s[32:35], 0 offen sc0 sc1\n\t"
      "SPINF_%=:\n\t"
      "s_sleep 1\n\t"
      "buffer_load_dword v6, v30, s[32:35], 0 offen sc0 sc1\n\t"
      "s_waitcnt vmcnt(2)\n\t"
      "v_cmp_le_u32 vcc, s59, v31\n\t"
      "s_sub_u32 s56, s56, 1\n\t"
      "s_andn2_b32 s57, s58, vcc_lo\n\t"
      "s_cbranch_scc0 SPINR_%=\n\t"
      "s_lshr_b32 s57, vcc_lo, 1\n\t"
      "s_and_b32 s57, s57, vcc_lo\n\t"
      "s_and_b32 s57, s57, s21\n\t"
      "s_cbranch_scc1 SPINR_%=\n\t"
      "s_sleep 1\n\t"
      "buffer_load_dword v31, v30, s[32:35], 0 offen sc0 sc1\n\t"
      "s_waitcnt vmcnt(2)\n\t"
      "v_cmp_le_u32 vcc, s59, v5\n\t"
      "s_nop 0\n\t"
      "s_andn2_b32 s57, s58, vcc_lo\n\t"
      "s_cbranch_scc0 SPINR_%=\n\t"
      "s_lshr_b32 s57, vcc_lo, 1\n\t"
      "s_and_b32 s57, s57, vcc_lo\n\t"
      "s_and_b32 s57, s57, s21\n\t"
      "s_cbranch_scc1 SPINR_%=\n\t"
      "s_sleep 1\n\t"
      "buffer_load_dword v5, v30, s[32:35], 0 offen sc0 sc1\n\t"
      "s_waitcnt vmcnt(2)\n\t"
      "v_cmp_le_u32 vcc, s59, v6\n\t"
      "s_nop 0\n\t"
      "s_andn2_b32 s57, s58, vcc_lo\n\t"
      "s_cbranch_scc0 SPINR_%=\n\t"
      "s_lshr_b32 s57, vcc_lo, 1\n\t"
      "s_and_b32 s57, s57, vcc_lo\n\t"
      "s_and_b32 s57, s57, s21\n\t"
      "s_cbranch_scc1 SPINR_%=\n\t"
      "s_cmp_lg_u32 s56, 0\n\t"
      "s_cbranch_scc1 SPINF_%=\n\t"
      "v_mov_b32 v28, 0\n\t"
      "buffer_load_dword v31, v30, s[32:35], 0 offen sc0 sc1\n\t"
      "buffer_load_dword v29, v28, s[44:47], 0 offen sc1\n\t"
      "s_memrealtime s[70:71]\n\t"
      "s_waitcnt vmcnt(0) lgkmcnt(0)\n\t"
      "v_cmp_le_u32 vcc, s59, v31\n\t"
      "v_readfirstlane_b32 s56, v29\n\t"
      "s_sub_u32 s70, s70, s68\n\t"
      "s_andn2_b32 s57, s58, vcc_lo\n\t"
      "s_cbranch_scc0 SPINR_%=\n\t"
      "s_lshr_b32 s57, vcc_lo, 1\n\t"
      "s_and_b32 s57, s57, vcc_lo\n\t"
      "s_and_b32 s57, s57, s21\n\t"
      "s_cbranch_scc1 SPINR_%=\n\t"
      "s_cmp_eq_u32 s56, s59\n\t"
      "s_cbranch_scc1 SPINT_%=\n\t"
      "s_cmp_lt_u32 s70, 50000000\n\t"
      "s_cbranch_scc1 SPINL_%=\n\t"
      "SPINT_%=:\n\t"
      "v_mov_b32 v29, s59\n\t"
      "s_mov_b32 s61, 1\n\t"
      "buffer_store_dword v29, v28, s[44:47], 0 offen sc1\n\t"
      "SPIND_%=:\n\t"
      "s_waitcnt vmcnt(0)\n\t"
      "s_mov_b32 s57, 0\n\t"
      "SPINR_%=:\n\t"
      "s_mov_b32 s21, s57\n\t"
      "s_setpc_b64 s[66:67]\n\t"
      "SWAP_%=:\n\t"
      "s_ff1_i32_b32 s57, s21\n\t"
      "s_lshl_b32 s57, s57, 2\n\t"
      "s_lshr_b64 s[16:17], s[24:25], s56\n\t"
      "s_lshr_b64 s[18:19], s[24:25], s57\n\t"
      "s_and_b32 s19, s16, 31\n\t"
      "s_lshl_b32 s19, s19, 19\n\t"
      "s_xor_b32 s19, s19, s26\n\t"
      "s_xor_b32 s16, s16, s18\n\t"
      "s_and_b32 s16, s16, 0xff\n\t"
      "s_mov_b32 s17, 0\n\t"
      "s_lshl_b64 s[70:71], s[16:17], s56\n\t"
      "s_xor_b64 s[24:25], s[24:25], s[70:71]\n\t"
      "s_lshl_b64 s[70:71], s[16:17], s57\n\t"
      "s_xor_b64 s[24:25], s[24:25], s[70:71]\n\t"
      "s_and_b32 s63, s18, 3\n\t"
      "s_mul_i32 s63, s63, 10240\n\t"
      "s_bfe_u32 s16, s18, 0x30002\n\t"
      "s_lshl_b32 s16, s16, 20\n\t"
      "s_add_u32 s63, s63, s16\n\t"
      "s_add_u32 s63, s63, s60\n\t"
      "s_and_b32 s57, s18, 31\n\t"
      "s_lshl_b32 s57, s57, 19\n\t"
      "s_xor_b32 s57, s57, s26\n\t"
      "s_setpc_b64 s[66:67]\n\t"
      "KEND_%=:\n\t"
      :
      : "s"(__builtin_amdgcn_kernarg_segment_ptr()), "s"(blockIdx.x), "v"(threadIdx.x),
        "s"((unsigned)(size_t)(__attribute__((address_space(3))) uint8_t*)lds)
      :
        "v2", "v3", "v4", "v5", "v6", "v7", "v8", "v9", "v10", "v11", "v12", "v13", "v14", "v15", "v16", "v17", "v18", "v19", "v20", "v21", "v22", "v23", "v24", "v25",
        "v26", "v27", "v28", "v29", "v30", "v31", "v32", "v33", "v34", "v35", "v36", "v37", "v38", "v39", "v40", "v41", "v42", "v43", "v44", "v45", "v46", "v47", "v48", "v49",
        "v50", "v51", "v52", "v53", "v54", "v55", "v56", "v57", "v58", "v59", "v60", "v61", "v62", "v63", "v64", "v65", "v66", "v67", "v68", "v69", "v70", "v71", "v72", "v73",
        "v74", "v75", "v76", "v77", "v78", "v79", "v80", "v81", "v82", "v83", "v84", "v85", "v86", "v87", "v88", "v89", "v90", "v91", "v92", "v93", "v94", "v95", "v96", "v97",
        "v98", "v99", "v100", "v101", "v102", "v103", "v104", "v105", "v106", "v107", "v108", "v109", "v110", "v111", "v112", "v113", "v114", "v115", "v116", "v117", "v118", "v119", "v120", "v121",
        "v122", "v123", "v124", "v125", "v126", "v127", "v128", "v129", "v130", "v131", "v132", "v133", "v134", "v135", "v136", "v137", "v138", "v139", "v140", "v141", "v142", "v143", "v144", "v145",
        "v146", "v147", "v148", "v149", "v150", "v151", "v152", "v153", "v154", "v155", "v156", "v157", "v158", "v159", "v160", "v161", "v162", "v163", "v164", "v165", "v166", "v167", "v168", "v169",
        "v170", "v171", "v172", "v173", "v174", "v175", "v176", "v177", "v178", "v179", "v180", "v181", "v182", "v183", "v184", "v185", "v186", "v187", "v188", "v189", "v190", "v191", "v192", "v193",
        "v194", "v195", "v196", "v197", "v198", "v199", "v200", "v201", "v202", "v203", "v204", "v205", "v206", "v207", "v208", "v209", "v210", "v211", "v212", "v213", "v214", "v215", "v216", "v217",
        "v218", "v219", "v220", "v221", "v222", "v223", "v224", "v225", "v226", "v227", "v228", "v229", "v230", "v231", "v232", "v233", "v234", "v235", "v236", "v237", "v238", "v239", "v240", "v241",
        "v242", "v243", "v244", "v245", "v246", "v247", "v248", "v249", "v250", "v251", "v252", "v253", "v254", "v255", "a0", "a1", "a2", "a3", "a4", "a5", "a6", "a7", "a8", "a9",
        "a10", "a11", "a12", "a13", "a14", "a15", "a16", "a17", "a18", "a19", "a20", "a21", "a22", "a23", "a24", "a25", "a26", "a27", "a28", "a29", "a30", "a31", "a32", "a33",
        "a34", "a35", "a36", "a37", "a38", "a39", "a40", "a41", "a42", "a43", "a44", "a45", "a46", "a47", "a48", "a49", "a50", "a51", "a52", "a53", "a54", "a55", "a56", "a57",
        "a58", "a59", "a60", "a61", "a62", "a63", "a64", "a65", "a66", "a67", "a68", "a69", "a70", "a71", "a72", "a73", "a74", "a75", "a76", "a77", "a78", "a79", "a80", "a81",
        "a82", "a83", "a84", "a85", "a86", "a87", "a88", "a89", "a90", "a91", "a92", "a93", "a94", "a95", "a96", "a97", "a98", "a99", "a100", "a101", "a102", "a103", "a104", "a105",
        "a106", "a107", "a108", "a109", "a110", "a111", "a112", "a113", "a114", "a115", "a116", "a117", "a118", "a119", "a120", "a121", "a122", "a123", "a124", "a125", "a126", "a127", "a128", "a129",
        "a130", "a131", "a132", "a133", "a134", "a135", "a136", "a137", "a138", "a139", "a140", "a141", "a142", "a143", "a144", "a145", "a146", "a147", "a148", "a149", "a150", "a151", "a152", "a153",
        "a154", "a155", "a156", "a157", "a158", "a159", "a160", "a161", "a162", "a163", "a164", "a165", "a166", "a167", "a168", "a169", "a170", "a171", "a172", "a173", "a174", "a175", "a176", "a177",
        "a178", "a179", "a180", "a181", "a182", "a183", "a184", "a185", "a186", "a187", "a188", "a189", "a190", "a191", "a192", "a193", "a194", "a195", "a196", "a197", "a198", "a199", "a200", "a201",
        "a202", "a203", "a204", "a205", "a206", "a207", "a208", "a209", "a210", "a211", "a212", "a213", "a214", "a215", "a216", "a217", "a218", "a219", "a220", "a221", "a222", "a223", "a224", "a225",
        "a226", "a227", "a228", "a229", "a230", "a231", "a232", "a233", "a234", "a235", "a236", "a237", "a238", "a239", "a240", "a241", "a242", "a243", "a244", "a245", "a246", "a247", "s8", "s9",
        "s10", "s11", "s12", "s13", "s14", "s15", "s16", "s17", "s18", "s19", "s20", "s21", "s22", "s23", "s24", "s25", "s26", "s27", "s28", "s29", "s30", "s31", "s33", "s34",
        "s35", "s36", "s37", "s38", "s39", "s40", "s41", "s42", "s43", "s44", "s45", "s46", "s47", "s48", "s49", "s50", "s51", "s52", "s53", "s54", "s55", "s56", "s57", "s58",
        "s59", "s60", "s61", "s62", "s63", "s64", "s65", "s66", "s67", "s68", "s69", "s70", "s71", "s72", "s73", "s74", "s75", "s76", "s77", "s78", "s79", "s80", "s81", "s82",
        "s83", "s84", "s85", "s86", "s87", "s88", "s89", "s90", "s91", "s92", "s93", "s94", "s95", "s96", "s97", "s98", "s99", "vcc", "scc", "memory"
  );
  __builtin_unreachable();
}

extern "C" void kernel_launch(int rank, int nranks, ncclComm_t comm, hipStream_t stream,
                              void* const* d_in, const int* in_sizes, int n_in,
                              void* d_out, int out_size, const int* in_dtypes, int out_dtype,
                              void* const* d_out_all, void* const* d_ws_all, size_t ws_bytes) {
  (void)comm; (void)in_dtypes; (void)out_dtype; (void)d_out_all;
  if (nranks != 8 || (unsigned)rank >= 8u || n_in != 4 || in_sizes[0] != 4096 * 512 || in_sizes[1] != 4096 * 4096 ||
      out_size != 512 * 4096 || ws_bytes < WS_END) {
    fprintf(stderr, "kernel_launch: unexpected problem shape\n");
    return;
  }
  Args a;
  a.x = (const uint8_t*)d_in[0]; a.w = (const uint8_t*)d_in[1];
  a.xown = a.x + (size_t)rank * 512 * 512;
  a.sx = (const float*)d_in[2]; a.sw = (const float*)d_in[3];
  a.out = (float*)d_out;
  a.myws = (uint8_t*)d_ws_all[rank];
  a.rank = (unsigned)rank; a.colrot = (unsigned)(((uintptr_t)a.w >> 7) & 7u);
  const char* ku[7] = {"0000000122221121111223333333", "0000000122221121111223333333", "0110000100111122222223333333", "0011000110011122222223333333", "0000000122221121111223333333", "0000000122221121111223333333", "0111000100011122222223333333"};
  const char* perm[4] = {"01234567", "01234567", "01234567", "01234567"};
  for (int rb = 0; rb < 7; ++rb)
    for (int wv = 0; wv < 4; ++wv)
      for (int nb = 0; nb < 4; ++nb) {
        a.tab[rb * 4 + wv][nb][0] = (uint8_t)(rank * 4 + wv);
        for (int u = 1; u < 8; ++u) {
          const int p = 4 * (perm[nb][u] - '1') + wv;
          a.tab[rb * 4 + wv][nb][u] = (uint8_t)(((rank + 1 + (p + rb) % 7) & 7) * 4 + (ku[rb][p] - '0'));
        }
      }
  const int toff[3] = {104,96,88}, qmap[12] = {0,4,8,1,5,9,2,6,10,3,7,11};
  for (int g = 0; g < 32; ++g) {
    const int t = g >> 3, x = g & 7;
    int li = 0, c = 0, role = 0;
    if (x < 4) {
      if (t < 3) { li = t * 2 + (x & 1); c = x >> 1; }
      else if (x == 1) { li = 6; c = 0; }
      else { role = 1; c = x; }
    } else {
      const int n = t * 4 + x - 4;
      if (n < 14) { li = n % 7; c = 2 + n / 7; } else if (n == 14) { li = 6; c = 1; } else { role = 1; c = 1; }
    }
    const int dst = (rank + 1 + li) & 7;
    for (int k = 0; k < 16; ++k) {
      const int i = k < 14 ? k : k - 4, q = i < 3 ? qmap[c * 3 + i] : 4 * i + c, p = (q >> 3) * 4 + (q & 3), rb = (34 - li - p) % 7;
      a.svc[g].item[k & 3][k >> 2] = (uint16_t)(q << 6 | ((q >> 2) & 1) << 5 | rb << 2 | (ku[rb][p] - '0'));
    }
    a.svc[g].peer = (uint8_t*)d_ws_all[dst] + ((size_t)rank << 20);
    a.svc[g].misc = (unsigned)(c << 4 | role << 6 | toff[x < 2 ? 0 : x < 4 ? 1 : 2] << 8); a.svc[g].src = (unsigned)dst << 18;
  }
  a2a_gemm<<<dim3(256), dim3(256), 0, stream>>>(a);
}
```
